```python
import numpy as np

kernel_name = "gpu_gemm_rs_m4096_k4096_n16384_w8a8_dqrelu"


ABSMAX_RTOL = 0.001

COMPUTE_DTYPE = "int8"
EPILOGUE = "dequant_relu"
IN_QUANT = "int8"
OUT_QUANT = None
IN_DTYPES = {"x": "int8", "w_mat": "int8",
             "scale_x": "float32", "scale_w": "float32"}
WORLD_SIZE = 8

def setup_inputs():

    x = np.random.randint(-127, 128, size=(4096, 4096)).astype(np.int8)
    w_mat = np.random.randint(-127, 128, size=(4096, 16384)).astype(np.int8)
    scale_x = (0.02 * (1.0 + np.random.rand(1))).astype(np.float32)
    scale_w = (0.02 * (1.0 + np.random.rand(1))).astype(np.float32)
    return {"x": x, "w_mat": w_mat, "scale_x": scale_x, "scale_w": scale_w}

def reference(x, w_mat, scale_x, scale_w):
    acc = x.astype(np.float64) @ w_mat.astype(np.float64)
    y = (acc.astype(np.float32) * (scale_x * scale_w)).astype(np.float32)
    return np.maximum(y, 0.0).astype(np.float32)
```

```cpp
#include <hip/hip_runtime.h>
#include <rccl/rccl.h>
#include <cstdio>
#include <cstdint>

#define HIP_CHECK(expr) do { hipError_t _e = (expr); if (_e != hipSuccess) { \
    fprintf(stderr, "HIP error %s at %s:%d\n", hipGetErrorString(_e), __FILE__, __LINE__); \
    return; } } while (0)

typedef unsigned u32;
typedef unsigned long long u64;
typedef unsigned u32x2 __attribute__((ext_vector_type(2)));
typedef unsigned u32x4 __attribute__((ext_vector_type(4)));
typedef int i32x4 __attribute__((ext_vector_type(4)));
typedef float f32x4 __attribute__((ext_vector_type(4)));
#define GAS __attribute__((address_space(1)))
#define LAS __attribute__((address_space(3)))

#define PROF 0
#define LNK_LEAD 1

constexpr int NR = 8;
constexpr int MM = 4096, KS = 512, NN = 16384, NSH = 2048;
constexpr u32 MSG = 16384;
constexpr u32 MiB = 1u << 20;
constexpr u32 XI_OFF = 0;
constexpr u32 WI_OFF = 16 * MiB;
constexpr u32 WSO_OFF = 24 * MiB;
constexpr u32 OSP_OFF = 32 * MiB;
constexpr u32 FL_OFF = 46 * MiB;
constexpr u32 FX_OFF = FL_OFF;
constexpr u32 FW_OFF = FL_OFF + 4096;
constexpr u32 LNK_OFF = FL_OFF + 8192;
constexpr u32 RDY_OFF = FL_OFF + 16384;
constexpr u32 OF_OFF = FL_OFF + 32768;
constexpr u32 OSR_OFF = 48 * MiB;
constexpr u32 PAR_BYTES = 76 * MiB;
constexpr u32 CNT_OFF = 2 * PAR_BYTES;
constexpr u32 TMO_OFF = CNT_OFF + 4096;
constexpr u32 PROF_OFF = CNT_OFF + 65536;
constexpr u32 WS_NEED = PROF_OFF + 256 * 8 * 16 * 8;
static_assert(OSP_OFF + 224u * 16u * 4096u <= FL_OFF, "");
static_assert(OF_OFF + 8u * 4u * 8u * 4u * 4u * 8u <= OSR_OFF, "");
static_assert(OSR_OFF + 224u * 16u * 8192u <= PAR_BYTES, "");

constexpr u32 SLOT = 49152;
constexpr u32 AUX_L = 3 * SLOT;
constexpr u32 SYNC_L = 163840 - 512;
constexpr u32 FULL_L = SYNC_L, FREE_L = SYNC_L + 16, ABORT_L = SYNC_L + 32;
constexpr u32 LDS_BYTES = 163840;
constexpr u32 PK_STRIDE = 12288;
constexpr u64 DEADLINE = 50000000ull;
constexpr u32 RAW_MODE = 1u << 20;

struct Args {
  const char* x;
  const char* w;
  const float* sx;
  const float* sw;
  char* out;
  char* ws[NR];
  int rank;
};

__device__ __forceinline__ u64 rtime() { return __builtin_amdgcn_s_memrealtime(); }
__device__ __forceinline__ u32 rfl(u32 v) { return __builtin_amdgcn_readfirstlane(v); }
__device__ __forceinline__ u32 rdl(u32 v, int l) { return __builtin_amdgcn_readlane(v, l); }
__device__ __forceinline__ u32 mbcnt(u64 m) {
  return __builtin_amdgcn_mbcnt_hi((u32)(m >> 32), __builtin_amdgcn_mbcnt_lo((u32)m, 0u));
}
#define VM_WAIT(n) asm volatile("s_waitcnt vmcnt(" #n ")" ::: "memory")
#define LGKM_WAIT0() do { asm volatile("" ::: "memory"); __builtin_amdgcn_s_waitcnt(0xC07F); asm volatile("" ::: "memory"); } while (0)

template <int IMM>
__device__ __forceinline__ void st128(const char* base, u32 off, u32x4 v) {
  asm volatile("global_store_dwordx4 %0, %1, %2 offset:%3 sc0 sc1\n\ts_nop 1" :: "v"(off), "v"(v), "s"(base), "n"(IMM) : "memory");
}
__device__ __forceinline__ void st64(const char* base, u32 off, u32x2 v) {
  asm volatile("global_store_dwordx2 %0, %1, %2 sc0 sc1\n\ts_nop 1" :: "v"(off), "v"(v), "s"(base) : "memory");
}
__device__ __forceinline__ void st32(const char* base, u32 off, u32 v) {
  asm volatile("global_store_dword %0, %1, %2 sc0 sc1\n\ts_nop 1" :: "v"(off), "v"(v), "s"(base) : "memory");
}
__device__ __forceinline__ u32 ld32w(const char* base, u32 off) {
  u32 v;
  asm volatile("global_load_dword %0, %1, %2 sc0 sc1\n\ts_waitcnt vmcnt(0)" : "=&v"(v) : "v"(off), "s"(base) : "memory");
  return v;
}
__device__ __forceinline__ u32x2 ld64w(const char* base, u32 off) {
  u32x2 v;
  asm volatile("global_load_dwordx2 %0, %1, %2 sc0 sc1\n\ts_waitcnt vmcnt(0)" : "=&v"(v) : "v"(off), "s"(base) : "memory");
  return v;
}
__device__ __forceinline__ void st_out(const char* base, u32 off, f32x4 v) {
  asm volatile("global_store_dwordx4 %0, %1, %2 sc0 sc1\n\ts_nop 1" :: "v"(off), "v"(v), "s"(base) : "memory");
}
__device__ __forceinline__ void glds4k(u32 lds, const char* base, u32 off) {
  asm volatile("s_mov_b32 m0, %0\n\ts_nop 0\n\t"
               "global_load_lds_dwordx4 %1, %2 sc0 sc1\n\t"
               "global_load_lds_dwordx4 %1, %2 offset:1024 sc0 sc1\n\t"
               "global_load_lds_dwordx4 %1, %2 offset:2048 sc0 sc1\n\t"
               "global_load_lds_dwordx4 %1, %2 offset:3072 sc0 sc1"
               :: "s"(lds), "v"(off), "s"(base) : "memory");
}
template <int IMM>
__device__ __forceinline__ void glds1k(u32 lds, const char* base, u32 off) {
  asm volatile("s_mov_b32 m0, %0\n\ts_nop 0\n\tglobal_load_lds_dwordx4 %1, %2 offset:%3 sc0 sc1"
               :: "s"(lds), "v"(off), "s"(base), "n"(IMM) : "memory");
}
__device__ __forceinline__ void ld8kw(u32x4 (&R)[8], const char* base, u32 off) {
  asm volatile("global_load_dwordx4 %0, %8, %10 sc0 sc1\n\tglobal_load_dwordx4 %1, %8, %10 offset:1024 sc0 sc1\n\t"
               "global_load_dwordx4 %2, %8, %10 offset:2048 sc0 sc1\n\tglobal_load_dwordx4 %3, %8, %10 offset:3072 sc0 sc1\n\t"
               "global_load_dwordx4 %4, %9, %10 sc0 sc1\n\tglobal_load_dwordx4 %5, %9, %10 offset:1024 sc0 sc1\n\t"
               "global_load_dwordx4 %6, %9, %10 offset:2048 sc0 sc1\n\tglobal_load_dwordx4 %7, %9, %10 offset:3072 sc0 sc1\n\t"
               "s_waitcnt vmcnt(0)"
               : "=&v"(R[0]), "=&v"(R[1]), "=&v"(R[2]), "=&v"(R[3]), "=&v"(R[4]), "=&v"(R[5]), "=&v"(R[6]), "=&v"(R[7])
               : "v"(off), "v"(off + 4096u), "s"(base) : "memory");
}
#define PL(r, o) "global_load_dwordx4 %[" #r "], %[vs], %[pp] offset:" #o " sc0 sc1\n\t"
#define PS(r, o) "global_store_dwordx4 %[vd], %[" #r "], %[wp] offset:" #o " sc0 sc1\n\t"
#define PLA PL(a0, 0) PL(a1, 1024) PL(a2, 2048) PL(a3, 3072) "v_add_u32 %[vs], 0x1000, %[vs]\n\t"
#define PLB PL(b0, 0) PL(b1, 1024) PL(b2, 2048) PL(b3, 3072) "v_add_u32 %[vs], 0x1000, %[vs]\n\t"
#define PSA PS(a0, 0) PS(a1, 1024) PS(a2, 2048) PS(a3, 3072) "v_add_u32 %[vd], 0x1000, %[vd]\n\t"
#define PSB PS(b0, 0) PS(b1, 1024) PS(b2, 2048) PS(b3, 3072) "v_add_u32 %[vd], 0x1000, %[vd]\n\t"
__device__ __forceinline__ void pull_batch(u32 k, u32 k1, u32 klnk, const char* pp, const char* wp, u32 so, u32 dof, u32 lf, u32 vo, u32 call) {
  u32x4 a0, a1, a2, a3, b0, b1, b2, b3;
  u32 vs, vd, t;
  u64 fm, t2;
  asm volatile(
      "v_readlane_b32 %[t], %[so], %[k]\n\t"
      "s_mov_b64 %[fm], 0\n\ts_nop 0\n\t"
      "v_add_u32 %[vs], %[t], %[vo]\n\t"
      PLA PLB
      "v_readlane_b32 %[t], %[dof], %[k]\n\ts_nop 1\n\t"
      "v_add_u32 %[vd], %[t], %[vo]\n\t"
      "s_waitcnt vmcnt(4)\n\t"
      "s_branch 2f\n"
      "1:\n\t"
      PLB
      "v_readlane_b32 %[t], %[dof], %[k]\n\ts_nop 1\n\t"
      "v_add_u32 %[vd], %[t], %[vo]\n\t"
      "s_waitcnt vmcnt(8)\n"
      "2:\n\t"
      PSA PLA
      "s_waitcnt vmcnt(8)\n\t"
      PSB
      "s_cmp_eq_u32 %[k], %[klnk]\n\t"
      "s_cselect_b64 %[t2], %[top], 0\n\t"
      "s_or_b64 exec, %[fm], %[t2]\n\t"
      "global_store_dword %[lf], %[call], %[wp] sc0 sc1\n\t"
      "s_mov_b64 exec, -1\n\t"
      "s_lshl_b64 %[fm], 1, %[k]\n\t"
      PLB
      "s_waitcnt vmcnt(8)\n\t"
      PSA
      "s_add_u32 %[k], %[k], 1\n\t"
      "s_cmp_eq_u32 %[k], %[k1]\n\t"
      "s_cbranch_scc1 3f\n\t"
      "v_readlane_b32 %[t], %[so], %[k]\n\ts_nop 1\n\t"
      "v_add_u32 %[vs], %[t], %[vo]\n\t"
      PLA
      "s_waitcnt vmcnt(8)\n\t"
      PSB
      "s_branch 1b\n"
      "3:\n\t"
      "s_waitcnt vmcnt(4)\n\t"
      PSB
      "s_waitcnt vmcnt(0)\n\t"
      "s_mov_b64 exec, %[fm]\n\t"
      "global_store_dword %[lf], %[call], %[wp] sc0 sc1\n\t"
      "s_mov_b64 exec, -1\n\t"
      "s_nop 1"
      : [a0] "=&v"(a0), [a1] "=&v"(a1), [a2] "=&v"(a2), [a3] "=&v"(a3), [b0] "=&v"(b0), [b1] "=&v"(b1), [b2] "=&v"(b2), [b3] "=&v"(b3),
        [vs] "=&v"(vs), [vd] "=&v"(vd), [t] "=&s"(t), [fm] "=&s"(fm), [t2] "=&s"(t2), [k] "+s"(k)
      : [k1] "s"(k1), [klnk] "s"(klnk), [pp] "s"(pp), [wp] "s"(wp), [so] "v"(so), [dof] "v"(dof), [lf] "v"(lf), [vo] "v"(vo),
        [call] "v"(call), [top] "s"(1ull << 63)
      : "memory", "scc");
}

__device__ __forceinline__ int rtmap(int rank, int i) {
  return i < 28 ? ((rank + 1 + i % 7) & 7) * 4 + i / 7 : rank * 4 + (i - 28);
}

struct Ctx {
  const char* wsme;
  u32 call;
  u64 t0;
  volatile LAS u32* abortw;
  __device__ __forceinline__ bool expired() const {
    const u64 dt = rtime() - t0;
    if (dt < 100000ull) return false;
    if (rfl(ld32w(wsme, TMO_OFF)) == call || dt > DEADLINE) {
      st32(wsme, TMO_OFF, call);
      *abortw = 1u;
      return true;
    }
    return false;
  }
  __device__ __forceinline__ bool expired_lds() const {
    const u64 dt = rtime() - t0;
    if (dt < 100000ull) return false;
    if (rfl(*abortw) != 0u) return true;
    if (dt > DEADLINE) { *abortw = 1u; return true; }
    return false;
  }
};

#define STAMP(k) do { } while (0)

__global__ void __launch_bounds__(512) gemm_rs_kernel(Args a) {
  extern __shared__ __attribute__((aligned(16))) char lds_raw[];
  LAS char* L8 = (LAS char*)lds_raw;
  const int lane = threadIdx.x & 63;
  const int wave = (int)rfl(threadIdx.x >> 6);
  const int b = blockIdx.x;
  const int rank = a.rank & 7;
  char* wsme = a.ws[rank];
  const u64 t0 = rtime();
  const int ti = b >> 3, ct = b & 7;
  const int rt = rtmap(rank, ti);

  u32x4 sv[16];
  int s_kp = 0, s_a = 0, s_dst = 0;
  const bool stage_w = wave < 2, stage_x = wave == 2 && b < 128;
  if (stage_w) {
    const int id = b * 2 + wave;
    s_kp = id >> 7; s_dst = (rank + 1 + ((id >> 4) & 7)) & 7; s_a = id & 15;
    const char* src = a.w + (size_t)(s_kp * 128 + (lane >> 3) * 16) * NN + s_dst * NSH + s_a * 128 + (lane & 7) * 16;
#pragma unroll
    for (int i = 0; i < 16; ++i) sv[i] = *(const GAS u32x4*)(src + (size_t)i * NN);
  } else if (stage_x) {
    s_kp = b & 3; s_a = ((b >> 2) & 7) * 4 + (b >> 5);
    const char* src = a.x + (size_t)(s_a * 128 + (lane >> 3)) * KS + s_kp * 128;
#pragma unroll
    for (int i = 0; i < 16; ++i) {
      const int row = 8 * i + (lane >> 3);
      sv[i] = *(const GAS u32x4*)(src + i * 8 * KS + (((lane & 7) ^ ((row >> 1) & 7)) << 4));
    }
  }
  u32 call;
  {
    const char* cp = wsme + CNT_OFF + b * 4;
    asm volatile("s_load_dword %0, %1, 0x0\n\ts_waitcnt lgkmcnt(0)" : "=s"(call) : "s"(cp) : "memory");
    call += 1u;
  }
  if (wave == 7) { ((LAS u32*)(L8 + SYNC_L))[lane] = 0u; ((LAS u32*)(L8 + SYNC_L))[lane + 64] = 0u; }
  LGKM_WAIT0();
  __builtin_amdgcn_s_barrier();
  asm volatile("" ::: "memory");
  const u32 par = (call & 1u) * PAR_BYTES;
  const char* wp = wsme + par;
  volatile LAS u32* FULL = (volatile LAS u32*)(L8 + FULL_L);
  volatile LAS u32* FREE = (volatile LAS u32*)(L8 + FREE_L);
  Ctx cx{wsme, call, t0, (volatile LAS u32*)(L8 + ABORT_L)};
  STAMP(0);
  if (wave >= 4) __builtin_amdgcn_s_waitcnt(0x0F70);

  if (wave < 4) {
    if (stage_w) {
      u32x4 U[16];
#pragma unroll
      for (int g = 0; g < 4; ++g)
#pragma unroll
        for (int q = 0; q < 4; ++q) {
          const u32 r0 = sv[4 * g][q], r1 = sv[4 * g + 1][q], r2 = sv[4 * g + 2][q], r3 = sv[4 * g + 3][q];
          const u32 p0 = __builtin_amdgcn_perm(r1, r0, 0x05010400u), p1 = __builtin_amdgcn_perm(r1, r0, 0x07030602u);
          const u32 p2 = __builtin_amdgcn_perm(r3, r2, 0x05010400u), p3 = __builtin_amdgcn_perm(r3, r2, 0x07030602u);
          U[4 * q + 0][g] = __builtin_amdgcn_perm(p2, p0, 0x05040100u);
          U[4 * q + 1][g] = __builtin_amdgcn_perm(p2, p0, 0x07060302u);
          U[4 * q + 2][g] = __builtin_amdgcn_perm(p3, p1, 0x05040100u);
          U[4 * q + 3][g] = __builtin_amdgcn_perm(p3, p1, 0x07060302u);
        }
      const u32 mo = (s_dst == rank ? WI_OFF : WSO_OFF) + (u32)((s_dst * 16 + s_a) * 4 + s_kp) * MSG;
#pragma unroll
      for (int g = 0; g < 4; ++g) {
        const u32 off = mo + g * 4096 + ((lane ^ (g << 2)) << 4);
        st128<0>(wp, off, U[4 * g]); st128<1024>(wp, off, U[4 * g + 1]);
        st128<2048>(wp, off, U[4 * g + 2]); st128<3072>(wp, off, U[4 * g + 3]);
      }
      VM_WAIT(0);
      if (lane == 0) {
        if (s_dst == rank) st32(wp, FW_OFF + s_a * 128 + (rank * 4 + s_kp) * 4, call);
        else st32(a.ws[s_dst & 7] + par, RDY_OFF + rank * 1024 + (128 + s_a * 4 + s_kp) * 4, call);
      }
    } else if (stage_x) {
      const u32 mo = XI_OFF + (u32)((rank * 32 + s_a) * 4 + s_kp) * MSG;
#pragma unroll
      for (int g = 0; g < 4; ++g) {
        const u32 off = mo + g * 4096 + (lane << 4);
        st128<0>(wp, off, sv[4 * g]); st128<1024>(wp, off, sv[4 * g + 1]);
        st128<2048>(wp, off, sv[4 * g + 2]); st128<3072>(wp, off, sv[4 * g + 3]);
      }
      VM_WAIT(0);
      if (lane == 0) {
#pragma unroll
        for (int d = 1; d < 8; ++d) st32(a.ws[(rank + d) & 7] + par, RDY_OFF + rank * 1024 + (s_a * 4 + s_kp) * 4, call);
        st32(wp, FX_OFF + s_a * 128 + (rank * 4 + s_kp) * 4, call);
      }
    } else if (wave == 3 && lane == 0) {
      st32(wsme, CNT_OFF + b * 4, call);
    }
    STAMP(1);
    const int wr = wave >> 1, wc = wave & 1;
    const int c = lane & 15, kq = lane >> 4;
    const u32 a0 = wr * 8192 + c * 128 + ((kq ^ (c >> 1)) << 4);
    const u32 b0 = 16384 + wc * 16384 + (c & 3) * 4096 + (((kq * 8 + (c >> 2)) ^ ((c & 3) << 2)) << 4);
    i32x4 acc[4][2][4];
#pragma unroll
    for (int mt = 0; mt < 4; ++mt)
#pragma unroll
      for (int G = 0; G < 2; ++G)
#pragma unroll
        for (int t = 0; t < 4; ++t) acc[mt][G][t] = i32x4{0, 0, 0, 0};
    i32x4 A0[4], B0[2][4], A1[4], B1[2][4];
    auto load_half = [&](i32x4 (&A)[4], i32x4 (&B)[2][4], u32 sb, int kh) {
      LAS char* pa = L8 + ((a0 ^ (kh * 64)) + sb);
      LAS char* pb0 = L8 + (b0 + sb + kh * 512);
      LAS char* pb1 = L8 + ((b0 ^ 64) + sb + kh * 512);
      A[0] = *(LAS i32x4*)(pa);
#pragma unroll
      for (int t = 0; t < 4; ++t) B[0][t] = *(LAS i32x4*)(pb0 + t * 1024);
#pragma unroll
      for (int t = 0; t < 4; ++t) B[1][t] = *(LAS i32x4*)(pb1 + t * 1024);
#pragma unroll
      for (int mt = 1; mt < 4; ++mt) A[mt] = *(LAS i32x4*)(pa + mt * 2048);
    };
    auto mma_half = [&](i32x4 (&A)[4], i32x4 (&B)[2][4]) {
#pragma unroll
      for (int mt = 0; mt < 4; ++mt)
#pragma unroll
        for (int G = 0; G < 2; ++G)
#pragma unroll
          for (int t = 0; t < 4; ++t)
            acc[mt][G][t] = __builtin_amdgcn_mfma_i32_16x16x64_i8(A[mt], B[G][t], acc[mt][G][t], 0, 0, 0);
    };
    bool dead = false;
    while (rfl(FULL[0]) < 1u) { __builtin_amdgcn_s_sleep(1); if (cx.expired_lds()) { dead = true; break; } }
    STAMP(2);
    if (!dead) {
      asm volatile("" ::: "memory");
      load_half(A0, B0, 0, 0);
      u32 slot = 0;
      for (int s = 0; s < 32; ++s) {
        const u32 nslot = slot == 2 ? 0 : slot + 1;
        load_half(A1, B1, slot * SLOT, 1);
        u32 fl = FULL[nslot];
        __builtin_amdgcn_sched_barrier(0);
        mma_half(A0, B0);
        __builtin_amdgcn_sched_barrier(0);
        LGKM_WAIT0();
        if (lane == 0) FREE[wave] = (u32)(s + 1);
        if (s < 31) {
          while (rfl(fl) < (u32)(s + 2)) {
            if (cx.expired_lds()) { dead = true; break; }
            fl = FULL[nslot];
          }
          if (dead) break;
          asm volatile("" ::: "memory");
          load_half(A0, B0, nslot * SLOT, 0);
        }
        __builtin_amdgcn_sched_barrier(0);
        mma_half(A1, B1);
        __builtin_amdgcn_sched_barrier(0);
        slot = nslot;
      }
    }
    STAMP(3);
    float scale = __builtin_bit_cast(float, rfl(__builtin_bit_cast(u32, *(const GAS float*)a.sx * *(const GAS float*)a.sw)));
    if (scale < 0.f) {
      scale = -scale;
#pragma unroll
      for (int mt = 0; mt < 4; ++mt)
#pragma unroll
        for (int G = 0; G < 2; ++G)
#pragma unroll
          for (int t = 0; t < 4; ++t) acc[mt][G][t] = -acc[mt][G][t];
    }
    if (ti >= 28) {
      const u32 ob = (u32)(((ti - 28) * 128 + wr * 64 + kq * 4) * NN + rank * NSH + ct * 256 + wc * 128 + 4 * c) * 4u;
      if (dead) {
        const float nan = __builtin_nanf("");
        st_out(a.out, ob, f32x4{nan, nan, nan, nan});
      } else {
#pragma unroll
        for (int mt = 0; mt < 4; ++mt)
#pragma unroll
          for (int G = 0; G < 2; ++G)
#pragma unroll
            for (int r = 0; r < 4; ++r) {
              f32x4 y;
#pragma unroll
              for (int t = 0; t < 4; ++t) y[t] = (float)max(acc[mt][G][t][r], 0) * scale;
              st_out(a.out, ob + (u32)((mt * 16 + r) * NN + G * 64) * 4u, y);
            }
      }
    } else if (!dead) {
      for (;;) {
        const u32x4 f = *(volatile LAS u32x4*)FREE;
        const u32 mn = rfl(min(min(f[0], f[1]), min(f[2], f[3])));
        if (mn >= 30u) break;
        if (cx.expired_lds()) { dead = true; break; }
      }
      if (!dead) {
        asm volatile("" ::: "memory");
        LAS char* C8 = L8 + 2 * SLOT + wave * PK_STRIDE;
        const u32 slot0 = (u32)(b * 4 + wave) * 4u;
        u32 infos = 0;
#pragma unroll
        for (int mt = 0; mt < 4; ++mt) {
          u32 m = 0, cnt = 0, nexc = 0;
#pragma unroll
          for (int G = 0; G < 2; ++G)
#pragma unroll
            for (int r = 0; r < 4; ++r) {
              u32 idx[4];
#pragma unroll
              for (int t = 0; t < 4; ++t) {
                const int v = acc[mt][G][t][r];
                const bool pos = v > 0;
                const u64 bal = __builtin_amdgcn_ballot_w64(pos);
                idx[t] = mbcnt(bal) + cnt;
                *(LAS int*)(C8 + (pos ? idx[t] * 4u : 8192u + lane * 4u)) = v;
                m = m + m + (pos ? 1u : 0u);
                cnt += (u32)__builtin_popcountll(bal);
              }
              const int mx = max(max(acc[mt][G][0][r], acc[mt][G][1][r]), max(acc[mt][G][2][r], acc[mt][G][3][r]));
              if (__builtin_amdgcn_ballot_w64(mx >= (1 << 20)) != 0ull) {
#pragma unroll
                for (int t = 0; t < 4; ++t) {
                  const int v = acc[mt][G][t][r];
                  const bool big = v >= (1 << 20);
                  const u64 be = __builtin_amdgcn_ballot_w64(big);
                  const u32 e = min(nexc + mbcnt(be), 255u);
                  *(LAS u32*)(C8 + (big ? 8448u + e * 4u : 8192u + lane * 4u)) = ((u32)v & 0xFFF00000u) | idx[t];
                  nexc += (u32)__builtin_popcountll(be);
                }
              }
            }
          const u32 nch = (cnt + 7u) >> 3;
          u32 info;
          if (nch + nexc <= 192u) {
            const u32 so = OSP_OFF + (slot0 + mt) * 4096u;
            st32(wp, so + lane * 4, m);
#pragma unroll
            for (int i = 0; i < 3; ++i) {
              const u32 ch = lane + 64 * i;
              if (64u * i < nch + nexc) {
                u32 d = 0;
                if (ch < nch) {
                  const u32x4 q0 = *(LAS u32x4*)(C8 + ch * 32), q1 = *(LAS u32x4*)(C8 + ch * 32 + 16);
                  u32x4 lo;
                  lo[0] = __builtin_amdgcn_perm(q0[1], q0[0], 0x05040100u); lo[1] = __builtin_amdgcn_perm(q0[3], q0[2], 0x05040100u);
                  lo[2] = __builtin_amdgcn_perm(q1[1], q1[0], 0x05040100u); lo[3] = __builtin_amdgcn_perm(q1[3], q1[2], 0x05040100u);
                  const u32 ne = __builtin_amdgcn_perm(q0[2], q0[0], 0x0c0c0602u) | __builtin_amdgcn_perm(q1[2], q1[0], 0x06020c0cu);
                  const u32 no = __builtin_amdgcn_perm(q0[3], q0[1], 0x0c0c0602u) | __builtin_amdgcn_perm(q1[3], q1[1], 0x06020c0cu);
                  d = (ne & 0x0F0F0F0Fu) | ((no << 4) & 0xF0F0F0F0u);
                  st128<256>(wp, so + ch * 16, lo);
                } else if (ch < nch + nexc) {
                  d = *(LAS u32*)(C8 + 8448u + (ch - nch) * 4u);
                }
                if (ch < nch + nexc) st32(wp, so + 256u + nch * 16u + ch * 4u, d);
              }
            }
            info = cnt | (nexc << 12);
          } else {
            const u32 so = OSR_OFF + (slot0 + mt) * 8192u + lane * 16;
#pragma unroll
            for (int G = 0; G < 2; ++G)
#pragma unroll
              for (int r = 0; r < 4; ++r) {
                u32x4 y;
#pragma unroll
                for (int t = 0; t < 4; ++t) y[t] = (u32)acc[mt][G][t][r];
                st128<0>(wp, so + (G * 4 + r) * 1024, y);
              }
            info = RAW_MODE;
          }
          if (lane == mt) infos = info;
        }
        VM_WAIT(0);
        const int owner = rt >> 2, j = rt & 3;
        if (lane < 4) st64(a.ws[owner & 7] + par, OF_OFF + (u32)((((rank * 4 + j) * 8 + ct) * 4 + wave) * 4 + lane) * 8u, u32x2{call, infos});
      }
    }
    STAMP(4);
  } else if (wave < 6) {
    __builtin_amdgcn_s_setprio(2);
    const int Lp = wave - 4;
    const int which = lane >> 4, idx = lane & 15;
    const u32 fo = (which == 0 ? FX_OFF + rt * 128 : FW_OFF + (2 * ct + (which == 2 ? 1 : 0)) * 128) + ((2 * (idx >> 2) + Lp) * 4 + (idx & 3)) * 4;
    u32 done = 0, ready = 0;
    u32 slot = Lp;
    const u32 vo = lane * 16;
    for (int n = 0; n < 16; ++n) {
      const u32 s = 2 * n + Lp;
      bool dead = false;
      while ((ready & ~done) == 0u) {
        const u32 v = ld32w(wp, fo);
        const u64 bal = __builtin_amdgcn_ballot_w64(v == call);
        ready = (u32)bal & (u32)(bal >> 16) & (u32)(bal >> 32) & 0xFFFFu;
        if ((ready & ~done) == 0u) {
          __builtin_amdgcn_s_sleep(4);
          if (cx.expired()) { dead = true; break; }
        }
      }
      if (!dead && s >= 3) {
        for (;;) {
          const u32x4 f = *(volatile LAS u32x4*)FREE;
          const u32 mn = rfl(min(min(f[0], f[1]), min(f[2], f[3])));
          if (mn + 2u >= s) break;
          __builtin_amdgcn_s_sleep(1);
          if (cx.expired_lds()) { dead = true; break; }
        }
      }
      if (dead) break;
      const u32 pi = (u32)__builtin_ctz(ready & ~done);
      done |= 1u << pi;
      const u32 src = 2 * (pi >> 2) + Lp, kp = pi & 3;
      const char* xa = wp + (XI_OFF + ((src * 32 + rt) * 4 + kp) * MSG);
      const char* wa = wp + (WI_OFF + ((src * 16 + 2 * ct) * 4 + kp) * MSG);
      const u32 lb = slot * SLOT;
#pragma unroll
      for (int g = 0; g < 4; ++g) glds4k(lb + g * 4096, xa, vo + g * 4096);
#pragma unroll
      for (int g = 0; g < 4; ++g) glds4k(lb + 16384 + g * 4096, wa, vo + g * 4096);
#pragma unroll
      for (int g = 0; g < 4; ++g) glds4k(lb + 32768 + g * 4096, wa, vo + 4 * MSG + g * 4096);
      VM_WAIT(0);
      if (lane == 0) FULL[slot] = s + 1;
      slot = slot == 0 ? 2 : slot - 1;
      if (n == 0 || n == 15) STAMP(n == 0 ? 1 : 2);
    }
  } else if (wave == 6) {
    if (b >= 200) {
      __builtin_amdgcn_s_setprio(3);
      const int l = (b - 200) % 7, sub = (b - 200) / 7;
      const int p = (rank + 1 + l) & 7;
      const char* pp = a.ws[p] + par;
      u32 so, dof, lf, rf;
      {
        const int m = min(lane, 23) * 8 + sub;
        int kp, i = -1, ch = 0;
        if (m < 92) { kp = m / 23; const int e = m % 23; if (e < 7) i = e; else ch = e - 7; }
        else if (m < 176) { const int m2 = (m - 92) % 28; kp = m2 / 7; i = 7 + ((m - 92) / 28) * 7 + m2 % 7; }
        else { kp = (m - 176) >> 2; i = 28 + ((m - 176) & 3); }
        if (i >= 0) {
          const int r2 = rtmap(rank, i);
          so = dof = XI_OFF + (u32)((p * 32 + r2) * 4 + kp) * MSG;
          lf = FX_OFF + r2 * 128 + (p * 4 + kp) * 4;
          rf = RDY_OFF + p * 1024 + (r2 * 4 + kp) * 4;
        } else {
          so = WSO_OFF + (u32)((rank * 16 + ch) * 4 + kp) * MSG;
          dof = WI_OFF + (u32)((p * 16 + ch) * 4 + kp) * MSG;
          lf = FW_OFF + ch * 128 + (p * 4 + kp) * 4;
          rf = RDY_OFF + p * 1024 + (128 + ch * 4 + kp) * 4;
        }
      }
      if (lane == 63) lf = LNK_OFF + l * 128 + sub * 4;
      {
        u32 k = 0;
        while (k < 24u) {
          const u32 v = ld32w(wp, rf);
          const u32 rdy = (u32)__builtin_amdgcn_ballot_w64(v == call) & 0xFFFFFFu;
          const u32 k1 = k + (u32)__builtin_ctz(~(rdy >> k));
          if (k1 == k) { if (cx.expired()) break; continue; }
          if (k == 0) STAMP(1);
          pull_batch(k, k1, 24 - LNK_LEAD, pp, wp, so, dof, lf, lane * 16, call);
          k = k1;
        }
        STAMP(2);
      }
    }
  } else {
    if (b < 224) {
      __builtin_amdgcn_s_setprio(3);
      const int l = b % 7, uu = b / 7;
      const int oct = uu >> 2, owv = uu & 3;
      const int p = (rank + 1 + l) & 7;
      const char* pp = a.ws[p] + par;
      const int li = (rank - p - 1) & 7;
      const float scale = __builtin_bit_cast(float, rfl(__builtin_bit_cast(u32, *(const GAS float*)a.sx * *(const GAS float*)a.sw)) & 0x7FFFFFFFu);
      LAS char* I8 = L8 + AUX_L + 8192;
      if (lane == 0) *(LAS u32*)(I8 + 6144) = 0u;
      const int c = lane & 15, kq = lane >> 4;
      const u32 ob = (u32)(((owv >> 1) * 64 + kq * 4) * NN + p * NSH + oct * 256 + (owv & 1) * 128 + 4 * c) * 4u;
      const u32 fo = OF_OFF + (u32)((((p * 4 + ((lane >> 2) & 3)) * 8 + oct) * 4 + owv) * 4 + (lane & 3)) * 8u;
      u32x2 fl = u32x2{0u, 0u};
      bool dead = false;
      for (;;) {
        const u32 v = ld32w(wp, LNK_OFF + l * 128 + (lane & 7) * 4);
        if (__builtin_amdgcn_ballot_w64(v == call) == ~0ull) break;
        __builtin_amdgcn_s_sleep(8);
        if (cx.expired()) { dead = true; break; }
      }
      STAMP(1);
      auto ensure = [&](int n) -> bool {
        while (rdl(fl[0], n) != call) {
          fl = ld64w(wp, fo);
          if (rdl(fl[0], n) != call) {
            __builtin_amdgcn_s_sleep(4);
            if (cx.expired()) return false;
          }
        }
        return true;
      };
      auto slot_of = [&](int n) -> u32 { return (u32)(((((n >> 2) * 7 + li) * 8 + oct) * 4 + owv) * 4 + (n & 3)); };
      auto issue = [&](int n, u32 info) {
        const u32 cnt = info & 0xFFFu, nexc = (info >> 12) & 0xFFu, nch = (cnt + 7u) >> 3;
        const u32 len = 256u + nch * 20u + nexc * 4u;
        const u32 vo = lane * 16;
        const u32 so = OSP_OFF + slot_of(n) * 4096u + vo;
        const u32 lb = AUX_L + (n & 1) * 4096;
        if (vo < len) glds1k<0>(lb, pp, so);
        if (vo + 1024u < len) glds1k<1024>(lb, pp, so);
        if (vo + 2048u < len) glds1k<2048>(lb, pp, so);
        if (vo + 3072u < len) glds1k<3072>(lb, pp, so);
      };
      int n = 0;
      u32 info_next = 0;
      if (!dead) dead = !ensure(0);
      if (!dead) {
        info_next = rdl(fl[1], 0);
        if (!(info_next & RAW_MODE)) issue(0, info_next);
        VM_WAIT(0);
        for (; n < 16; ++n) {
          const u32 info = info_next;
          VM_WAIT(8);
          if (n < 15) {
            if (!ensure(n + 1)) dead = true;
            else { info_next = rdl(fl[1], n + 1); if (!(info_next & RAW_MODE)) issue(n + 1, info_next); }
          }
          const u32 on = ob + (u32)(((n >> 2) * 128 + (n & 3) * 16) * NN) * 4u;
          if (info & RAW_MODE) {
            const u32 so = OSR_OFF + slot_of(n) * 8192u + lane * 16;
            u32x4 R[8];
            ld8kw(R, pp, so);
#pragma unroll
            for (int g = 0; g < 8; ++g) {
              const u32x4 q = R[g];
              f32x4 y;
#pragma unroll
              for (int t = 0; t < 4; ++t) y[t] = (float)max((int)q[t], 0) * scale;
              st_out(a.out, on + (u32)((g & 3) * NN + (g >> 2) * 64) * 4u, y);
            }
          } else {
            const u32 cnt = info & 0xFFFu, nexc = (info >> 12) & 0xFFu, nch = (cnt + 7u) >> 3;
            LAS char* Bl = L8 + AUX_L + (n & 1) * 4096;
            u32 m = *(LAS u32*)(Bl + lane * 4);
            u32 d[3];
#pragma unroll
            for (int i = 0; i < 3; ++i) {
              const u32 ch = lane + 64 * i;
              d[i] = 0;
              if (64u * i < nch + nexc) {
                if (ch < nch + nexc) d[i] = *(LAS u32*)(Bl + 256u + nch * 16u + ch * 4u);
                if (ch < nch) {
                  const u32x4 lo = *(LAS u32x4*)(Bl + 256u + ch * 16u);
                  const u32 ne = d[i] & 0x0F0F0F0Fu, no = (d[i] >> 4) & 0x0F0F0F0Fu;
                  u32x4 q0, q1;
                  q0[0] = __builtin_amdgcn_perm(ne, lo[0], 0x0c040100u); q0[1] = __builtin_amdgcn_perm(no, lo[0], 0x0c040302u);
                  q0[2] = __builtin_amdgcn_perm(ne, lo[1], 0x0c050100u); q0[3] = __builtin_amdgcn_perm(no, lo[1], 0x0c050302u);
                  q1[0] = __builtin_amdgcn_perm(ne, lo[2], 0x0c060100u); q1[1] = __builtin_amdgcn_perm(no, lo[2], 0x0c060302u);
                  q1[2] = __builtin_amdgcn_perm(ne, lo[3], 0x0c070100u); q1[3] = __builtin_amdgcn_perm(no, lo[3], 0x0c070302u);
                  *(LAS u32x4*)(I8 + ch * 32u) = q0; *(LAS u32x4*)(I8 + ch * 32u + 16u) = q1;
                }
              }
            }
            if (nexc) {
#pragma unroll
              for (int i = 0; i < 3; ++i) {
                const u32 ch = lane + 64 * i;
                if (ch >= nch && ch < nch + nexc)
                  __hip_atomic_fetch_or((LAS u32*)(I8 + min(d[i] & 0x7FFu, 1535u) * 4u), d[i] & 0xFFF00000u, __ATOMIC_RELAXED, __HIP_MEMORY_SCOPE_WAVEFRONT);
              }
            }
            u32 base = 0;
#pragma unroll
            for (int g = 0; g < 8; ++g) {
              f32x4 y;
#pragma unroll
              for (int t = 0; t < 4; ++t) {
                const bool pos = (int)m < 0;
                m += m;
                const u64 bal = __builtin_amdgcn_ballot_w64(pos);
                const u32 ad = pos ? (mbcnt(bal) + base) * 4u : 6144u;
                base += (u32)__builtin_popcountll(bal);
                y[t] = (float)*(LAS int*)(I8 + ad) * scale;
              }
              st_out(a.out, on + (u32)((g & 3) * NN + (g >> 2) * 64) * 4u, y);
            }
          }
          if (dead) { ++n; break; }
        }
      }
      if (dead) {
        const float nan = __builtin_nanf("");
        st_out(a.out, ob + (u32)((((n & 15) >> 2) * 128 + (n & 3) * 16) * NN) * 4u, f32x4{nan, nan, nan, nan});
      }
      STAMP(2);
    }
  }
  VM_WAIT(0);
}

extern "C" void kernel_launch(int rank, int nranks, ncclComm_t comm,
                              hipStream_t stream,
                              void* const* d_in, const int* in_sizes, int n_in,
                              void* d_out, int out_size,
                              const int* in_dtypes, int out_dtype,
                              void* const* d_out_all,
                              void* const* d_ws_all, size_t ws_bytes) {
  (void)comm; (void)in_dtypes; (void)out_dtype; (void)d_out_all;
  if (nranks != NR || (unsigned)rank >= (unsigned)NR || n_in < 4 || ws_bytes < (size_t)WS_NEED ||
      in_sizes[0] != MM * KS || in_sizes[1] != KS * NN || out_size != (MM / NR) * NN) {
    fprintf(stderr, "kernel_launch: unexpected problem shape\n");
    return;
  }
  static bool attr_set[NR];
  if (!attr_set[rank]) {
    HIP_CHECK(hipFuncSetAttribute((const void*)gemm_rs_kernel, hipFuncAttributeMaxDynamicSharedMemorySize, (int)LDS_BYTES));
    attr_set[rank] = true;
  }
  Args a;
  a.x = (const char*)d_in[0]; a.w = (const char*)d_in[1];
  a.sx = (const float*)d_in[2]; a.sw = (const float*)d_in[3];
  a.out = (char*)d_out;
  for (int p = 0; p < NR; ++p) a.ws[p] = (char*)d_ws_all[p];
  a.rank = rank;
  hipLaunchKernelGGL(gemm_rs_kernel, dim3(256), dim3(512), LDS_BYTES, stream, a);
  HIP_CHECK(hipGetLastError());
}
```

```cpp
#include <hip/hip_runtime.h>
#include <rccl/rccl.h>
#include <cstdio>
#include <cstdint>
#include <cstddef>

#define HIP_CHECK(expr) do { hipError_t _e = (expr); if (_e != hipSuccess) { \
    fprintf(stderr, "HIP error %s at %s:%d\n", hipGetErrorString(_e), __FILE__, __LINE__); \
    return; } } while (0)

typedef unsigned u32;
typedef unsigned long long u64;
typedef unsigned u32x2 __attribute__((ext_vector_type(2)));
typedef unsigned u32x4 __attribute__((ext_vector_type(4)));
typedef int i32x4 __attribute__((ext_vector_type(4)));
typedef int i32x16 __attribute__((ext_vector_type(16)));
typedef float f32x4 __attribute__((ext_vector_type(4)));
#define GAS __attribute__((address_space(1)))
#define LAS __attribute__((address_space(3)))

#define PROF 0
#define DUMMY_SAME 0
#define ENC_DUMMY "v"(C + 7936u + lane * 4)
#define NPUSH 16
#define LNK_AT 195
#define HOLD 1
#define STG_T0 300
#define STG_DT 16
#define PULL_DLY 200
#define PULL_STG 0
#define LD_SLEEP 6
#define LD_PRIO 2
#define FREE_SLEEP 0
#define LD_CAP 0
#define JOIN 0
#define WARM 8
#define DPC 1
#define GATE7 0
#define WIN_0 16u
#define WIN_N 64u
#define LNK_SLEEP 8
#define LNK_RELAY 1
#define JOIN_N 0u
#define JOIN_B 112
#define FL_PRE 1
#define ORD 1
#define XCUT 72
#define GO_AT XCUT
#define PACE 5

constexpr int NR = 8;
constexpr int MM = 4096, KS = 512, NN = 16384, NSH = 2048;
constexpr u32 MSG = 16384;
constexpr u32 MiB = 1u << 20;
constexpr u32 XI_OFF = 0;
constexpr u32 WI_OFF = 16 * MiB;
constexpr u32 WSO_OFF = 24 * MiB;
constexpr u32 OSP_OFF = 32 * MiB;
constexpr u32 FL_OFF = 46 * MiB;
constexpr u32 FX_OFF = FL_OFF;
constexpr u32 FW_OFF = FL_OFF + 4096;
constexpr u32 LNK_OFF = FL_OFF + 8192;
constexpr u32 CTR_OFF = FL_OFF + 12288;
constexpr u32 OCT_OFF = FL_OFF + 14336;
constexpr u32 RDY_OFF = FL_OFF + 16384;
constexpr u32 OF_OFF = FL_OFF + 32768;
constexpr u32 OSR_OFF = 48 * MiB;
constexpr u32 PAR_BYTES = 76 * MiB;
constexpr u32 CNT_OFF = 2 * PAR_BYTES;
constexpr u32 TMO_OFF = CNT_OFF + 4096;
constexpr u32 PROF_OFF = CNT_OFF + 65536;
constexpr u32 PROF2_OFF = PROF_OFF + 256 * 8 * 16 * 8;
constexpr u32 PROF3_OFF = PROF2_OFF + 256 * 8 * 64 * 8;
constexpr u32 WS_NEED = PROF3_OFF + 7 * 192 * 8;
static_assert(OSP_OFF + 224u * 16u * 4096u <= FL_OFF, "");
static_assert(OF_OFF + 8u * 4u * 8u * 4u * 4u * 8u <= OSR_OFF, "");
static_assert(OSR_OFF + 224u * 16u * 8192u <= PAR_BYTES, "");
static_assert(((3u * 49152u) & 4096u) == 0u, "");

constexpr u32 SLOT = 49152;
constexpr u32 AUX_L = 3 * SLOT;
constexpr u32 SYNC_L = 163840 - 512;
constexpr u32 FULL_L = SYNC_L, FREE_L = SYNC_L + 16, ABORT_L = SYNC_L + 32, HOLD_L = SYNC_L + 48, DONE_L = SYNC_L + 52, STEP_L = SYNC_L + 56, OPEN_L = SYNC_L + 60;
constexpr u32 LDS_BYTES = 163840;
constexpr u32 PK_STRIDE = 12288;
constexpr u64 DEADLINE = 50000000ull;
constexpr u32 RAW_MODE = 1u << 20;

struct Args {
  const char* x;
  const char* w;
  const float* sx;
  const float* sw;
  char* out;
  char* ws[NR];
  int rank;
};

__device__ __forceinline__ u64 rtime() { return __builtin_amdgcn_s_memrealtime(); }
__device__ __forceinline__ u32 rfl(u32 v) { return __builtin_amdgcn_readfirstlane(v); }
__device__ __forceinline__ u32 rdl(u32 v, int l) { return __builtin_amdgcn_readlane(v, l); }
__device__ __forceinline__ u32 mbcnt(u64 m) {
  return __builtin_amdgcn_mbcnt_hi((u32)(m >> 32), __builtin_amdgcn_mbcnt_lo((u32)m, 0u));
}
#define VM_WAIT(n) asm volatile("s_waitcnt vmcnt(" #n ")" ::: "memory")
#define LGKM_WAIT0() do { asm volatile("" ::: "memory"); __builtin_amdgcn_s_waitcnt(0xC07F); asm volatile("" ::: "memory"); } while (0)

template <int IMM>
__device__ __forceinline__ void st128(const char* base, u32 off, u32x4 v) {
  asm volatile("global_store_dwordx4 %0, %1, %2 offset:%3 sc0 sc1\n\ts_nop 1" :: "v"(off), "v"(v), "s"(base), "n"(IMM) : "memory");
}
__device__ __forceinline__ void st64(const char* base, u32 off, u32x2 v) {
  asm volatile("global_store_dwordx2 %0, %1, %2 sc0 sc1\n\ts_nop 1" :: "v"(off), "v"(v), "s"(base) : "memory");
}
__device__ __forceinline__ void st32(const char* base, u32 off, u32 v) {
  asm volatile("global_store_dword %0, %1, %2 sc0 sc1\n\ts_nop 1" :: "v"(off), "v"(v), "s"(base) : "memory");
}
__device__ __forceinline__ void st32v(u64 addr, u32 v) {
  asm volatile("global_store_dword %0, %1, off sc0 sc1\n\ts_nop 1" :: "v"(addr), "v"(v) : "memory");
}
__device__ __forceinline__ u32 ld32w(const char* base, u32 off) {
  u32 v;
  asm volatile("global_load_dword %0, %1, %2 sc0 sc1\n\ts_waitcnt vmcnt(0)" : "=&v"(v) : "v"(off), "s"(base) : "memory");
  return v;
}
__device__ __forceinline__ u32x2 ld64w(const char* base, u32 off) {
  u32x2 v;
  asm volatile("global_load_dwordx2 %0, %1, %2 sc0 sc1\n\ts_waitcnt vmcnt(0)" : "=&v"(v) : "v"(off), "s"(base) : "memory");
  return v;
}
__device__ __forceinline__ void st_out(const char* base, u32 off, f32x4 v) {
  asm volatile("global_store_dwordx4 %0, %1, %2 sc0 sc1\n\ts_nop 1" :: "v"(off), "v"(v), "s"(base) : "memory");
}
__device__ __forceinline__ void glds4k(u32 lds, const char* base, u32 off) {
  asm volatile("s_mov_b32 m0, %0\n\ts_nop 0\n\t"
               "global_load_lds_dwordx4 %1, %2 sc0 sc1\n\t"
               "global_load_lds_dwordx4 %1, %2 offset:1024 sc0 sc1\n\t"
               "global_load_lds_dwordx4 %1, %2 offset:2048 sc0 sc1\n\t"
               "global_load_lds_dwordx4 %1, %2 offset:3072 sc0 sc1"
               :: "s"(lds), "v"(off), "s"(base) : "memory");
}
template <int IMM>
__device__ __forceinline__ void glds1k(u32 lds, const char* base, u32 off) {
  asm volatile("s_mov_b32 m0, %0\n\ts_nop 0\n\tglobal_load_lds_dwordx4 %1, %2 offset:%3 sc0 sc1"
               :: "s"(lds), "v"(off), "s"(base), "n"(IMM) : "memory");
}
__device__ __forceinline__ void ld8kw(u32x4 (&R)[8], const char* base, u32 off) {
  asm volatile("global_load_dwordx4 %0, %8, %10 sc0 sc1\n\tglobal_load_dwordx4 %1, %8, %10 offset:1024 sc0 sc1\n\t"
               "global_load_dwordx4 %2, %8, %10 offset:2048 sc0 sc1\n\tglobal_load_dwordx4 %3, %8, %10 offset:3072 sc0 sc1\n\t"
               "global_load_dwordx4 %4, %9, %10 sc0 sc1\n\tglobal_load_dwordx4 %5, %9, %10 offset:1024 sc0 sc1\n\t"
               "global_load_dwordx4 %6, %9, %10 offset:2048 sc0 sc1\n\tglobal_load_dwordx4 %7, %9, %10 offset:3072 sc0 sc1\n\t"
               "s_waitcnt vmcnt(0)"
               : "=&v"(R[0]), "=&v"(R[1]), "=&v"(R[2]), "=&v"(R[3]), "=&v"(R[4]), "=&v"(R[5]), "=&v"(R[6]), "=&v"(R[7])
               : "v"(off), "v"(off + 4096u), "s"(base) : "memory");
}
#define PL(r, o) "global_load_dwordx4 %[" #r "], %[vs], %[pp] offset:" #o " sc0 sc1\n\t"
#define PS(r, o) "global_store_dwordx4 %[vd], %[" #r "], %[wp] offset:" #o " sc0 sc1\n\t"
#define PLA PL(a0, 0) PL(a1, 1024) PL(a2, 2048) PL(a3, 3072) "v_add_u32 %[vs], 0x1000, %[vs]\n\t"
#define PLB PL(b0, 0) PL(b1, 1024) PL(b2, 2048) PL(b3, 3072) "v_add_u32 %[vs], 0x1000, %[vs]\n\t"
#define PSA PS(a0, 0) PS(a1, 1024) PS(a2, 2048) PS(a3, 3072) "v_add_u32 %[vd], 0x1000, %[vd]\n\t"
#define PSB PS(b0, 0) PS(b1, 1024) PS(b2, 2048) PS(b3, 3072) "v_add_u32 %[vd], 0x1000, %[vd]\n\t"
#define PTS(reg, o) ""
#define PEA ""
#define PDESC "s_lshl4_add_u32 %[t], %[idx], %[tab]\n\tv_mov_b32 %[d2], %[t]\n\t" \
              "ds_read_b32 %[d0], %[d2]\n\tds_read_b32 %[d1], %[d2] offset:4\n\tds_read_b32 %[d2], %[d2] offset:8\n\t" \
              "s_waitcnt lgkmcnt(0)\n\tv_add_u32 %[vs], %[d0], %[vo]\n\t" PEA
#define PLC PL(c0, 0) PL(c1, 1024) PL(c2, 2048) PL(c3, 3072) "v_add_u32 %[vs], 0x1000, %[vs]\n\t"
#define PLD PL(e0, 0) PL(e1, 1024) PL(e2, 2048) PL(e3, 3072) "v_add_u32 %[vs], 0x1000, %[vs]\n\t"
#define PSC PS(c0, 0) PS(c1, 1024) PS(c2, 2048) PS(c3, 3072) "v_add_u32 %[vd], 0x1000, %[vd]\n\t"
#define PSD PS(e0, 0) PS(e1, 1024) PS(e2, 2048) PS(e3, 3072) "v_add_u32 %[vd], 0x1000, %[vd]\n\t"
#define PATOM "s_mov_b64 exec, 1\n\tglobal_atomic_add_x2 v[64:65], %[ctr], %[one], %[wp] sc0 sc1\n\ts_mov_b64 exec, -1\n\t" \
              "global_load_dword %[r0], %[rf0], %[wp] sc0 sc1\n\tglobal_load_dword %[r1], %[rf1], %[wp] sc0 sc1\n\t" \
              "global_load_dword %[r2], %[rf2], %[wp] sc0 sc1\n\t"
#define PFLAG(r, e) "s_mov_b64 exec, 1\n\tglobal_store_dword %[" #r "], %[call], %[wp] sc0 sc1\n\t" PTS(e, 0) "s_mov_b64 exec, -1\n\t"
#define PNEXT \
      "v_readfirstlane_b32 %[t], v64\n\t" \
      "v_cmp_eq_u32 %[m0], %[r0], %[call]\n\t" \
      "v_cmp_eq_u32 %[m1], %[r1], %[call]\n\t" \
      "v_cmp_eq_u32 %[m2], %[r2], %[call]\n\t" \
      "s_sub_u32 %[idx], %[t], %[base]\n\t" \
      "s_cmp_lt_u32 %[idx], 64\n\t" \
      "s_cselect_b64 %[m0], %[m0], %[m1]\n\t" \
      "s_cmp_lt_u32 %[idx], 0x80\n\t" \
      "s_cselect_b64 %[m0], %[m0], %[m2]\n\t" \
      "s_cmp_lt_u32 %[idx], %[nm]\n\t" \
      "s_cselect_b64 %[m0], %[m0], 0\n\t" \
      "s_cmp_lg_u32 %[idx], %[gate]\n\t" \
      "s_cbranch_scc1 4f\n\t" \
      "s_mov_b64 exec, 1\n\t" \
      "global_store_dword %[lnk], %[call], %[wp] sc0 sc1\n\t" \
      "s_mov_b64 exec, -1\n" \
      "4:\n\t"
__device__ __forceinline__ u32 pull_stream(u32 idx, const char* pp, const char* wp, u32 tab, u32 ctr, u32 lnk, u32 rf0, u32 rf1, u32 rf2,
                                           u32 vo, u32 call, u32 nm, u32 gate) {
  u32x4 a0, a1, a2, a3, b0, b1, b2, b3, c0, c1, c2, c3, e0, e1, e2, e3;
  u32 vs, vd, t, d0, d1, d2, lfc, lfp, r0, r1, r2;
  u64 m0, m1, m2, got;
  asm volatile(
      "v_add_u32 %[lfp], 64, %[ctr]\n\t"
      PATOM
      PDESC
      PLA PLB PLC PLD
      "v_add_u32 %[vd], %[d1], %[vo]\n\t"
      "v_mov_b32 %[lfc], %[d2]\n\t"
      "s_waitcnt vmcnt(8)\n\t"
      "s_branch 2f\n"
      "1:\n\t"
      "s_waitcnt vmcnt(25)\n"
      "2:\n\t"
      PNEXT
      PSA
      "s_bitcmp1_b64 %[m0], %[idx]\n\t"
      "s_cbranch_scc0 3f\n\t"
      PDESC
      PATOM PLA
      "s_waitcnt vmcnt(29)\n\t"
      PSB PLB
      "s_waitcnt vmcnt(24)\n\t"
      PSC PFLAG(lfp, eq) PLC
      "s_waitcnt vmcnt(29)\n\t"
      PSD PLD
      "v_add_u32 %[vd], %[d1], %[vo]\n\t"
      "v_mov_b32 %[lfp], %[lfc]\n\t"
      "v_mov_b32 %[lfc], %[d2]\n\t"
      "s_branch 1b\n"
      "3:\n\t"
      "s_waitcnt vmcnt(21)\n\t"
      PSB
      "s_waitcnt vmcnt(12)\n\t"
      PSC PFLAG(lfp, ep)
      "s_waitcnt vmcnt(13)\n\t"
      PSD
      "s_waitcnt vmcnt(0)\n\t"
      PFLAG(lfc, ea)
      "s_nop 1"
      : [a0] "=&v"(a0), [a1] "=&v"(a1), [a2] "=&v"(a2), [a3] "=&v"(a3), [b0] "=&v"(b0), [b1] "=&v"(b1), [b2] "=&v"(b2), [b3] "=&v"(b3),
        [c0] "=&v"(c0), [c1] "=&v"(c1), [c2] "=&v"(c2), [c3] "=&v"(c3), [e0] "=&v"(e0), [e1] "=&v"(e1), [e2] "=&v"(e2), [e3] "=&v"(e3),
        [vs] "=&v"(vs), [vd] "=&v"(vd), [d0] "=&v"(d0), [d1] "=&v"(d1), [d2] "=&v"(d2), [lfc] "=&v"(lfc), [lfp] "=&v"(lfp),
        [r0] "=&v"(r0), [r1] "=&v"(r1), [r2] "=&v"(r2), "=&{v[64:65]}"(got),
        [t] "=&s"(t), [m0] "=&s"(m0), [m1] "=&s"(m1), [m2] "=&s"(m2), [idx] "+s"(idx)
      : [pp] "s"(pp), [wp] "s"(wp), [tab] "s"(tab), [base] "s"(call << 8), [ctr] "v"(ctr), [lnk] "v"(lnk), [one] "v"(1ull),
        [rf0] "v"(rf0), [rf1] "v"(rf1), [rf2] "v"(rf2), [vo] "v"(vo), [call] "v"(call), [nm] "s"(nm), [gate] "s"(gate)
      : "memory", "scc");
  return idx;
}

#define ASM_GEMM \
  ".set gi, 0\n\t" \
  ".rept 64\n\t" \
  "v_mov_b64 v[gi:gi+1], %[init]\n\t" \
  ".set gi, gi+2\n\t" \
  ".endr\n\t" \
  "s_mov_b32 %[dead], 0\n\t" \
  "s_mov_b32 %[st], 1\n\t" \
  "s_mov_b32 %[need], 1\n\t" \
  "s_mov_b32 %[sf], %[ful]\n\t" \
  "v_mov_b32 %[vf], %[ful]\n\t" \
  "5:\n" \
  "v_mov_b32 %[vi], %[abl]\n\t" \
  "ds_read_b32 %[vt], %[vf]\n\t" \
  "ds_read_b32 %[vi], %[vi]\n\t" \
  "s_memrealtime vcc\n\t" \
  "s_add_u32 %[pc], %[pc], 1\n\t" \
  "s_waitcnt lgkmcnt(0)\n\t" \
  "v_readfirstlane_b32 %[t], %[vt]\n\t" \
  "s_cmp_ge_u32 %[t], %[need]\n\t" \
  "s_cbranch_scc1 6f\n\t" \
  "v_readfirstlane_b32 %[t], %[vi]\n\t" \
  "s_cmp_lg_u32 %[t], 0\n\t" \
  "s_cbranch_scc1 9f\n\t" \
  "s_sub_u32 %[t], vcc_lo, %[t0]\n\t" \
  "s_cmp_lt_u32 %[t], %[dl]\n\t" \
  "s_sleep 1\n\t" \
  "s_cbranch_scc1 5b\n\t" \
  "s_branch 9f\n\t" \
  "6:\n" \
  "s_mov_b32 %[pc], 0\n\t" \
  "ds_read_b128 v[128:131], %[a0]\n\t" \
  "ds_read_b128 v[144:147], %[b0] offset:0\n\t" \
  "ds_read_b128 v[148:151], %[b0] offset:1024\n\t" \
  "ds_read_b128 v[152:155], %[b0] offset:2048\n\t" \
  "ds_read_b128 v[156:159], %[b0] offset:3072\n\t" \
  "ds_read_b128 v[160:163], %[b1] offset:0\n\t" \
  "ds_read_b128 v[164:167], %[b1] offset:1024\n\t" \
  "ds_read_b128 v[168:171], %[b1] offset:2048\n\t" \
  "ds_read_b128 v[172:175], %[b1] offset:3072\n\t" \
  "ds_read_b128 v[132:135], %[a0] offset:2048\n\t" \
  "ds_read_b128 v[136:139], %[a0] offset:4096\n\t" \
  "ds_read_b128 v[140:143], %[a0] offset:6144\n\t" \
  "s_waitcnt lgkmcnt(0)\n\t" \
  "1:\n" \
  "v_mfma_i32_16x16x64_i8 v[0:3], v[128:131], v[144:147], v[0:3]\n\t" \
  "ds_read_b128 v[176:179], %[a1]\n\t" \
  "v_mfma_i32_16x16x64_i8 v[4:7], v[128:131], v[148:151], v[4:7]\n\t" \
  "ds_read_b128 v[192:195], %[b0] offset:512\n\t" \
  "v_mfma_i32_16x16x64_i8 v[8:11], v[128:131], v[152:155], v[8:11]\n\t" \
  "s_add_u32 %[sf], %[sf], 4\n\t" \
  "s_cmp_eq_u32 %[sf], %[ful]+12\n\t" \
  "v_mfma_i32_16x16x64_i8 v[12:15], v[128:131], v[156:159], v[12:15]\n\t" \
  "ds_read_b128 v[196:199], %[b0] offset:1536\n\t" \
  "v_mfma_i32_16x16x64_i8 v[16:19], v[128:131], v[160:163], v[16:19]\n\t" \
  "ds_read_b128 v[200:203], %[b0] offset:2560\n\t" \
  "v_mfma_i32_16x16x64_i8 v[20:23], v[128:131], v[164:167], v[20:23]\n\t" \
  "s_cselect_b32 %[sf], %[ful], %[sf]\n\t" \
  "s_cselect_b32 %[d], %[wrap], %[slot]\n\t" \
  "v_mfma_i32_16x16x64_i8 v[24:27], v[128:131], v[168:171], v[24:27]\n\t" \
  "ds_read_b128 v[204:207], %[b0] offset:3584\n\t" \
  "v_mfma_i32_16x16x64_i8 v[28:31], v[128:131], v[172:175], v[28:31]\n\t" \
  "ds_read_b128 v[208:211], %[b1] offset:512\n\t" \
  "v_mfma_i32_16x16x64_i8 v[32:35], v[132:135], v[144:147], v[32:35]\n\t" \
  "v_mov_b32 %[vf], %[sf]\n\t" \
  "v_mov_b32 %[vi], %[st]\n\t" \
  "v_mfma_i32_16x16x64_i8 v[36:39], v[132:135], v[148:151], v[36:39]\n\t" \
  "ds_read_b128 v[212:215], %[b1] offset:1536\n\t" \
  "v_mfma_i32_16x16x64_i8 v[40:43], v[132:135], v[152:155], v[40:43]\n\t" \
  "ds_read_b128 v[216:219], %[b1] offset:2560\n\t" \
  "v_mfma_i32_16x16x64_i8 v[44:47], v[132:135], v[156:159], v[44:47]\n\t" \
  "s_add_u32 %[need], %[st], 1\n\t" \
  "s_cmp_eq_u32 %[st], 32\n\t" \
  "v_mfma_i32_16x16x64_i8 v[48:51], v[132:135], v[160:163], v[48:51]\n\t" \
  "ds_read_b128 v[220:223], %[b1] offset:3584\n\t" \
  "v_mfma_i32_16x16x64_i8 v[52:55], v[132:135], v[164:167], v[52:55]\n\t" \
  "ds_read_b128 v[180:183], %[a1] offset:2048\n\t" \
  "v_mfma_i32_16x16x64_i8 v[56:59], v[132:135], v[168:171], v[56:59]\n\t" \
  "s_cselect_b32 %[need], 0, %[need]\n\t" \
  "v_add_u32 %[a0], %[d], %[a0]\n\t" \
  "v_mfma_i32_16x16x64_i8 v[60:63], v[132:135], v[172:175], v[60:63]\n\t" \
  "ds_read_b128 v[184:187], %[a1] offset:4096\n\t" \
  "v_mfma_i32_16x16x64_i8 v[64:67], v[136:139], v[144:147], v[64:67]\n\t" \
  "ds_read_b128 v[188:191], %[a1] offset:6144\n\t" \
  "v_mfma_i32_16x16x64_i8 v[68:71], v[136:139], v[148:151], v[68:71]\n\t" \
  "v_add_u32 %[a1], %[d], %[a1]\n\t" \
  "v_add_u32 %[b0], %[d], %[b0]\n\t" \
  "v_mfma_i32_16x16x64_i8 v[72:75], v[136:139], v[152:155], v[72:75]\n\t" \
  "v_add_u32 %[b1], %[d], %[b1]\n\t" \
  "v_mfma_i32_16x16x64_i8 v[76:79], v[136:139], v[156:159], v[76:79]\n\t" \
  "v_mfma_i32_16x16x64_i8 v[80:83], v[136:139], v[160:163], v[80:83]\n\t" \
  "v_mfma_i32_16x16x64_i8 v[84:87], v[136:139], v[164:167], v[84:87]\n\t" \
  "s_waitcnt lgkmcnt(0)\n\t" \
  "s_mov_b64 exec, 1\n\t" \
  "ds_write_b32 %[vfree], %[vi]\n\t" \
  "s_mov_b64 exec, -1\n\t" \
  "v_mfma_i32_16x16x64_i8 v[88:91], v[136:139], v[168:171], v[88:91]\n\t" \
  "v_mfma_i32_16x16x64_i8 v[92:95], v[136:139], v[172:175], v[92:95]\n\t" \
  "ds_read_b32 %[vt], %[vf]\n\t" \
  "v_mfma_i32_16x16x64_i8 v[96:99], v[140:143], v[144:147], v[96:99]\n\t" \
  "v_mfma_i32_16x16x64_i8 v[100:103], v[140:143], v[148:151], v[100:103]\n\t" \
  "v_mfma_i32_16x16x64_i8 v[104:107], v[140:143], v[152:155], v[104:107]\n\t" \
  "v_mfma_i32_16x16x64_i8 v[108:111], v[140:143], v[156:159], v[108:111]\n\t" \
  "v_mfma_i32_16x16x64_i8 v[112:115], v[140:143], v[160:163], v[112:115]\n\t" \
  "v_mfma_i32_16x16x64_i8 v[116:119], v[140:143], v[164:167], v[116:119]\n\t" \
  "s_waitcnt lgkmcnt(0)\n\t" \
  "v_readfirstlane_b32 %[t], %[vt]\n\t" \
  "v_mfma_i32_16x16x64_i8 v[120:123], v[140:143], v[168:171], v[120:123]\n\t" \
  "s_cmp_ge_u32 %[t], %[need]\n\t" \
  "v_mfma_i32_16x16x64_i8 v[124:127], v[140:143], v[172:175], v[124:127]\n\t" \
  "s_cbranch_scc0 3f\n\t" \
  "2:\n" \
  "v_mfma_i32_16x16x64_i8 v[0:3], v[176:179], v[192:195], v[0:3]\n\t" \
  "ds_read_b128 v[128:131], %[a0]\n\t" \
  "v_mfma_i32_16x16x64_i8 v[4:7], v[176:179], v[196:199], v[4:7]\n\t" \
  "ds_read_b128 v[144:147], %[b0] offset:0\n\t" \
  "v_mfma_i32_16x16x64_i8 v[8:11], v[176:179], v[200:203], v[8:11]\n\t" \
  "v_mfma_i32_16x16x64_i8 v[12:15], v[176:179], v[204:207], v[12:15]\n\t" \
  "ds_read_b128 v[148:151], %[b0] offset:1024\n\t" \
  "v_mfma_i32_16x16x64_i8 v[16:19], v[176:179], v[208:211], v[16:19]\n\t" \
  "ds_read_b128 v[152:155], %[b0] offset:2048\n\t" \
  "v_mfma_i32_16x16x64_i8 v[20:23], v[176:179], v[212:215], v[20:23]\n\t" \
  "v_mfma_i32_16x16x64_i8 v[24:27], v[176:179], v[216:219], v[24:27]\n\t" \
  "ds_read_b128 v[156:159], %[b0] offset:3072\n\t" \
  "v_mfma_i32_16x16x64_i8 v[28:31], v[176:179], v[220:223], v[28:31]\n\t" \
  "ds_read_b128 v[160:163], %[b1] offset:0\n\t" \
  "v_mfma_i32_16x16x64_i8 v[32:35], v[180:183], v[192:195], v[32:35]\n\t" \
  "v_mfma_i32_16x16x64_i8 v[36:39], v[180:183], v[196:199], v[36:39]\n\t" \
  "ds_read_b128 v[164:167], %[b1] offset:1024\n\t" \
  "v_mfma_i32_16x16x64_i8 v[40:43], v[180:183], v[200:203], v[40:43]\n\t" \
  "ds_read_b128 v[168:171], %[b1] offset:2048\n\t" \
  "v_mfma_i32_16x16x64_i8 v[44:47], v[180:183], v[204:207], v[44:47]\n\t" \
  "v_mfma_i32_16x16x64_i8 v[48:51], v[180:183], v[208:211], v[48:51]\n\t" \
  "ds_read_b128 v[172:175], %[b1] offset:3072\n\t" \
  "v_mfma_i32_16x16x64_i8 v[52:55], v[180:183], v[212:215], v[52:55]\n\t" \
  "ds_read_b128 v[132:135], %[a0] offset:2048\n\t" \
  "v_mfma_i32_16x16x64_i8 v[56:59], v[180:183], v[216:219], v[56:59]\n\t" \
  "v_mfma_i32_16x16x64_i8 v[60:63], v[180:183], v[220:223], v[60:63]\n\t" \
  "ds_read_b128 v[136:139], %[a0] offset:4096\n\t" \
  "v_mfma_i32_16x16x64_i8 v[64:67], v[184:187], v[192:195], v[64:67]\n\t" \
  "ds_read_b128 v[140:143], %[a0] offset:6144\n\t" \
  "v_mfma_i32_16x16x64_i8 v[68:71], v[184:187], v[196:199], v[68:71]\n\t" \
  "v_mfma_i32_16x16x64_i8 v[72:75], v[184:187], v[200:203], v[72:75]\n\t" \
  "v_mfma_i32_16x16x64_i8 v[76:79], v[184:187], v[204:207], v[76:79]\n\t" \
  "v_mfma_i32_16x16x64_i8 v[80:83], v[184:187], v[208:211], v[80:83]\n\t" \
  "v_mfma_i32_16x16x64_i8 v[84:87], v[184:187], v[212:215], v[84:87]\n\t" \
  "v_mfma_i32_16x16x64_i8 v[88:91], v[184:187], v[216:219], v[88:91]\n\t" \
  "v_mfma_i32_16x16x64_i8 v[92:95], v[184:187], v[220:223], v[92:95]\n\t" \
  "v_mfma_i32_16x16x64_i8 v[96:99], v[188:191], v[192:195], v[96:99]\n\t" \
  "v_mfma_i32_16x16x64_i8 v[100:103], v[188:191], v[196:199], v[100:103]\n\t" \
  "v_mfma_i32_16x16x64_i8 v[104:107], v[188:191], v[200:203], v[104:107]\n\t" \
  "v_mfma_i32_16x16x64_i8 v[108:111], v[188:191], v[204:207], v[108:111]\n\t" \
  "v_mfma_i32_16x16x64_i8 v[112:115], v[188:191], v[208:211], v[112:115]\n\t" \
  "v_mfma_i32_16x16x64_i8 v[116:119], v[188:191], v[212:215], v[116:119]\n\t" \
  "s_add_u32 %[st], %[st], 1\n\t" \
  "s_cmp_le_u32 %[st], 32\n\t" \
  "v_mfma_i32_16x16x64_i8 v[120:123], v[188:191], v[216:219], v[120:123]\n\t" \
  "s_waitcnt lgkmcnt(0)\n\t" \
  "v_mfma_i32_16x16x64_i8 v[124:127], v[188:191], v[220:223], v[124:127]\n\t" \
  "s_cbranch_scc1 1b\n\t" \
  "s_cmp_eq_u32 %[neg], 0\n\t" \
  "s_nop 7\n\t" \
  "s_cbranch_scc1 8f\n\t" \
  ".set gi, 0\n\t" \
  ".rept 128\n\t" \
  "v_not_b32 v[gi], v[gi]\n\t" \
  ".set gi, gi+1\n\t" \
  ".endr\n\t" \
  "s_branch 8f\n\t" \
  "3:\n" \
  "v_mov_b32 %[vi], %[abl]\n\t" \
  "ds_read_b32 %[vt], %[vf]\n\t" \
  "ds_read_b32 %[vi], %[vi]\n\t" \
  "s_memrealtime vcc\n\t" \
  "s_add_u32 %[pc], %[pc], 1\n\t" \
  "s_waitcnt lgkmcnt(0)\n\t" \
  "v_readfirstlane_b32 %[t], %[vt]\n\t" \
  "s_cmp_ge_u32 %[t], %[need]\n\t" \
  "s_cbranch_scc1 2b\n\t" \
  "v_readfirstlane_b32 %[t], %[vi]\n\t" \
  "s_cmp_lg_u32 %[t], 0\n\t" \
  "s_cbranch_scc1 9f\n\t" \
  "s_sub_u32 %[t], vcc_lo, %[t0]\n\t" \
  "s_cmp_lt_u32 %[t], %[dl]\n\t" \
  "s_sleep 1\n\t" \
  "s_cbranch_scc1 3b\n\t" \
  "s_branch 9f\n\t" \
  "9:\n" \
  "v_mov_b32 %[vt], 1\n\t" \
  "v_mov_b32 %[vi], %[abl]\n\t" \
  "ds_write_b32 %[vi], %[vt]\n\t" \
  "s_mov_b32 %[dead], 1\n\t" \
  "s_waitcnt lgkmcnt(0)\n\t" \
  "s_nop 7\n\t" \
  "8:\n" \
  ""
#define ASM_ENC1 \
  "v_alignbit_b32 %[m0], %[m0], v0, 31\n\t" \
  "v_alignbit_b32 %[m1], %[m1], v32, 31\n\t" \
  "v_alignbit_b32 %[m2], %[m2], v64, 31\n\t" \
  "v_alignbit_b32 %[m3], %[m3], v96, 31\n\t" \
  "v_alignbit_b32 %[m0], %[m0], v4, 31\n\t" \
  "v_alignbit_b32 %[m1], %[m1], v36, 31\n\t" \
  "v_alignbit_b32 %[m2], %[m2], v68, 31\n\t" \
  "v_alignbit_b32 %[m3], %[m3], v100, 31\n\t" \
  "v_alignbit_b32 %[m0], %[m0], v8, 31\n\t" \
  "v_alignbit_b32 %[m1], %[m1], v40, 31\n\t" \
  "v_alignbit_b32 %[m2], %[m2], v72, 31\n\t" \
  "v_alignbit_b32 %[m3], %[m3], v104, 31\n\t" \
  "v_alignbit_b32 %[m0], %[m0], v12, 31\n\t" \
  "v_alignbit_b32 %[m1], %[m1], v44, 31\n\t" \
  "v_alignbit_b32 %[m2], %[m2], v76, 31\n\t" \
  "v_alignbit_b32 %[m3], %[m3], v108, 31\n\t" \
  "v_alignbit_b32 %[m0], %[m0], v1, 31\n\t" \
  "v_alignbit_b32 %[m1], %[m1], v33, 31\n\t" \
  "v_alignbit_b32 %[m2], %[m2], v65, 31\n\t" \
  "v_alignbit_b32 %[m3], %[m3], v97, 31\n\t" \
  "v_alignbit_b32 %[m0], %[m0], v5, 31\n\t" \
  "v_alignbit_b32 %[m1], %[m1], v37, 31\n\t" \
  "v_alignbit_b32 %[m2], %[m2], v69, 31\n\t" \
  "v_alignbit_b32 %[m3], %[m3], v101, 31\n\t" \
  "v_alignbit_b32 %[m0], %[m0], v9, 31\n\t" \
  "v_alignbit_b32 %[m1], %[m1], v41, 31\n\t" \
  "v_alignbit_b32 %[m2], %[m2], v73, 31\n\t" \
  "v_alignbit_b32 %[m3], %[m3], v105, 31\n\t" \
  "v_alignbit_b32 %[m0], %[m0], v13, 31\n\t" \
  "v_alignbit_b32 %[m1], %[m1], v45, 31\n\t" \
  "v_alignbit_b32 %[m2], %[m2], v77, 31\n\t" \
  "v_alignbit_b32 %[m3], %[m3], v109, 31\n\t" \
  "v_alignbit_b32 %[m0], %[m0], v2, 31\n\t" \
  "v_alignbit_b32 %[m1], %[m1], v34, 31\n\t" \
  "v_alignbit_b32 %[m2], %[m2], v66, 31\n\t" \
  "v_alignbit_b32 %[m3], %[m3], v98, 31\n\t" \
  "v_alignbit_b32 %[m0], %[m0], v6, 31\n\t" \
  "v_alignbit_b32 %[m1], %[m1], v38, 31\n\t" \
  "v_alignbit_b32 %[m2], %[m2], v70, 31\n\t" \
  "v_alignbit_b32 %[m3], %[m3], v102, 31\n\t" \
  "v_alignbit_b32 %[m0], %[m0], v10, 31\n\t" \
  "v_alignbit_b32 %[m1], %[m1], v42, 31\n\t" \
  "v_alignbit_b32 %[m2], %[m2], v74, 31\n\t" \
  "v_alignbit_b32 %[m3], %[m3], v106, 31\n\t" \
  "v_alignbit_b32 %[m0], %[m0], v14, 31\n\t" \
  "v_alignbit_b32 %[m1], %[m1], v46, 31\n\t" \
  "v_alignbit_b32 %[m2], %[m2], v78, 31\n\t" \
  "v_alignbit_b32 %[m3], %[m3], v110, 31\n\t" \
  "v_alignbit_b32 %[m0], %[m0], v3, 31\n\t" \
  "v_alignbit_b32 %[m1], %[m1], v35, 31\n\t" \
  "v_alignbit_b32 %[m2], %[m2], v67, 31\n\t" \
  "v_alignbit_b32 %[m3], %[m3], v99, 31\n\t" \
  "v_alignbit_b32 %[m0], %[m0], v7, 31\n\t" \
  "v_alignbit_b32 %[m1], %[m1], v39, 31\n\t" \
  "v_alignbit_b32 %[m2], %[m2], v71, 31\n\t" \
  "v_alignbit_b32 %[m3], %[m3], v103, 31\n\t" \
  "v_alignbit_b32 %[m0], %[m0], v11, 31\n\t" \
  "v_alignbit_b32 %[m1], %[m1], v43, 31\n\t" \
  "v_alignbit_b32 %[m2], %[m2], v75, 31\n\t" \
  "v_alignbit_b32 %[m3], %[m3], v107, 31\n\t" \
  "v_alignbit_b32 %[m0], %[m0], v15, 31\n\t" \
  "v_alignbit_b32 %[m1], %[m1], v47, 31\n\t" \
  "v_alignbit_b32 %[m2], %[m2], v79, 31\n\t" \
  "v_alignbit_b32 %[m3], %[m3], v111, 31\n\t" \
  "v_alignbit_b32 %[m0], %[m0], v16, 31\n\t" \
  "v_alignbit_b32 %[m1], %[m1], v48, 31\n\t" \
  "v_alignbit_b32 %[m2], %[m2], v80, 31\n\t" \
  "v_alignbit_b32 %[m3], %[m3], v112, 31\n\t" \
  "v_alignbit_b32 %[m0], %[m0], v20, 31\n\t" \
  "v_alignbit_b32 %[m1], %[m1], v52, 31\n\t" \
  "v_alignbit_b32 %[m2], %[m2], v84, 31\n\t" \
  "v_alignbit_b32 %[m3], %[m3], v116, 31\n\t" \
  "v_alignbit_b32 %[m0], %[m0], v24, 31\n\t" \
  "v_alignbit_b32 %[m1], %[m1], v56, 31\n\t" \
  "v_alignbit_b32 %[m2], %[m2], v88, 31\n\t" \
  "v_alignbit_b32 %[m3], %[m3], v120, 31\n\t" \
  "v_alignbit_b32 %[m0], %[m0], v28, 31\n\t" \
  "v_alignbit_b32 %[m1], %[m1], v60, 31\n\t" \
  "v_alignbit_b32 %[m2], %[m2], v92, 31\n\t" \
  "v_alignbit_b32 %[m3], %[m3], v124, 31\n\t" \
  "v_alignbit_b32 %[m0], %[m0], v17, 31\n\t" \
  "v_alignbit_b32 %[m1], %[m1], v49, 31\n\t" \
  "v_alignbit_b32 %[m2], %[m2], v81, 31\n\t" \
  "v_alignbit_b32 %[m3], %[m3], v113, 31\n\t" \
  "v_alignbit_b32 %[m0], %[m0], v21, 31\n\t" \
  "v_alignbit_b32 %[m1], %[m1], v53, 31\n\t" \
  "v_alignbit_b32 %[m2], %[m2], v85, 31\n\t" \
  "v_alignbit_b32 %[m3], %[m3], v117, 31\n\t" \
  "v_alignbit_b32 %[m0], %[m0], v25, 31\n\t" \
  "v_alignbit_b32 %[m1], %[m1], v57, 31\n\t" \
  "v_alignbit_b32 %[m2], %[m2], v89, 31\n\t" \
  "v_alignbit_b32 %[m3], %[m3], v121, 31\n\t" \
  "v_alignbit_b32 %[m0], %[m0], v29, 31\n\t" \
  "v_alignbit_b32 %[m1], %[m1], v61, 31\n\t" \
  "v_alignbit_b32 %[m2], %[m2], v93, 31\n\t" \
  "v_alignbit_b32 %[m3], %[m3], v125, 31\n\t" \
  "v_alignbit_b32 %[m0], %[m0], v18, 31\n\t" \
  "v_alignbit_b32 %[m1], %[m1], v50, 31\n\t" \
  "v_alignbit_b32 %[m2], %[m2], v82, 31\n\t" \
  "v_alignbit_b32 %[m3], %[m3], v114, 31\n\t" \
  "v_alignbit_b32 %[m0], %[m0], v22, 31\n\t" \
  "v_alignbit_b32 %[m1], %[m1], v54, 31\n\t" \
  "v_alignbit_b32 %[m2], %[m2], v86, 31\n\t" \
  "v_alignbit_b32 %[m3], %[m3], v118, 31\n\t" \
  "v_alignbit_b32 %[m0], %[m0], v26, 31\n\t" \
  "v_alignbit_b32 %[m1], %[m1], v58, 31\n\t" \
  "v_alignbit_b32 %[m2], %[m2], v90, 31\n\t" \
  "v_alignbit_b32 %[m3], %[m3], v122, 31\n\t" \
  "v_alignbit_b32 %[m0], %[m0], v30, 31\n\t" \
  "v_alignbit_b32 %[m1], %[m1], v62, 31\n\t" \
  "v_alignbit_b32 %[m2], %[m2], v94, 31\n\t" \
  "v_alignbit_b32 %[m3], %[m3], v126, 31\n\t" \
  "v_alignbit_b32 %[m0], %[m0], v19, 31\n\t" \
  "v_alignbit_b32 %[m1], %[m1], v51, 31\n\t" \
  "v_alignbit_b32 %[m2], %[m2], v83, 31\n\t" \
  "v_alignbit_b32 %[m3], %[m3], v115, 31\n\t" \
  "v_alignbit_b32 %[m0], %[m0], v23, 31\n\t" \
  "v_alignbit_b32 %[m1], %[m1], v55, 31\n\t" \
  "v_alignbit_b32 %[m2], %[m2], v87, 31\n\t" \
  "v_alignbit_b32 %[m3], %[m3], v119, 31\n\t" \
  "v_alignbit_b32 %[m0], %[m0], v27, 31\n\t" \
  "v_alignbit_b32 %[m1], %[m1], v59, 31\n\t" \
  "v_alignbit_b32 %[m2], %[m2], v91, 31\n\t" \
  "v_alignbit_b32 %[m3], %[m3], v123, 31\n\t" \
  "v_alignbit_b32 %[m0], %[m0], v31, 31\n\t" \
  "v_alignbit_b32 %[m1], %[m1], v63, 31\n\t" \
  "v_alignbit_b32 %[m2], %[m2], v95, 31\n\t" \
  "v_alignbit_b32 %[m3], %[m3], v127, 31\n\t" \
  "v_bcnt_u32_b32 v128, %[m0], 0\n\t" \
  "v_bcnt_u32_b32 v129, %[m1], 0\n\t" \
  "v_bcnt_u32_b32 v130, %[m2], 0\n\t" \
  "v_bcnt_u32_b32 v131, %[m3], 0\n\t" \
  "v_add_u32_dpp v132, v128, v128 row_shr:1 row_mask:0xf bank_mask:0xf bound_ctrl:0\n\t" \
  "v_add_u32_dpp v133, v129, v129 row_shr:1 row_mask:0xf bank_mask:0xf bound_ctrl:0\n\t" \
  "v_add_u32_dpp v134, v130, v130 row_shr:1 row_mask:0xf bank_mask:0xf bound_ctrl:0\n\t" \
  "v_add_u32_dpp v135, v131, v131 row_shr:1 row_mask:0xf bank_mask:0xf bound_ctrl:0\n\t" \
  "v_add_u32_dpp v132, v132, v132 row_shr:2 row_mask:0xf bank_mask:0xf bound_ctrl:0\n\t" \
  "v_add_u32_dpp v133, v133, v133 row_shr:2 row_mask:0xf bank_mask:0xf bound_ctrl:0\n\t" \
  "v_add_u32_dpp v134, v134, v134 row_shr:2 row_mask:0xf bank_mask:0xf bound_ctrl:0\n\t" \
  "v_add_u32_dpp v135, v135, v135 row_shr:2 row_mask:0xf bank_mask:0xf bound_ctrl:0\n\t" \
  "v_add_u32_dpp v132, v132, v132 row_shr:4 row_mask:0xf bank_mask:0xf bound_ctrl:0\n\t" \
  "v_add_u32_dpp v133, v133, v133 row_shr:4 row_mask:0xf bank_mask:0xf bound_ctrl:0\n\t" \
  "v_add_u32_dpp v134, v134, v134 row_shr:4 row_mask:0xf bank_mask:0xf bound_ctrl:0\n\t" \
  "v_add_u32_dpp v135, v135, v135 row_shr:4 row_mask:0xf bank_mask:0xf bound_ctrl:0\n\t" \
  "v_add_u32_dpp v132, v132, v132 row_shr:8 row_mask:0xf bank_mask:0xf bound_ctrl:0\n\t" \
  "v_add_u32_dpp v133, v133, v133 row_shr:8 row_mask:0xf bank_mask:0xf bound_ctrl:0\n\t" \
  "v_add_u32_dpp v134, v134, v134 row_shr:8 row_mask:0xf bank_mask:0xf bound_ctrl:0\n\t" \
  "v_add_u32_dpp v135, v135, v135 row_shr:8 row_mask:0xf bank_mask:0xf bound_ctrl:0\n\t" \
  "v_add_u32_dpp v132, v132, v132 row_bcast:15 row_mask:0xa bank_mask:0xf\n\t" \
  "v_add_u32_dpp v133, v133, v133 row_bcast:15 row_mask:0xa bank_mask:0xf\n\t" \
  "v_add_u32_dpp v134, v134, v134 row_bcast:15 row_mask:0xa bank_mask:0xf\n\t" \
  "v_add_u32_dpp v135, v135, v135 row_bcast:15 row_mask:0xa bank_mask:0xf\n\t" \
  "v_add_u32_dpp v132, v132, v132 row_bcast:31 row_mask:0xc bank_mask:0xf\n\t" \
  "v_add_u32_dpp v133, v133, v133 row_bcast:31 row_mask:0xc bank_mask:0xf\n\t" \
  "v_add_u32_dpp v134, v134, v134 row_bcast:31 row_mask:0xc bank_mask:0xf\n\t" \
  "v_add_u32_dpp v135, v135, v135 row_bcast:31 row_mask:0xc bank_mask:0xf\n\t" \
  "v_sub_u32 v128, v128, v132\n\t" \
  "v_sub_u32 v129, v129, v133\n\t" \
  "v_sub_u32 v130, v130, v134\n\t" \
  "v_sub_u32 v131, v131, v135\n\t" \
  "v_readlane_b32 %[n0], v132, 63\n\t" \
  "v_readlane_b32 %[n1], v133, 63\n\t" \
  "v_readlane_b32 %[n2], v134, 63\n\t" \
  "v_readlane_b32 %[n3], v135, 63\n\t" \
  "v_lshl_add_u32 %[p0], v128, 2, %[pb]\n\t" \
  "v_lshl_add_u32 %[p1], v129, 2, %[pb]\n\t" \
  "v_lshl_add_u32 %[p2], v130, 2, %[pb]\n\t" \
  "v_lshl_add_u32 %[p3], v131, 2, %[pb]\n\t" \
  ""
#define ASM_ENC2_0 \
  "s_lshl_b32 %[t], %[cnt], 2\n\t" \
  "v_mov_b32 v128, 0\n\t" \
  "v_add_u32 v129, %[t], %[c4]\n\t" \
  "s_mov_b32 %[nexc], 0\n\t" \
  "ds_write_b32 v129, v128\n\t" \
  "ds_write_b32 %[c4], %[m] offset:6400\n\t" \
  "v_ashrrev_i32 v128, 31, v0\n\t" \
  "v_bfi_b32 v132, v128, %[dum], %[p]\n\t" \
  "v_lshl_add_u32 %[p], v128, 2, %[p]\n\t" \
  "ds_write_b32 v132, v0 offset:0\n\t" \
  "v_ashrrev_i32 v129, 31, v4\n\t" \
  "v_bfi_b32 v133, v129, %[dum], %[p]\n\t" \
  "v_lshl_add_u32 %[p], v129, 2, %[p]\n\t" \
  "ds_write_b32 v133, v4 offset:4\n\t" \
  "v_ashrrev_i32 v130, 31, v8\n\t" \
  "v_bfi_b32 v134, v130, %[dum], %[p]\n\t" \
  "v_lshl_add_u32 %[p], v130, 2, %[p]\n\t" \
  "ds_write_b32 v134, v8 offset:8\n\t" \
  "v_ashrrev_i32 v131, 31, v12\n\t" \
  "v_bfi_b32 v135, v131, %[dum], %[p]\n\t" \
  "v_lshl_add_u32 %[p], v131, 2, %[p]\n\t" \
  "ds_write_b32 v135, v12 offset:12\n\t" \
  "v_ashrrev_i32 v128, 31, v1\n\t" \
  "v_bfi_b32 v132, v128, %[dum], %[p]\n\t" \
  "v_lshl_add_u32 %[p], v128, 2, %[p]\n\t" \
  "ds_write_b32 v132, v1 offset:16\n\t" \
  "v_ashrrev_i32 v129, 31, v5\n\t" \
  "v_bfi_b32 v133, v129, %[dum], %[p]\n\t" \
  "v_lshl_add_u32 %[p], v129, 2, %[p]\n\t" \
  "ds_write_b32 v133, v5 offset:20\n\t" \
  "v_ashrrev_i32 v130, 31, v9\n\t" \
  "v_bfi_b32 v134, v130, %[dum], %[p]\n\t" \
  "v_lshl_add_u32 %[p], v130, 2, %[p]\n\t" \
  "ds_write_b32 v134, v9 offset:24\n\t" \
  "v_ashrrev_i32 v131, 31, v13\n\t" \
  "v_bfi_b32 v135, v131, %[dum], %[p]\n\t" \
  "v_lshl_add_u32 %[p], v131, 2, %[p]\n\t" \
  "ds_write_b32 v135, v13 offset:28\n\t" \
  "v_ashrrev_i32 v128, 31, v2\n\t" \
  "v_bfi_b32 v132, v128, %[dum], %[p]\n\t" \
  "v_lshl_add_u32 %[p], v128, 2, %[p]\n\t" \
  "ds_write_b32 v132, v2 offset:32\n\t" \
  "v_ashrrev_i32 v129, 31, v6\n\t" \
  "v_bfi_b32 v133, v129, %[dum], %[p]\n\t" \
  "v_lshl_add_u32 %[p], v129, 2, %[p]\n\t" \
  "ds_write_b32 v133, v6 offset:36\n\t" \
  "v_ashrrev_i32 v130, 31, v10\n\t" \
  "v_bfi_b32 v134, v130, %[dum], %[p]\n\t" \
  "v_lshl_add_u32 %[p], v130, 2, %[p]\n\t" \
  "ds_write_b32 v134, v10 offset:40\n\t" \
  "v_ashrrev_i32 v131, 31, v14\n\t" \
  "v_bfi_b32 v135, v131, %[dum], %[p]\n\t" \
  "v_lshl_add_u32 %[p], v131, 2, %[p]\n\t" \
  "ds_write_b32 v135, v14 offset:44\n\t" \
  "v_ashrrev_i32 v128, 31, v3\n\t" \
  "v_bfi_b32 v132, v128, %[dum], %[p]\n\t" \
  "v_lshl_add_u32 %[p], v128, 2, %[p]\n\t" \
  "ds_write_b32 v132, v3 offset:48\n\t" \
  "v_ashrrev_i32 v129, 31, v7\n\t" \
  "v_bfi_b32 v133, v129, %[dum], %[p]\n\t" \
  "v_lshl_add_u32 %[p], v129, 2, %[p]\n\t" \
  "ds_write_b32 v133, v7 offset:52\n\t" \
  "v_ashrrev_i32 v130, 31, v11\n\t" \
  "v_bfi_b32 v134, v130, %[dum], %[p]\n\t" \
  "v_lshl_add_u32 %[p], v130, 2, %[p]\n\t" \
  "ds_write_b32 v134, v11 offset:56\n\t" \
  "v_ashrrev_i32 v131, 31, v15\n\t" \
  "v_bfi_b32 v135, v131, %[dum], %[p]\n\t" \
  "v_lshl_add_u32 %[p], v131, 2, %[p]\n\t" \
  "ds_write_b32 v135, v15 offset:60\n\t" \
  "v_ashrrev_i32 v128, 31, v16\n\t" \
  "v_bfi_b32 v132, v128, %[dum], %[p]\n\t" \
  "v_lshl_add_u32 %[p], v128, 2, %[p]\n\t" \
  "ds_write_b32 v132, v16 offset:64\n\t" \
  "v_ashrrev_i32 v129, 31, v20\n\t" \
  "v_bfi_b32 v133, v129, %[dum], %[p]\n\t" \
  "v_lshl_add_u32 %[p], v129, 2, %[p]\n\t" \
  "ds_write_b32 v133, v20 offset:68\n\t" \
  "v_ashrrev_i32 v130, 31, v24\n\t" \
  "v_bfi_b32 v134, v130, %[dum], %[p]\n\t" \
  "v_lshl_add_u32 %[p], v130, 2, %[p]\n\t" \
  "ds_write_b32 v134, v24 offset:72\n\t" \
  "v_ashrrev_i32 v131, 31, v28\n\t" \
  "v_bfi_b32 v135, v131, %[dum], %[p]\n\t" \
  "v_lshl_add_u32 %[p], v131, 2, %[p]\n\t" \
  "ds_write_b32 v135, v28 offset:76\n\t" \
  "v_ashrrev_i32 v128, 31, v17\n\t" \
  "v_bfi_b32 v132, v128, %[dum], %[p]\n\t" \
  "v_lshl_add_u32 %[p], v128, 2, %[p]\n\t" \
  "ds_write_b32 v132, v17 offset:80\n\t" \
  "v_ashrrev_i32 v129, 31, v21\n\t" \
  "v_bfi_b32 v133, v129, %[dum], %[p]\n\t" \
  "v_lshl_add_u32 %[p], v129, 2, %[p]\n\t" \
  "ds_write_b32 v133, v21 offset:84\n\t" \
  "v_ashrrev_i32 v130, 31, v25\n\t" \
  "v_bfi_b32 v134, v130, %[dum], %[p]\n\t" \
  "v_lshl_add_u32 %[p], v130, 2, %[p]\n\t" \
  "ds_write_b32 v134, v25 offset:88\n\t" \
  "v_ashrrev_i32 v131, 31, v29\n\t" \
  "v_bfi_b32 v135, v131, %[dum], %[p]\n\t" \
  "v_lshl_add_u32 %[p], v131, 2, %[p]\n\t" \
  "ds_write_b32 v135, v29 offset:92\n\t" \
  "v_ashrrev_i32 v128, 31, v18\n\t" \
  "v_bfi_b32 v132, v128, %[dum], %[p]\n\t" \
  "v_lshl_add_u32 %[p], v128, 2, %[p]\n\t" \
  "ds_write_b32 v132, v18 offset:96\n\t" \
  "v_ashrrev_i32 v129, 31, v22\n\t" \
  "v_bfi_b32 v133, v129, %[dum], %[p]\n\t" \
  "v_lshl_add_u32 %[p], v129, 2, %[p]\n\t" \
  "ds_write_b32 v133, v22 offset:100\n\t" \
  "v_ashrrev_i32 v130, 31, v26\n\t" \
  "v_bfi_b32 v134, v130, %[dum], %[p]\n\t" \
  "v_lshl_add_u32 %[p], v130, 2, %[p]\n\t" \
  "ds_write_b32 v134, v26 offset:104\n\t" \
  "v_ashrrev_i32 v131, 31, v30\n\t" \
  "v_bfi_b32 v135, v131, %[dum], %[p]\n\t" \
  "v_lshl_add_u32 %[p], v131, 2, %[p]\n\t" \
  "ds_write_b32 v135, v30 offset:108\n\t" \
  "v_ashrrev_i32 v128, 31, v19\n\t" \
  "v_bfi_b32 v132, v128, %[dum], %[p]\n\t" \
  "v_lshl_add_u32 %[p], v128, 2, %[p]\n\t" \
  "ds_write_b32 v132, v19 offset:112\n\t" \
  "v_ashrrev_i32 v129, 31, v23\n\t" \
  "v_bfi_b32 v133, v129, %[dum], %[p]\n\t" \
  "v_lshl_add_u32 %[p], v129, 2, %[p]\n\t" \
  "ds_write_b32 v133, v23 offset:116\n\t" \
  "v_ashrrev_i32 v130, 31, v27\n\t" \
  "v_bfi_b32 v134, v130, %[dum], %[p]\n\t" \
  "v_lshl_add_u32 %[p], v130, 2, %[p]\n\t" \
  "ds_write_b32 v134, v27 offset:120\n\t" \
  "v_ashrrev_i32 v131, 31, v31\n\t" \
  "v_bfi_b32 v135, v131, %[dum], %[p]\n\t" \
  "v_lshl_add_u32 %[p], v131, 2, %[p]\n\t" \
  "ds_write_b32 v135, v31 offset:124\n\t" \
  "ds_read_b128 v[136:139], %[c32] offset:0\n\t" \
  "ds_read_b128 v[140:143], %[c32] offset:16\n\t" \
  "ds_read_b128 v[144:147], %[c32] offset:2048\n\t" \
  "ds_read_b128 v[148:151], %[c32] offset:2064\n\t" \
  "ds_read_b128 v[152:155], %[c32] offset:4096\n\t" \
  "ds_read_b128 v[156:159], %[c32] offset:4112\n\t" \
  "s_lshl_b32 %[t], %[nch], 2\n\t" \
  "s_add_u32 %[rb], %[t], %[rec0]\n\t" \
  "s_mov_b32 %[t2], %[nch]\n\t" \
  "v_cmpx_gt_i32 vcc, %[t2], %[lane]\n\t" \
  "s_cbranch_execz 4f\n\t" \
  "s_waitcnt lgkmcnt(4)\n\t" \
  "v_perm_b32 v160, v137, v136, %[s10]\n\t" \
  "v_perm_b32 v161, v139, v138, %[s10]\n\t" \
  "v_perm_b32 v162, v141, v140, %[s10]\n\t" \
  "v_perm_b32 v163, v143, v142, %[s10]\n\t" \
  "v_perm_b32 v164, v138, v136, %[s62]\n\t" \
  "v_perm_b32 v166, v142, v140, %[s62]\n\t" \
  "v_perm_b32 v165, v139, v137, %[s62]\n\t" \
  "v_perm_b32 v167, v143, v141, %[s62]\n\t" \
  "global_store_dwordx4 %[so], v[160:163], %[wp] offset:0 sc0 sc1\n\t" \
  "v_perm_b32 v164, v166, v164, %[s10]\n\t" \
  "v_perm_b32 v165, v167, v165, %[s10]\n\t" \
  "v_or3_b32 v166, v136, v137, v138\n\t" \
  "v_or3_b32 v167, v139, v140, v141\n\t" \
  "v_lshlrev_b32 v168, 4, v165\n\t" \
  "v_or3_b32 v166, v142, v143, v166\n\t" \
  "v_bfi_b32 v168, %[k0f], v164, v168\n\t" \
  "v_or_b32 v166, v166, v167\n\t" \
  "ds_write_b32 %[c4], v168 offset:6656\n\t" \
  "v_cmp_lt_u32 vcc, %[lim], v166\n\t" \
  "s_cbranch_vccz 1f\n\t" \
  "v_lshrrev_b32 v164, 4, v164\n\t" \
  "v_perm_b32 v166, v138, v136, %[s73]\n\t" \
  "v_perm_b32 v167, v142, v140, %[s73]\n\t" \
  "v_bfi_b32 v164, %[k0f], v164, v165\n\t" \
  "v_perm_b32 v166, v167, v166, %[s10]\n\t" \
  "v_perm_b32 v167, v139, v137, %[s73]\n\t" \
  "v_perm_b32 v168, v143, v141, %[s73]\n\t" \
  "v_mbcnt_lo_u32_b32 v165, vcc_lo, 0\n\t" \
  "v_perm_b32 v167, v168, v167, %[s10]\n\t" \
  "v_mbcnt_hi_u32_b32 v165, vcc_hi, v165\n\t" \
  "v_lshl_or_b32 v166, v167, 2, v166\n\t" \
  "v_add_u32 v165, %[nexc], v165\n\t" \
  "v_add_u32 v168, 0x0, %[vx]\n\t" \
  "v_min_u32 v165, 63, v165\n\t" \
  "v_and_or_b32 v166, v166, %[k0f], v168\n\t" \
  "v_lshl_add_u32 v165, v165, 3, %[rb]\n\t" \
  "s_bcnt1_i32_b64 %[t], vcc\n\t" \
  "s_mov_b64 exec, vcc\n\t" \
  "ds_write2_b32 v165, v164, v166 offset1:1\n\t" \
  "s_add_u32 %[nexc], %[nexc], %[t]\n\t" \
  "1:\n" \
  "s_mov_b64 exec, -1\n\t" \
  "s_sub_u32 %[t2], %[t2], 64\n\t" \
  "v_cmpx_gt_i32 vcc, %[t2], %[lane]\n\t" \
  "s_cbranch_execz 4f\n\t" \
  "s_waitcnt lgkmcnt(3)\n\t" \
  "v_perm_b32 v160, v145, v144, %[s10]\n\t" \
  "v_perm_b32 v161, v147, v146, %[s10]\n\t" \
  "v_perm_b32 v162, v149, v148, %[s10]\n\t" \
  "v_perm_b32 v163, v151, v150, %[s10]\n\t" \
  "v_perm_b32 v164, v146, v144, %[s62]\n\t" \
  "v_perm_b32 v166, v150, v148, %[s62]\n\t" \
  "v_perm_b32 v165, v147, v145, %[s62]\n\t" \
  "v_perm_b32 v167, v151, v149, %[s62]\n\t" \
  "global_store_dwordx4 %[so], v[160:163], %[wp] offset:1024 sc0 sc1\n\t" \
  "v_perm_b32 v164, v166, v164, %[s10]\n\t" \
  "v_perm_b32 v165, v167, v165, %[s10]\n\t" \
  "v_or3_b32 v166, v144, v145, v146\n\t" \
  "v_or3_b32 v167, v147, v148, v149\n\t" \
  "v_lshlrev_b32 v168, 4, v165\n\t" \
  "v_or3_b32 v166, v150, v151, v166\n\t" \
  "v_bfi_b32 v168, %[k0f], v164, v168\n\t" \
  "v_or_b32 v166, v166, v167\n\t" \
  "ds_write_b32 %[c4], v168 offset:6912\n\t" \
  "v_cmp_lt_u32 vcc, %[lim], v166\n\t" \
  "s_cbranch_vccz 2f\n\t" \
  "v_lshrrev_b32 v164, 4, v164\n\t" \
  "v_perm_b32 v166, v146, v144, %[s73]\n\t" \
  "v_perm_b32 v167, v150, v148, %[s73]\n\t" \
  "v_bfi_b32 v164, %[k0f], v164, v165\n\t" \
  "v_perm_b32 v166, v167, v166, %[s10]\n\t" \
  "v_perm_b32 v167, v147, v145, %[s73]\n\t" \
  "v_perm_b32 v168, v151, v149, %[s73]\n\t" \
  "v_mbcnt_lo_u32_b32 v165, vcc_lo, 0\n\t" \
  "v_perm_b32 v167, v168, v167, %[s10]\n\t" \
  "v_mbcnt_hi_u32_b32 v165, vcc_hi, v165\n\t" \
  "v_lshl_or_b32 v166, v167, 2, v166\n\t" \
  "v_add_u32 v165, %[nexc], v165\n\t" \
  "v_add_u32 v168, 0x4000, %[vx]\n\t" \
  "v_min_u32 v165, 63, v165\n\t" \
  "v_and_or_b32 v166, v166, %[k0f], v168\n\t" \
  "v_lshl_add_u32 v165, v165, 3, %[rb]\n\t" \
  "s_bcnt1_i32_b64 %[t], vcc\n\t" \
  "s_mov_b64 exec, vcc\n\t" \
  "ds_write2_b32 v165, v164, v166 offset1:1\n\t" \
  "s_add_u32 %[nexc], %[nexc], %[t]\n\t" \
  "2:\n" \
  "s_mov_b64 exec, -1\n\t" \
  "s_sub_u32 %[t2], %[t2], 64\n\t" \
  "v_cmpx_gt_i32 vcc, %[t2], %[lane]\n\t" \
  "s_cbranch_execz 4f\n\t" \
  "s_waitcnt lgkmcnt(2)\n\t" \
  "v_perm_b32 v160, v153, v152, %[s10]\n\t" \
  "v_perm_b32 v161, v155, v154, %[s10]\n\t" \
  "v_perm_b32 v162, v157, v156, %[s10]\n\t" \
  "v_perm_b32 v163, v159, v158, %[s10]\n\t" \
  "v_perm_b32 v164, v154, v152, %[s62]\n\t" \
  "v_perm_b32 v166, v158, v156, %[s62]\n\t" \
  "v_perm_b32 v165, v155, v153, %[s62]\n\t" \
  "v_perm_b32 v167, v159, v157, %[s62]\n\t" \
  "global_store_dwordx4 %[so], v[160:163], %[wp] offset:2048 sc0 sc1\n\t" \
  "v_perm_b32 v164, v166, v164, %[s10]\n\t" \
  "v_perm_b32 v165, v167, v165, %[s10]\n\t" \
  "v_or3_b32 v166, v152, v153, v154\n\t" \
  "v_or3_b32 v167, v155, v156, v157\n\t" \
  "v_lshlrev_b32 v168, 4, v165\n\t" \
  "v_or3_b32 v166, v158, v159, v166\n\t" \
  "v_bfi_b32 v168, %[k0f], v164, v168\n\t" \
  "v_or_b32 v166, v166, v167\n\t" \
  "ds_write_b32 %[c4], v168 offset:7168\n\t" \
  "v_cmp_lt_u32 vcc, %[lim], v166\n\t" \
  "s_cbranch_vccz 3f\n\t" \
  "v_lshrrev_b32 v164, 4, v164\n\t" \
  "v_perm_b32 v166, v154, v152, %[s73]\n\t" \
  "v_perm_b32 v167, v158, v156, %[s73]\n\t" \
  "v_bfi_b32 v164, %[k0f], v164, v165\n\t" \
  "v_perm_b32 v166, v167, v166, %[s10]\n\t" \
  "v_perm_b32 v167, v155, v153, %[s73]\n\t" \
  "v_perm_b32 v168, v159, v157, %[s73]\n\t" \
  "v_mbcnt_lo_u32_b32 v165, vcc_lo, 0\n\t" \
  "v_perm_b32 v167, v168, v167, %[s10]\n\t" \
  "v_mbcnt_hi_u32_b32 v165, vcc_hi, v165\n\t" \
  "v_lshl_or_b32 v166, v167, 2, v166\n\t" \
  "v_add_u32 v165, %[nexc], v165\n\t" \
  "v_add_u32 v168, 0x8000, %[vx]\n\t" \
  "v_min_u32 v165, 63, v165\n\t" \
  "v_and_or_b32 v166, v166, %[k0f], v168\n\t" \
  "v_lshl_add_u32 v165, v165, 3, %[rb]\n\t" \
  "s_bcnt1_i32_b64 %[t], vcc\n\t" \
  "s_mov_b64 exec, vcc\n\t" \
  "ds_write2_b32 v165, v164, v166 offset1:1\n\t" \
  "s_add_u32 %[nexc], %[nexc], %[t]\n\t" \
  "3:\n" \
  "s_mov_b64 exec, -1\n\t" \
  "4:\n" \
  "s_mov_b64 exec, -1\n\t" \
  "ds_read_b128 v[136:139], %[t16]\n\t" \
  "ds_read_b128 v[140:143], %[t16] offset:1024\n\t" \
  "s_lshl_b32 %[t], %[nch], 4\n\t" \
  "s_min_u32 %[t2], %[nexc], 64\n\t" \
  "v_add_u32 v129, %[t], %[so]\n\t" \
  "s_sub_u32 %[t], 0x1000, %[t]\n\t" \
  "s_lshl_b32 %[t2], %[t2], 3\n\t" \
  "s_add_u32 %[t2], %[t2], %[rb]\n\t" \
  "s_sub_u32 %[t2], %[t2], %[rec0]\n\t" \
  "s_add_u32 %[t2], %[t2], 0x100\n\t" \
  "s_min_u32 %[t2], %[t2], %[t]\n\t" \
  "v_cmp_gt_i32 vcc, %[t2], %[l16]\n\t" \
  "s_sub_u32 %[t2], %[t2], 0x400\n\t" \
  "s_mov_b64 exec, vcc\n\t" \
  "v_cmp_gt_i32 vcc, %[t2], %[l16]\n\t" \
  "s_waitcnt lgkmcnt(0)\n\t" \
  "global_store_dwordx4 v129, v[136:139], %[wp] sc0 sc1\n\t" \
  "s_mov_b64 exec, vcc\n\t" \
  "global_store_dwordx4 v129, v[140:143], %[wp] offset:1024 sc0 sc1\n\t" \
  "s_mov_b64 exec, -1\n\t" \
  "s_nop 1\n\t" \
  ""
#define ASM_ENC2_1 \
  "s_lshl_b32 %[t], %[cnt], 2\n\t" \
  "v_mov_b32 v128, 0\n\t" \
  "v_add_u32 v129, %[t], %[c4]\n\t" \
  "s_mov_b32 %[nexc], 0\n\t" \
  "ds_write_b32 v129, v128\n\t" \
  "ds_write_b32 %[c4], %[m] offset:6400\n\t" \
  "v_ashrrev_i32 v128, 31, v32\n\t" \
  "v_bfi_b32 v132, v128, %[dum], %[p]\n\t" \
  "v_lshl_add_u32 %[p], v128, 2, %[p]\n\t" \
  "ds_write_b32 v132, v32 offset:0\n\t" \
  "v_ashrrev_i32 v129, 31, v36\n\t" \
  "v_bfi_b32 v133, v129, %[dum], %[p]\n\t" \
  "v_lshl_add_u32 %[p], v129, 2, %[p]\n\t" \
  "ds_write_b32 v133, v36 offset:4\n\t" \
  "v_ashrrev_i32 v130, 31, v40\n\t" \
  "v_bfi_b32 v134, v130, %[dum], %[p]\n\t" \
  "v_lshl_add_u32 %[p], v130, 2, %[p]\n\t" \
  "ds_write_b32 v134, v40 offset:8\n\t" \
  "v_ashrrev_i32 v131, 31, v44\n\t" \
  "v_bfi_b32 v135, v131, %[dum], %[p]\n\t" \
  "v_lshl_add_u32 %[p], v131, 2, %[p]\n\t" \
  "ds_write_b32 v135, v44 offset:12\n\t" \
  "v_ashrrev_i32 v128, 31, v33\n\t" \
  "v_bfi_b32 v132, v128, %[dum], %[p]\n\t" \
  "v_lshl_add_u32 %[p], v128, 2, %[p]\n\t" \
  "ds_write_b32 v132, v33 offset:16\n\t" \
  "v_ashrrev_i32 v129, 31, v37\n\t" \
  "v_bfi_b32 v133, v129, %[dum], %[p]\n\t" \
  "v_lshl_add_u32 %[p], v129, 2, %[p]\n\t" \
  "ds_write_b32 v133, v37 offset:20\n\t" \
  "v_ashrrev_i32 v130, 31, v41\n\t" \
  "v_bfi_b32 v134, v130, %[dum], %[p]\n\t" \
  "v_lshl_add_u32 %[p], v130, 2, %[p]\n\t" \
  "ds_write_b32 v134, v41 offset:24\n\t" \
  "v_ashrrev_i32 v131, 31, v45\n\t" \
  "v_bfi_b32 v135, v131, %[dum], %[p]\n\t" \
  "v_lshl_add_u32 %[p], v131, 2, %[p]\n\t" \
  "ds_write_b32 v135, v45 offset:28\n\t" \
  "v_ashrrev_i32 v128, 31, v34\n\t" \
  "v_bfi_b32 v132, v128, %[dum], %[p]\n\t" \
  "v_lshl_add_u32 %[p], v128, 2, %[p]\n\t" \
  "ds_write_b32 v132, v34 offset:32\n\t" \
  "v_ashrrev_i32 v129, 31, v38\n\t" \
  "v_bfi_b32 v133, v129, %[dum], %[p]\n\t" \
  "v_lshl_add_u32 %[p], v129, 2, %[p]\n\t" \
  "ds_write_b32 v133, v38 offset:36\n\t" \
  "v_ashrrev_i32 v130, 31, v42\n\t" \
  "v_bfi_b32 v134, v130, %[dum], %[p]\n\t" \
  "v_lshl_add_u32 %[p], v130, 2, %[p]\n\t" \
  "ds_write_b32 v134, v42 offset:40\n\t" \
  "v_ashrrev_i32 v131, 31, v46\n\t" \
  "v_bfi_b32 v135, v131, %[dum], %[p]\n\t" \
  "v_lshl_add_u32 %[p], v131, 2, %[p]\n\t" \
  "ds_write_b32 v135, v46 offset:44\n\t" \
  "v_ashrrev_i32 v128, 31, v35\n\t" \
  "v_bfi_b32 v132, v128, %[dum], %[p]\n\t" \
  "v_lshl_add_u32 %[p], v128, 2, %[p]\n\t" \
  "ds_write_b32 v132, v35 offset:48\n\t" \
  "v_ashrrev_i32 v129, 31, v39\n\t" \
  "v_bfi_b32 v133, v129, %[dum], %[p]\n\t" \
  "v_lshl_add_u32 %[p], v129, 2, %[p]\n\t" \
  "ds_write_b32 v133, v39 offset:52\n\t" \
  "v_ashrrev_i32 v130, 31, v43\n\t" \
  "v_bfi_b32 v134, v130, %[dum], %[p]\n\t" \
  "v_lshl_add_u32 %[p], v130, 2, %[p]\n\t" \
  "ds_write_b32 v134, v43 offset:56\n\t" \
  "v_ashrrev_i32 v131, 31, v47\n\t" \
  "v_bfi_b32 v135, v131, %[dum], %[p]\n\t" \
  "v_lshl_add_u32 %[p], v131, 2, %[p]\n\t" \
  "ds_write_b32 v135, v47 offset:60\n\t" \
  "v_ashrrev_i32 v128, 31, v48\n\t" \
  "v_bfi_b32 v132, v128, %[dum], %[p]\n\t" \
  "v_lshl_add_u32 %[p], v128, 2, %[p]\n\t" \
  "ds_write_b32 v132, v48 offset:64\n\t" \
  "v_ashrrev_i32 v129, 31, v52\n\t" \
  "v_bfi_b32 v133, v129, %[dum], %[p]\n\t" \
  "v_lshl_add_u32 %[p], v129, 2, %[p]\n\t" \
  "ds_write_b32 v133, v52 offset:68\n\t" \
  "v_ashrrev_i32 v130, 31, v56\n\t" \
  "v_bfi_b32 v134, v130, %[dum], %[p]\n\t" \
  "v_lshl_add_u32 %[p], v130, 2, %[p]\n\t" \
  "ds_write_b32 v134, v56 offset:72\n\t" \
  "v_ashrrev_i32 v131, 31, v60\n\t" \
  "v_bfi_b32 v135, v131, %[dum], %[p]\n\t" \
  "v_lshl_add_u32 %[p], v131, 2, %[p]\n\t" \
  "ds_write_b32 v135, v60 offset:76\n\t" \
  "v_ashrrev_i32 v128, 31, v49\n\t" \
  "v_bfi_b32 v132, v128, %[dum], %[p]\n\t" \
  "v_lshl_add_u32 %[p], v128, 2, %[p]\n\t" \
  "ds_write_b32 v132, v49 offset:80\n\t" \
  "v_ashrrev_i32 v129, 31, v53\n\t" \
  "v_bfi_b32 v133, v129, %[dum], %[p]\n\t" \
  "v_lshl_add_u32 %[p], v129, 2, %[p]\n\t" \
  "ds_write_b32 v133, v53 offset:84\n\t" \
  "v_ashrrev_i32 v130, 31, v57\n\t" \
  "v_bfi_b32 v134, v130, %[dum], %[p]\n\t" \
  "v_lshl_add_u32 %[p], v130, 2, %[p]\n\t" \
  "ds_write_b32 v134, v57 offset:88\n\t" \
  "v_ashrrev_i32 v131, 31, v61\n\t" \
  "v_bfi_b32 v135, v131, %[dum], %[p]\n\t" \
  "v_lshl_add_u32 %[p], v131, 2, %[p]\n\t" \
  "ds_write_b32 v135, v61 offset:92\n\t" \
  "v_ashrrev_i32 v128, 31, v50\n\t" \
  "v_bfi_b32 v132, v128, %[dum], %[p]\n\t" \
  "v_lshl_add_u32 %[p], v128, 2, %[p]\n\t" \
  "ds_write_b32 v132, v50 offset:96\n\t" \
  "v_ashrrev_i32 v129, 31, v54\n\t" \
  "v_bfi_b32 v133, v129, %[dum], %[p]\n\t" \
  "v_lshl_add_u32 %[p], v129, 2, %[p]\n\t" \
  "ds_write_b32 v133, v54 offset:100\n\t" \
  "v_ashrrev_i32 v130, 31, v58\n\t" \
  "v_bfi_b32 v134, v130, %[dum], %[p]\n\t" \
  "v_lshl_add_u32 %[p], v130, 2, %[p]\n\t" \
  "ds_write_b32 v134, v58 offset:104\n\t" \
  "v_ashrrev_i32 v131, 31, v62\n\t" \
  "v_bfi_b32 v135, v131, %[dum], %[p]\n\t" \
  "v_lshl_add_u32 %[p], v131, 2, %[p]\n\t" \
  "ds_write_b32 v135, v62 offset:108\n\t" \
  "v_ashrrev_i32 v128, 31, v51\n\t" \
  "v_bfi_b32 v132, v128, %[dum], %[p]\n\t" \
  "v_lshl_add_u32 %[p], v128, 2, %[p]\n\t" \
  "ds_write_b32 v132, v51 offset:112\n\t" \
  "v_ashrrev_i32 v129, 31, v55\n\t" \
  "v_bfi_b32 v133, v129, %[dum], %[p]\n\t" \
  "v_lshl_add_u32 %[p], v129, 2, %[p]\n\t" \
  "ds_write_b32 v133, v55 offset:116\n\t" \
  "v_ashrrev_i32 v130, 31, v59\n\t" \
  "v_bfi_b32 v134, v130, %[dum], %[p]\n\t" \
  "v_lshl_add_u32 %[p], v130, 2, %[p]\n\t" \
  "ds_write_b32 v134, v59 offset:120\n\t" \
  "v_ashrrev_i32 v131, 31, v63\n\t" \
  "v_bfi_b32 v135, v131, %[dum], %[p]\n\t" \
  "v_lshl_add_u32 %[p], v131, 2, %[p]\n\t" \
  "ds_write_b32 v135, v63 offset:124\n\t" \
  "ds_read_b128 v[136:139], %[c32] offset:0\n\t" \
  "ds_read_b128 v[140:143], %[c32] offset:16\n\t" \
  "ds_read_b128 v[144:147], %[c32] offset:2048\n\t" \
  "ds_read_b128 v[148:151], %[c32] offset:2064\n\t" \
  "ds_read_b128 v[152:155], %[c32] offset:4096\n\t" \
  "ds_read_b128 v[156:159], %[c32] offset:4112\n\t" \
  "s_lshl_b32 %[t], %[nch], 2\n\t" \
  "s_add_u32 %[rb], %[t], %[rec0]\n\t" \
  "s_mov_b32 %[t2], %[nch]\n\t" \
  "v_cmpx_gt_i32 vcc, %[t2], %[lane]\n\t" \
  "s_cbranch_execz 4f\n\t" \
  "s_waitcnt lgkmcnt(4)\n\t" \
  "v_perm_b32 v160, v137, v136, %[s10]\n\t" \
  "v_perm_b32 v161, v139, v138, %[s10]\n\t" \
  "v_perm_b32 v162, v141, v140, %[s10]\n\t" \
  "v_perm_b32 v163, v143, v142, %[s10]\n\t" \
  "v_perm_b32 v164, v138, v136, %[s62]\n\t" \
  "v_perm_b32 v166, v142, v140, %[s62]\n\t" \
  "v_perm_b32 v165, v139, v137, %[s62]\n\t" \
  "v_perm_b32 v167, v143, v141, %[s62]\n\t" \
  "global_store_dwordx4 %[so], v[160:163], %[wp] offset:0 sc0 sc1\n\t" \
  "v_perm_b32 v164, v166, v164, %[s10]\n\t" \
  "v_perm_b32 v165, v167, v165, %[s10]\n\t" \
  "v_or3_b32 v166, v136, v137, v138\n\t" \
  "v_or3_b32 v167, v139, v140, v141\n\t" \
  "v_lshlrev_b32 v168, 4, v165\n\t" \
  "v_or3_b32 v166, v142, v143, v166\n\t" \
  "v_bfi_b32 v168, %[k0f], v164, v168\n\t" \
  "v_or_b32 v166, v166, v167\n\t" \
  "ds_write_b32 %[c4], v168 offset:6656\n\t" \
  "v_cmp_lt_u32 vcc, %[lim], v166\n\t" \
  "s_cbranch_vccz 1f\n\t" \
  "v_lshrrev_b32 v164, 4, v164\n\t" \
  "v_perm_b32 v166, v138, v136, %[s73]\n\t" \
  "v_perm_b32 v167, v142, v140, %[s73]\n\t" \
  "v_bfi_b32 v164, %[k0f], v164, v165\n\t" \
  "v_perm_b32 v166, v167, v166, %[s10]\n\t" \
  "v_perm_b32 v167, v139, v137, %[s73]\n\t" \
  "v_perm_b32 v168, v143, v141, %[s73]\n\t" \
  "v_mbcnt_lo_u32_b32 v165, vcc_lo, 0\n\t" \
  "v_perm_b32 v167, v168, v167, %[s10]\n\t" \
  "v_mbcnt_hi_u32_b32 v165, vcc_hi, v165\n\t" \
  "v_lshl_or_b32 v166, v167, 2, v166\n\t" \
  "v_add_u32 v165, %[nexc], v165\n\t" \
  "v_add_u32 v168, 0x0, %[vx]\n\t" \
  "v_min_u32 v165, 63, v165\n\t" \
  "v_and_or_b32 v166, v166, %[k0f], v168\n\t" \
  "v_lshl_add_u32 v165, v165, 3, %[rb]\n\t" \
  "s_bcnt1_i32_b64 %[t], vcc\n\t" \
  "s_mov_b64 exec, vcc\n\t" \
  "ds_write2_b32 v165, v164, v166 offset1:1\n\t" \
  "s_add_u32 %[nexc], %[nexc], %[t]\n\t" \
  "1:\n" \
  "s_mov_b64 exec, -1\n\t" \
  "s_sub_u32 %[t2], %[t2], 64\n\t" \
  "v_cmpx_gt_i32 vcc, %[t2], %[lane]\n\t" \
  "s_cbranch_execz 4f\n\t" \
  "s_waitcnt lgkmcnt(3)\n\t" \
  "v_perm_b32 v160, v145, v144, %[s10]\n\t" \
  "v_perm_b32 v161, v147, v146, %[s10]\n\t" \
  "v_perm_b32 v162, v149, v148, %[s10]\n\t" \
  "v_perm_b32 v163, v151, v150, %[s10]\n\t" \
  "v_perm_b32 v164, v146, v144, %[s62]\n\t" \
  "v_perm_b32 v166, v150, v148, %[s62]\n\t" \
  "v_perm_b32 v165, v147, v145, %[s62]\n\t" \
  "v_perm_b32 v167, v151, v149, %[s62]\n\t" \
  "global_store_dwordx4 %[so], v[160:163], %[wp] offset:1024 sc0 sc1\n\t" \
  "v_perm_b32 v164, v166, v164, %[s10]\n\t" \
  "v_perm_b32 v165, v167, v165, %[s10]\n\t" \
  "v_or3_b32 v166, v144, v145, v146\n\t" \
  "v_or3_b32 v167, v147, v148, v149\n\t" \
  "v_lshlrev_b32 v168, 4, v165\n\t" \
  "v_or3_b32 v166, v150, v151, v166\n\t" \
  "v_bfi_b32 v168, %[k0f], v164, v168\n\t" \
  "v_or_b32 v166, v166, v167\n\t" \
  "ds_write_b32 %[c4], v168 offset:6912\n\t" \
  "v_cmp_lt_u32 vcc, %[lim], v166\n\t" \
  "s_cbranch_vccz 2f\n\t" \
  "v_lshrrev_b32 v164, 4, v164\n\t" \
  "v_perm_b32 v166, v146, v144, %[s73]\n\t" \
  "v_perm_b32 v167, v150, v148, %[s73]\n\t" \
  "v_bfi_b32 v164, %[k0f], v164, v165\n\t" \
  "v_perm_b32 v166, v167, v166, %[s10]\n\t" \
  "v_perm_b32 v167, v147, v145, %[s73]\n\t" \
  "v_perm_b32 v168, v151, v149, %[s73]\n\t" \
  "v_mbcnt_lo_u32_b32 v165, vcc_lo, 0\n\t" \
  "v_perm_b32 v167, v168, v167, %[s10]\n\t" \
  "v_mbcnt_hi_u32_b32 v165, vcc_hi, v165\n\t" \
  "v_lshl_or_b32 v166, v167, 2, v166\n\t" \
  "v_add_u32 v165, %[nexc], v165\n\t" \
  "v_add_u32 v168, 0x4000, %[vx]\n\t" \
  "v_min_u32 v165, 63, v165\n\t" \
  "v_and_or_b32 v166, v166, %[k0f], v168\n\t" \
  "v_lshl_add_u32 v165, v165, 3, %[rb]\n\t" \
  "s_bcnt1_i32_b64 %[t], vcc\n\t" \
  "s_mov_b64 exec, vcc\n\t" \
  "ds_write2_b32 v165, v164, v166 offset1:1\n\t" \
  "s_add_u32 %[nexc], %[nexc], %[t]\n\t" \
  "2:\n" \
  "s_mov_b64 exec, -1\n\t" \
  "s_sub_u32 %[t2], %[t2], 64\n\t" \
  "v_cmpx_gt_i32 vcc, %[t2], %[lane]\n\t" \
  "s_cbranch_execz 4f\n\t" \
  "s_waitcnt lgkmcnt(2)\n\t" \
  "v_perm_b32 v160, v153, v152, %[s10]\n\t" \
  "v_perm_b32 v161, v155, v154, %[s10]\n\t" \
  "v_perm_b32 v162, v157, v156, %[s10]\n\t" \
  "v_perm_b32 v163, v159, v158, %[s10]\n\t" \
  "v_perm_b32 v164, v154, v152, %[s62]\n\t" \
  "v_perm_b32 v166, v158, v156, %[s62]\n\t" \
  "v_perm_b32 v165, v155, v153, %[s62]\n\t" \
  "v_perm_b32 v167, v159, v157, %[s62]\n\t" \
  "global_store_dwordx4 %[so], v[160:163], %[wp] offset:2048 sc0 sc1\n\t" \
  "v_perm_b32 v164, v166, v164, %[s10]\n\t" \
  "v_perm_b32 v165, v167, v165, %[s10]\n\t" \
  "v_or3_b32 v166, v152, v153, v154\n\t" \
  "v_or3_b32 v167, v155, v156, v157\n\t" \
  "v_lshlrev_b32 v168, 4, v165\n\t" \
  "v_or3_b32 v166, v158, v159, v166\n\t" \
  "v_bfi_b32 v168, %[k0f], v164, v168\n\t" \
  "v_or_b32 v166, v166, v167\n\t" \
  "ds_write_b32 %[c4], v168 offset:7168\n\t" \
  "v_cmp_lt_u32 vcc, %[lim], v166\n\t" \
  "s_cbranch_vccz 3f\n\t" \
  "v_lshrrev_b32 v164, 4, v164\n\t" \
  "v_perm_b32 v166, v154, v152, %[s73]\n\t" \
  "v_perm_b32 v167, v158, v156, %[s73]\n\t" \
  "v_bfi_b32 v164, %[k0f], v164, v165\n\t" \
  "v_perm_b32 v166, v167, v166, %[s10]\n\t" \
  "v_perm_b32 v167, v155, v153, %[s73]\n\t" \
  "v_perm_b32 v168, v159, v157, %[s73]\n\t" \
  "v_mbcnt_lo_u32_b32 v165, vcc_lo, 0\n\t" \
  "v_perm_b32 v167, v168, v167, %[s10]\n\t" \
  "v_mbcnt_hi_u32_b32 v165, vcc_hi, v165\n\t" \
  "v_lshl_or_b32 v166, v167, 2, v166\n\t" \
  "v_add_u32 v165, %[nexc], v165\n\t" \
  "v_add_u32 v168, 0x8000, %[vx]\n\t" \
  "v_min_u32 v165, 63, v165\n\t" \
  "v_and_or_b32 v166, v166, %[k0f], v168\n\t" \
  "v_lshl_add_u32 v165, v165, 3, %[rb]\n\t" \
  "s_bcnt1_i32_b64 %[t], vcc\n\t" \
  "s_mov_b64 exec, vcc\n\t" \
  "ds_write2_b32 v165, v164, v166 offset1:1\n\t" \
  "s_add_u32 %[nexc], %[nexc], %[t]\n\t" \
  "3:\n" \
  "s_mov_b64 exec, -1\n\t" \
  "4:\n" \
  "s_mov_b64 exec, -1\n\t" \
  "ds_read_b128 v[136:139], %[t16]\n\t" \
  "ds_read_b128 v[140:143], %[t16] offset:1024\n\t" \
  "s_lshl_b32 %[t], %[nch], 4\n\t" \
  "s_min_u32 %[t2], %[nexc], 64\n\t" \
  "v_add_u32 v129, %[t], %[so]\n\t" \
  "s_sub_u32 %[t], 0x1000, %[t]\n\t" \
  "s_lshl_b32 %[t2], %[t2], 3\n\t" \
  "s_add_u32 %[t2], %[t2], %[rb]\n\t" \
  "s_sub_u32 %[t2], %[t2], %[rec0]\n\t" \
  "s_add_u32 %[t2], %[t2], 0x100\n\t" \
  "s_min_u32 %[t2], %[t2], %[t]\n\t" \
  "v_cmp_gt_i32 vcc, %[t2], %[l16]\n\t" \
  "s_sub_u32 %[t2], %[t2], 0x400\n\t" \
  "s_mov_b64 exec, vcc\n\t" \
  "v_cmp_gt_i32 vcc, %[t2], %[l16]\n\t" \
  "s_waitcnt lgkmcnt(0)\n\t" \
  "global_store_dwordx4 v129, v[136:139], %[wp] sc0 sc1\n\t" \
  "s_mov_b64 exec, vcc\n\t" \
  "global_store_dwordx4 v129, v[140:143], %[wp] offset:1024 sc0 sc1\n\t" \
  "s_mov_b64 exec, -1\n\t" \
  "s_nop 1\n\t" \
  ""
#define ASM_ENC2_2 \
  "s_lshl_b32 %[t], %[cnt], 2\n\t" \
  "v_mov_b32 v128, 0\n\t" \
  "v_add_u32 v129, %[t], %[c4]\n\t" \
  "s_mov_b32 %[nexc], 0\n\t" \
  "ds_write_b32 v129, v128\n\t" \
  "ds_write_b32 %[c4], %[m] offset:6400\n\t" \
  "v_ashrrev_i32 v128, 31, v64\n\t" \
  "v_bfi_b32 v132, v128, %[dum], %[p]\n\t" \
  "v_lshl_add_u32 %[p], v128, 2, %[p]\n\t" \
  "ds_write_b32 v132, v64 offset:0\n\t" \
  "v_ashrrev_i32 v129, 31, v68\n\t" \
  "v_bfi_b32 v133, v129, %[dum], %[p]\n\t" \
  "v_lshl_add_u32 %[p], v129, 2, %[p]\n\t" \
  "ds_write_b32 v133, v68 offset:4\n\t" \
  "v_ashrrev_i32 v130, 31, v72\n\t" \
  "v_bfi_b32 v134, v130, %[dum], %[p]\n\t" \
  "v_lshl_add_u32 %[p], v130, 2, %[p]\n\t" \
  "ds_write_b32 v134, v72 offset:8\n\t" \
  "v_ashrrev_i32 v131, 31, v76\n\t" \
  "v_bfi_b32 v135, v131, %[dum], %[p]\n\t" \
  "v_lshl_add_u32 %[p], v131, 2, %[p]\n\t" \
  "ds_write_b32 v135, v76 offset:12\n\t" \
  "v_ashrrev_i32 v128, 31, v65\n\t" \
  "v_bfi_b32 v132, v128, %[dum], %[p]\n\t" \
  "v_lshl_add_u32 %[p], v128, 2, %[p]\n\t" \
  "ds_write_b32 v132, v65 offset:16\n\t" \
  "v_ashrrev_i32 v129, 31, v69\n\t" \
  "v_bfi_b32 v133, v129, %[dum], %[p]\n\t" \
  "v_lshl_add_u32 %[p], v129, 2, %[p]\n\t" \
  "ds_write_b32 v133, v69 offset:20\n\t" \
  "v_ashrrev_i32 v130, 31, v73\n\t" \
  "v_bfi_b32 v134, v130, %[dum], %[p]\n\t" \
  "v_lshl_add_u32 %[p], v130, 2, %[p]\n\t" \
  "ds_write_b32 v134, v73 offset:24\n\t" \
  "v_ashrrev_i32 v131, 31, v77\n\t" \
  "v_bfi_b32 v135, v131, %[dum], %[p]\n\t" \
  "v_lshl_add_u32 %[p], v131, 2, %[p]\n\t" \
  "ds_write_b32 v135, v77 offset:28\n\t" \
  "v_ashrrev_i32 v128, 31, v66\n\t" \
  "v_bfi_b32 v132, v128, %[dum], %[p]\n\t" \
  "v_lshl_add_u32 %[p], v128, 2, %[p]\n\t" \
  "ds_write_b32 v132, v66 offset:32\n\t" \
  "v_ashrrev_i32 v129, 31, v70\n\t" \
  "v_bfi_b32 v133, v129, %[dum], %[p]\n\t" \
  "v_lshl_add_u32 %[p], v129, 2, %[p]\n\t" \
  "ds_write_b32 v133, v70 offset:36\n\t" \
  "v_ashrrev_i32 v130, 31, v74\n\t" \
  "v_bfi_b32 v134, v130, %[dum], %[p]\n\t" \
  "v_lshl_add_u32 %[p], v130, 2, %[p]\n\t" \
  "ds_write_b32 v134, v74 offset:40\n\t" \
  "v_ashrrev_i32 v131, 31, v78\n\t" \
  "v_bfi_b32 v135, v131, %[dum], %[p]\n\t" \
  "v_lshl_add_u32 %[p], v131, 2, %[p]\n\t" \
  "ds_write_b32 v135, v78 offset:44\n\t" \
  "v_ashrrev_i32 v128, 31, v67\n\t" \
  "v_bfi_b32 v132, v128, %[dum], %[p]\n\t" \
  "v_lshl_add_u32 %[p], v128, 2, %[p]\n\t" \
  "ds_write_b32 v132, v67 offset:48\n\t" \
  "v_ashrrev_i32 v129, 31, v71\n\t" \
  "v_bfi_b32 v133, v129, %[dum], %[p]\n\t" \
  "v_lshl_add_u32 %[p], v129, 2, %[p]\n\t" \
  "ds_write_b32 v133, v71 offset:52\n\t" \
  "v_ashrrev_i32 v130, 31, v75\n\t" \
  "v_bfi_b32 v134, v130, %[dum], %[p]\n\t" \
  "v_lshl_add_u32 %[p], v130, 2, %[p]\n\t" \
  "ds_write_b32 v134, v75 offset:56\n\t" \
  "v_ashrrev_i32 v131, 31, v79\n\t" \
  "v_bfi_b32 v135, v131, %[dum], %[p]\n\t" \
  "v_lshl_add_u32 %[p], v131, 2, %[p]\n\t" \
  "ds_write_b32 v135, v79 offset:60\n\t" \
  "v_ashrrev_i32 v128, 31, v80\n\t" \
  "v_bfi_b32 v132, v128, %[dum], %[p]\n\t" \
  "v_lshl_add_u32 %[p], v128, 2, %[p]\n\t" \
  "ds_write_b32 v132, v80 offset:64\n\t" \
  "v_ashrrev_i32 v129, 31, v84\n\t" \
  "v_bfi_b32 v133, v129, %[dum], %[p]\n\t" \
  "v_lshl_add_u32 %[p], v129, 2, %[p]\n\t" \
  "ds_write_b32 v133, v84 offset:68\n\t" \
  "v_ashrrev_i32 v130, 31, v88\n\t" \
  "v_bfi_b32 v134, v130, %[dum], %[p]\n\t" \
  "v_lshl_add_u32 %[p], v130, 2, %[p]\n\t" \
  "ds_write_b32 v134, v88 offset:72\n\t" \
  "v_ashrrev_i32 v131, 31, v92\n\t" \
  "v_bfi_b32 v135, v131, %[dum], %[p]\n\t" \
  "v_lshl_add_u32 %[p], v131, 2, %[p]\n\t" \
  "ds_write_b32 v135, v92 offset:76\n\t" \
  "v_ashrrev_i32 v128, 31, v81\n\t" \
  "v_bfi_b32 v132, v128, %[dum], %[p]\n\t" \
  "v_lshl_add_u32 %[p], v128, 2, %[p]\n\t" \
  "ds_write_b32 v132, v81 offset:80\n\t" \
  "v_ashrrev_i32 v129, 31, v85\n\t" \
  "v_bfi_b32 v133, v129, %[dum], %[p]\n\t" \
  "v_lshl_add_u32 %[p], v129, 2, %[p]\n\t" \
  "ds_write_b32 v133, v85 offset:84\n\t" \
  "v_ashrrev_i32 v130, 31, v89\n\t" \
  "v_bfi_b32 v134, v130, %[dum], %[p]\n\t" \
  "v_lshl_add_u32 %[p], v130, 2, %[p]\n\t" \
  "ds_write_b32 v134, v89 offset:88\n\t" \
  "v_ashrrev_i32 v131, 31, v93\n\t" \
  "v_bfi_b32 v135, v131, %[dum], %[p]\n\t" \
  "v_lshl_add_u32 %[p], v131, 2, %[p]\n\t" \
  "ds_write_b32 v135, v93 offset:92\n\t" \
  "v_ashrrev_i32 v128, 31, v82\n\t" \
  "v_bfi_b32 v132, v128, %[dum], %[p]\n\t" \
  "v_lshl_add_u32 %[p], v128, 2, %[p]\n\t" \
  "ds_write_b32 v132, v82 offset:96\n\t" \
  "v_ashrrev_i32 v129, 31, v86\n\t" \
  "v_bfi_b32 v133, v129, %[dum], %[p]\n\t" \
  "v_lshl_add_u32 %[p], v129, 2, %[p]\n\t" \
  "ds_write_b32 v133, v86 offset:100\n\t" \
  "v_ashrrev_i32 v130, 31, v90\n\t" \
  "v_bfi_b32 v134, v130, %[dum], %[p]\n\t" \
  "v_lshl_add_u32 %[p], v130, 2, %[p]\n\t" \
  "ds_write_b32 v134, v90 offset:104\n\t" \
  "v_ashrrev_i32 v131, 31, v94\n\t" \
  "v_bfi_b32 v135, v131, %[dum], %[p]\n\t" \
  "v_lshl_add_u32 %[p], v131, 2, %[p]\n\t" \
  "ds_write_b32 v135, v94 offset:108\n\t" \
  "v_ashrrev_i32 v128, 31, v83\n\t" \
  "v_bfi_b32 v132, v128, %[dum], %[p]\n\t" \
  "v_lshl_add_u32 %[p], v128, 2, %[p]\n\t" \
  "ds_write_b32 v132, v83 offset:112\n\t" \
  "v_ashrrev_i32 v129, 31, v87\n\t" \
  "v_bfi_b32 v133, v129, %[dum], %[p]\n\t" \
  "v_lshl_add_u32 %[p], v129, 2, %[p]\n\t" \
  "ds_write_b32 v133, v87 offset:116\n\t" \
  "v_ashrrev_i32 v130, 31, v91\n\t" \
  "v_bfi_b32 v134, v130, %[dum], %[p]\n\t" \
  "v_lshl_add_u32 %[p], v130, 2, %[p]\n\t" \
  "ds_write_b32 v134, v91 offset:120\n\t" \
  "v_ashrrev_i32 v131, 31, v95\n\t" \
  "v_bfi_b32 v135, v131, %[dum], %[p]\n\t" \
  "v_lshl_add_u32 %[p], v131, 2, %[p]\n\t" \
  "ds_write_b32 v135, v95 offset:124\n\t" \
  "ds_read_b128 v[136:139], %[c32] offset:0\n\t" \
  "ds_read_b128 v[140:143], %[c32] offset:16\n\t" \
  "ds_read_b128 v[144:147], %[c32] offset:2048\n\t" \
  "ds_read_b128 v[148:151], %[c32] offset:2064\n\t" \
  "ds_read_b128 v[152:155], %[c32] offset:4096\n\t" \
  "ds_read_b128 v[156:159], %[c32] offset:4112\n\t" \
  "s_lshl_b32 %[t], %[nch], 2\n\t" \
  "s_add_u32 %[rb], %[t], %[rec0]\n\t" \
  "s_mov_b32 %[t2], %[nch]\n\t" \
  "v_cmpx_gt_i32 vcc, %[t2], %[lane]\n\t" \
  "s_cbranch_execz 4f\n\t" \
  "s_waitcnt lgkmcnt(4)\n\t" \
  "v_perm_b32 v160, v137, v136, %[s10]\n\t" \
  "v_perm_b32 v161, v139, v138, %[s10]\n\t" \
  "v_perm_b32 v162, v141, v140, %[s10]\n\t" \
  "v_perm_b32 v163, v143, v142, %[s10]\n\t" \
  "v_perm_b32 v164, v138, v136, %[s62]\n\t" \
  "v_perm_b32 v166, v142, v140, %[s62]\n\t" \
  "v_perm_b32 v165, v139, v137, %[s62]\n\t" \
  "v_perm_b32 v167, v143, v141, %[s62]\n\t" \
  "global_store_dwordx4 %[so], v[160:163], %[wp] offset:0 sc0 sc1\n\t" \
  "v_perm_b32 v164, v166, v164, %[s10]\n\t" \
  "v_perm_b32 v165, v167, v165, %[s10]\n\t" \
  "v_or3_b32 v166, v136, v137, v138\n\t" \
  "v_or3_b32 v167, v139, v140, v141\n\t" \
  "v_lshlrev_b32 v168, 4, v165\n\t" \
  "v_or3_b32 v166, v142, v143, v166\n\t" \
  "v_bfi_b32 v168, %[k0f], v164, v168\n\t" \
  "v_or_b32 v166, v166, v167\n\t" \
  "ds_write_b32 %[c4], v168 offset:6656\n\t" \
  "v_cmp_lt_u32 vcc, %[lim], v166\n\t" \
  "s_cbranch_vccz 1f\n\t" \
  "v_lshrrev_b32 v164, 4, v164\n\t" \
  "v_perm_b32 v166, v138, v136, %[s73]\n\t" \
  "v_perm_b32 v167, v142, v140, %[s73]\n\t" \
  "v_bfi_b32 v164, %[k0f], v164, v165\n\t" \
  "v_perm_b32 v166, v167, v166, %[s10]\n\t" \
  "v_perm_b32 v167, v139, v137, %[s73]\n\t" \
  "v_perm_b32 v168, v143, v141, %[s73]\n\t" \
  "v_mbcnt_lo_u32_b32 v165, vcc_lo, 0\n\t" \
  "v_perm_b32 v167, v168, v167, %[s10]\n\t" \
  "v_mbcnt_hi_u32_b32 v165, vcc_hi, v165\n\t" \
  "v_lshl_or_b32 v166, v167, 2, v166\n\t" \
  "v_add_u32 v165, %[nexc], v165\n\t" \
  "v_add_u32 v168, 0x0, %[vx]\n\t" \
  "v_min_u32 v165, 63, v165\n\t" \
  "v_and_or_b32 v166, v166, %[k0f], v168\n\t" \
  "v_lshl_add_u32 v165, v165, 3, %[rb]\n\t" \
  "s_bcnt1_i32_b64 %[t], vcc\n\t" \
  "s_mov_b64 exec, vcc\n\t" \
  "ds_write2_b32 v165, v164, v166 offset1:1\n\t" \
  "s_add_u32 %[nexc], %[nexc], %[t]\n\t" \
  "1:\n" \
  "s_mov_b64 exec, -1\n\t" \
  "s_sub_u32 %[t2], %[t2], 64\n\t" \
  "v_cmpx_gt_i32 vcc, %[t2], %[lane]\n\t" \
  "s_cbranch_execz 4f\n\t" \
  "s_waitcnt lgkmcnt(3)\n\t" \
  "v_perm_b32 v160, v145, v144, %[s10]\n\t" \
  "v_perm_b32 v161, v147, v146, %[s10]\n\t" \
  "v_perm_b32 v162, v149, v148, %[s10]\n\t" \
  "v_perm_b32 v163, v151, v150, %[s10]\n\t" \
  "v_perm_b32 v164, v146, v144, %[s62]\n\t" \
  "v_perm_b32 v166, v150, v148, %[s62]\n\t" \
  "v_perm_b32 v165, v147, v145, %[s62]\n\t" \
  "v_perm_b32 v167, v151, v149, %[s62]\n\t" \
  "global_store_dwordx4 %[so], v[160:163], %[wp] offset:1024 sc0 sc1\n\t" \
  "v_perm_b32 v164, v166, v164, %[s10]\n\t" \
  "v_perm_b32 v165, v167, v165, %[s10]\n\t" \
  "v_or3_b32 v166, v144, v145, v146\n\t" \
  "v_or3_b32 v167, v147, v148, v149\n\t" \
  "v_lshlrev_b32 v168, 4, v165\n\t" \
  "v_or3_b32 v166, v150, v151, v166\n\t" \
  "v_bfi_b32 v168, %[k0f], v164, v168\n\t" \
  "v_or_b32 v166, v166, v167\n\t" \
  "ds_write_b32 %[c4], v168 offset:6912\n\t" \
  "v_cmp_lt_u32 vcc, %[lim], v166\n\t" \
  "s_cbranch_vccz 2f\n\t" \
  "v_lshrrev_b32 v164, 4, v164\n\t" \
  "v_perm_b32 v166, v146, v144, %[s73]\n\t" \
  "v_perm_b32 v167, v150, v148, %[s73]\n\t" \
  "v_bfi_b32 v164, %[k0f], v164, v165\n\t" \
  "v_perm_b32 v166, v167, v166, %[s10]\n\t" \
  "v_perm_b32 v167, v147, v145, %[s73]\n\t" \
  "v_perm_b32 v168, v151, v149, %[s73]\n\t" \
  "v_mbcnt_lo_u32_b32 v165, vcc_lo, 0\n\t" \
  "v_perm_b32 v167, v168, v167, %[s10]\n\t" \
  "v_mbcnt_hi_u32_b32 v165, vcc_hi, v165\n\t" \
  "v_lshl_or_b32 v166, v167, 2, v166\n\t" \
  "v_add_u32 v165, %[nexc], v165\n\t" \
  "v_add_u32 v168, 0x4000, %[vx]\n\t" \
  "v_min_u32 v165, 63, v165\n\t" \
  "v_and_or_b32 v166, v166, %[k0f], v168\n\t" \
  "v_lshl_add_u32 v165, v165, 3, %[rb]\n\t" \
  "s_bcnt1_i32_b64 %[t], vcc\n\t" \
  "s_mov_b64 exec, vcc\n\t" \
  "ds_write2_b32 v165, v164, v166 offset1:1\n\t" \
  "s_add_u32 %[nexc], %[nexc], %[t]\n\t" \
  "2:\n" \
  "s_mov_b64 exec, -1\n\t" \
  "s_sub_u32 %[t2], %[t2], 64\n\t" \
  "v_cmpx_gt_i32 vcc, %[t2], %[lane]\n\t" \
  "s_cbranch_execz 4f\n\t" \
  "s_waitcnt lgkmcnt(2)\n\t" \
  "v_perm_b32 v160, v153, v152, %[s10]\n\t" \
  "v_perm_b32 v161, v155, v154, %[s10]\n\t" \
  "v_perm_b32 v162, v157, v156, %[s10]\n\t" \
  "v_perm_b32 v163, v159, v158, %[s10]\n\t" \
  "v_perm_b32 v164, v154, v152, %[s62]\n\t" \
  "v_perm_b32 v166, v158, v156, %[s62]\n\t" \
  "v_perm_b32 v165, v155, v153, %[s62]\n\t" \
  "v_perm_b32 v167, v159, v157, %[s62]\n\t" \
  "global_store_dwordx4 %[so], v[160:163], %[wp] offset:2048 sc0 sc1\n\t" \
  "v_perm_b32 v164, v166, v164, %[s10]\n\t" \
  "v_perm_b32 v165, v167, v165, %[s10]\n\t" \
  "v_or3_b32 v166, v152, v153, v154\n\t" \
  "v_or3_b32 v167, v155, v156, v157\n\t" \
  "v_lshlrev_b32 v168, 4, v165\n\t" \
  "v_or3_b32 v166, v158, v159, v166\n\t" \
  "v_bfi_b32 v168, %[k0f], v164, v168\n\t" \
  "v_or_b32 v166, v166, v167\n\t" \
  "ds_write_b32 %[c4], v168 offset:7168\n\t" \
  "v_cmp_lt_u32 vcc, %[lim], v166\n\t" \
  "s_cbranch_vccz 3f\n\t" \
  "v_lshrrev_b32 v164, 4, v164\n\t" \
  "v_perm_b32 v166, v154, v152, %[s73]\n\t" \
  "v_perm_b32 v167, v158, v156, %[s73]\n\t" \
  "v_bfi_b32 v164, %[k0f], v164, v165\n\t" \
  "v_perm_b32 v166, v167, v166, %[s10]\n\t" \
  "v_perm_b32 v167, v155, v153, %[s73]\n\t" \
  "v_perm_b32 v168, v159, v157, %[s73]\n\t" \
  "v_mbcnt_lo_u32_b32 v165, vcc_lo, 0\n\t" \
  "v_perm_b32 v167, v168, v167, %[s10]\n\t" \
  "v_mbcnt_hi_u32_b32 v165, vcc_hi, v165\n\t" \
  "v_lshl_or_b32 v166, v167, 2, v166\n\t" \
  "v_add_u32 v165, %[nexc], v165\n\t" \
  "v_add_u32 v168, 0x8000, %[vx]\n\t" \
  "v_min_u32 v165, 63, v165\n\t" \
  "v_and_or_b32 v166, v166, %[k0f], v168\n\t" \
  "v_lshl_add_u32 v165, v165, 3, %[rb]\n\t" \
  "s_bcnt1_i32_b64 %[t], vcc\n\t" \
  "s_mov_b64 exec, vcc\n\t" \
  "ds_write2_b32 v165, v164, v166 offset1:1\n\t" \
  "s_add_u32 %[nexc], %[nexc], %[t]\n\t" \
  "3:\n" \
  "s_mov_b64 exec, -1\n\t" \
  "4:\n" \
  "s_mov_b64 exec, -1\n\t" \
  "ds_read_b128 v[136:139], %[t16]\n\t" \
  "ds_read_b128 v[140:143], %[t16] offset:1024\n\t" \
  "s_lshl_b32 %[t], %[nch], 4\n\t" \
  "s_min_u32 %[t2], %[nexc], 64\n\t" \
  "v_add_u32 v129, %[t], %[so]\n\t" \
  "s_sub_u32 %[t], 0x1000, %[t]\n\t" \
  "s_lshl_b32 %[t2], %[t2], 3\n\t" \
  "s_add_u32 %[t2], %[t2], %[rb]\n\t" \
  "s_sub_u32 %[t2], %[t2], %[rec0]\n\t" \
  "s_add_u32 %[t2], %[t2], 0x100\n\t" \
  "s_min_u32 %[t2], %[t2], %[t]\n\t" \
  "v_cmp_gt_i32 vcc, %[t2], %[l16]\n\t" \
  "s_sub_u32 %[t2], %[t2], 0x400\n\t" \
  "s_mov_b64 exec, vcc\n\t" \
  "v_cmp_gt_i32 vcc, %[t2], %[l16]\n\t" \
  "s_waitcnt lgkmcnt(0)\n\t" \
  "global_store_dwordx4 v129, v[136:139], %[wp] sc0 sc1\n\t" \
  "s_mov_b64 exec, vcc\n\t" \
  "global_store_dwordx4 v129, v[140:143], %[wp] offset:1024 sc0 sc1\n\t" \
  "s_mov_b64 exec, -1\n\t" \
  "s_nop 1\n\t" \
  ""
#define ASM_ENC2_3 \
  "s_lshl_b32 %[t], %[cnt], 2\n\t" \
  "v_mov_b32 v128, 0\n\t" \
  "v_add_u32 v129, %[t], %[c4]\n\t" \
  "s_mov_b32 %[nexc], 0\n\t" \
  "ds_write_b32 v129, v128\n\t" \
  "ds_write_b32 %[c4], %[m] offset:6400\n\t" \
  "v_ashrrev_i32 v128, 31, v96\n\t" \
  "v_bfi_b32 v132, v128, %[dum], %[p]\n\t" \
  "v_lshl_add_u32 %[p], v128, 2, %[p]\n\t" \
  "ds_write_b32 v132, v96 offset:0\n\t" \
  "v_ashrrev_i32 v129, 31, v100\n\t" \
  "v_bfi_b32 v133, v129, %[dum], %[p]\n\t" \
  "v_lshl_add_u32 %[p], v129, 2, %[p]\n\t" \
  "ds_write_b32 v133, v100 offset:4\n\t" \
  "v_ashrrev_i32 v130, 31, v104\n\t" \
  "v_bfi_b32 v134, v130, %[dum], %[p]\n\t" \
  "v_lshl_add_u32 %[p], v130, 2, %[p]\n\t" \
  "ds_write_b32 v134, v104 offset:8\n\t" \
  "v_ashrrev_i32 v131, 31, v108\n\t" \
  "v_bfi_b32 v135, v131, %[dum], %[p]\n\t" \
  "v_lshl_add_u32 %[p], v131, 2, %[p]\n\t" \
  "ds_write_b32 v135, v108 offset:12\n\t" \
  "v_ashrrev_i32 v128, 31, v97\n\t" \
  "v_bfi_b32 v132, v128, %[dum], %[p]\n\t" \
  "v_lshl_add_u32 %[p], v128, 2, %[p]\n\t" \
  "ds_write_b32 v132, v97 offset:16\n\t" \
  "v_ashrrev_i32 v129, 31, v101\n\t" \
  "v_bfi_b32 v133, v129, %[dum], %[p]\n\t" \
  "v_lshl_add_u32 %[p], v129, 2, %[p]\n\t" \
  "ds_write_b32 v133, v101 offset:20\n\t" \
  "v_ashrrev_i32 v130, 31, v105\n\t" \
  "v_bfi_b32 v134, v130, %[dum], %[p]\n\t" \
  "v_lshl_add_u32 %[p], v130, 2, %[p]\n\t" \
  "ds_write_b32 v134, v105 offset:24\n\t" \
  "v_ashrrev_i32 v131, 31, v109\n\t" \
  "v_bfi_b32 v135, v131, %[dum], %[p]\n\t" \
  "v_lshl_add_u32 %[p], v131, 2, %[p]\n\t" \
  "ds_write_b32 v135, v109 offset:28\n\t" \
  "v_ashrrev_i32 v128, 31, v98\n\t" \
  "v_bfi_b32 v132, v128, %[dum], %[p]\n\t" \
  "v_lshl_add_u32 %[p], v128, 2, %[p]\n\t" \
  "ds_write_b32 v132, v98 offset:32\n\t" \
  "v_ashrrev_i32 v129, 31, v102\n\t" \
  "v_bfi_b32 v133, v129, %[dum], %[p]\n\t" \
  "v_lshl_add_u32 %[p], v129, 2, %[p]\n\t" \
  "ds_write_b32 v133, v102 offset:36\n\t" \
  "v_ashrrev_i32 v130, 31, v106\n\t" \
  "v_bfi_b32 v134, v130, %[dum], %[p]\n\t" \
  "v_lshl_add_u32 %[p], v130, 2, %[p]\n\t" \
  "ds_write_b32 v134, v106 offset:40\n\t" \
  "v_ashrrev_i32 v131, 31, v110\n\t" \
  "v_bfi_b32 v135, v131, %[dum], %[p]\n\t" \
  "v_lshl_add_u32 %[p], v131, 2, %[p]\n\t" \
  "ds_write_b32 v135, v110 offset:44\n\t" \
  "v_ashrrev_i32 v128, 31, v99\n\t" \
  "v_bfi_b32 v132, v128, %[dum], %[p]\n\t" \
  "v_lshl_add_u32 %[p], v128, 2, %[p]\n\t" \
  "ds_write_b32 v132, v99 offset:48\n\t" \
  "v_ashrrev_i32 v129, 31, v103\n\t" \
  "v_bfi_b32 v133, v129, %[dum], %[p]\n\t" \
  "v_lshl_add_u32 %[p], v129, 2, %[p]\n\t" \
  "ds_write_b32 v133, v103 offset:52\n\t" \
  "v_ashrrev_i32 v130, 31, v107\n\t" \
  "v_bfi_b32 v134, v130, %[dum], %[p]\n\t" \
  "v_lshl_add_u32 %[p], v130, 2, %[p]\n\t" \
  "ds_write_b32 v134, v107 offset:56\n\t" \
  "v_ashrrev_i32 v131, 31, v111\n\t" \
  "v_bfi_b32 v135, v131, %[dum], %[p]\n\t" \
  "v_lshl_add_u32 %[p], v131, 2, %[p]\n\t" \
  "ds_write_b32 v135, v111 offset:60\n\t" \
  "v_ashrrev_i32 v128, 31, v112\n\t" \
  "v_bfi_b32 v132, v128, %[dum], %[p]\n\t" \
  "v_lshl_add_u32 %[p], v128, 2, %[p]\n\t" \
  "ds_write_b32 v132, v112 offset:64\n\t" \
  "v_ashrrev_i32 v129, 31, v116\n\t" \
  "v_bfi_b32 v133, v129, %[dum], %[p]\n\t" \
  "v_lshl_add_u32 %[p], v129, 2, %[p]\n\t" \
  "ds_write_b32 v133, v116 offset:68\n\t" \
  "v_ashrrev_i32 v130, 31, v120\n\t" \
  "v_bfi_b32 v134, v130, %[dum], %[p]\n\t" \
  "v_lshl_add_u32 %[p], v130, 2, %[p]\n\t" \
  "ds_write_b32 v134, v120 offset:72\n\t" \
  "v_ashrrev_i32 v131, 31, v124\n\t" \
  "v_bfi_b32 v135, v131, %[dum], %[p]\n\t" \
  "v_lshl_add_u32 %[p], v131, 2, %[p]\n\t" \
  "ds_write_b32 v135, v124 offset:76\n\t" \
  "v_ashrrev_i32 v128, 31, v113\n\t" \
  "v_bfi_b32 v132, v128, %[dum], %[p]\n\t" \
  "v_lshl_add_u32 %[p], v128, 2, %[p]\n\t" \
  "ds_write_b32 v132, v113 offset:80\n\t" \
  "v_ashrrev_i32 v129, 31, v117\n\t" \
  "v_bfi_b32 v133, v129, %[dum], %[p]\n\t" \
  "v_lshl_add_u32 %[p], v129, 2, %[p]\n\t" \
  "ds_write_b32 v133, v117 offset:84\n\t" \
  "v_ashrrev_i32 v130, 31, v121\n\t" \
  "v_bfi_b32 v134, v130, %[dum], %[p]\n\t" \
  "v_lshl_add_u32 %[p], v130, 2, %[p]\n\t" \
  "ds_write_b32 v134, v121 offset:88\n\t" \
  "v_ashrrev_i32 v131, 31, v125\n\t" \
  "v_bfi_b32 v135, v131, %[dum], %[p]\n\t" \
  "v_lshl_add_u32 %[p], v131, 2, %[p]\n\t" \
  "ds_write_b32 v135, v125 offset:92\n\t" \
  "v_ashrrev_i32 v128, 31, v114\n\t" \
  "v_bfi_b32 v132, v128, %[dum], %[p]\n\t" \
  "v_lshl_add_u32 %[p], v128, 2, %[p]\n\t" \
  "ds_write_b32 v132, v114 offset:96\n\t" \
  "v_ashrrev_i32 v129, 31, v118\n\t" \
  "v_bfi_b32 v133, v129, %[dum], %[p]\n\t" \
  "v_lshl_add_u32 %[p], v129, 2, %[p]\n\t" \
  "ds_write_b32 v133, v118 offset:100\n\t" \
  "v_ashrrev_i32 v130, 31, v122\n\t" \
  "v_bfi_b32 v134, v130, %[dum], %[p]\n\t" \
  "v_lshl_add_u32 %[p], v130, 2, %[p]\n\t" \
  "ds_write_b32 v134, v122 offset:104\n\t" \
  "v_ashrrev_i32 v131, 31, v126\n\t" \
  "v_bfi_b32 v135, v131, %[dum], %[p]\n\t" \
  "v_lshl_add_u32 %[p], v131, 2, %[p]\n\t" \
  "ds_write_b32 v135, v126 offset:108\n\t" \
  "v_ashrrev_i32 v128, 31, v115\n\t" \
  "v_bfi_b32 v132, v128, %[dum], %[p]\n\t" \
  "v_lshl_add_u32 %[p], v128, 2, %[p]\n\t" \
  "ds_write_b32 v132, v115 offset:112\n\t" \
  "v_ashrrev_i32 v129, 31, v119\n\t" \
  "v_bfi_b32 v133, v129, %[dum], %[p]\n\t" \
  "v_lshl_add_u32 %[p], v129, 2, %[p]\n\t" \
  "ds_write_b32 v133, v119 offset:116\n\t" \
  "v_ashrrev_i32 v130, 31, v123\n\t" \
  "v_bfi_b32 v134, v130, %[dum], %[p]\n\t" \
  "v_lshl_add_u32 %[p], v130, 2, %[p]\n\t" \
  "ds_write_b32 v134, v123 offset:120\n\t" \
  "v_ashrrev_i32 v131, 31, v127\n\t" \
  "v_bfi_b32 v135, v131, %[dum], %[p]\n\t" \
  "v_lshl_add_u32 %[p], v131, 2, %[p]\n\t" \
  "ds_write_b32 v135, v127 offset:124\n\t" \
  "ds_read_b128 v[136:139], %[c32] offset:0\n\t" \
  "ds_read_b128 v[140:143], %[c32] offset:16\n\t" \
  "ds_read_b128 v[144:147], %[c32] offset:2048\n\t" \
  "ds_read_b128 v[148:151], %[c32] offset:2064\n\t" \
  "ds_read_b128 v[152:155], %[c32] offset:4096\n\t" \
  "ds_read_b128 v[156:159], %[c32] offset:4112\n\t" \
  "s_lshl_b32 %[t], %[nch], 2\n\t" \
  "s_add_u32 %[rb], %[t], %[rec0]\n\t" \
  "s_mov_b32 %[t2], %[nch]\n\t" \
  "v_cmpx_gt_i32 vcc, %[t2], %[lane]\n\t" \
  "s_cbranch_execz 4f\n\t" \
  "s_waitcnt lgkmcnt(4)\n\t" \
  "v_perm_b32 v160, v137, v136, %[s10]\n\t" \
  "v_perm_b32 v161, v139, v138, %[s10]\n\t" \
  "v_perm_b32 v162, v141, v140, %[s10]\n\t" \
  "v_perm_b32 v163, v143, v142, %[s10]\n\t" \
  "v_perm_b32 v164, v138, v136, %[s62]\n\t" \
  "v_perm_b32 v166, v142, v140, %[s62]\n\t" \
  "v_perm_b32 v165, v139, v137, %[s62]\n\t" \
  "v_perm_b32 v167, v143, v141, %[s62]\n\t" \
  "global_store_dwordx4 %[so], v[160:163], %[wp] offset:0 sc0 sc1\n\t" \
  "v_perm_b32 v164, v166, v164, %[s10]\n\t" \
  "v_perm_b32 v165, v167, v165, %[s10]\n\t" \
  "v_or3_b32 v166, v136, v137, v138\n\t" \
  "v_or3_b32 v167, v139, v140, v141\n\t" \
  "v_lshlrev_b32 v168, 4, v165\n\t" \
  "v_or3_b32 v166, v142, v143, v166\n\t" \
  "v_bfi_b32 v168, %[k0f], v164, v168\n\t" \
  "v_or_b32 v166, v166, v167\n\t" \
  "ds_write_b32 %[c4], v168 offset:6656\n\t" \
  "v_cmp_lt_u32 vcc, %[lim], v166\n\t" \
  "s_cbranch_vccz 1f\n\t" \
  "v_lshrrev_b32 v164, 4, v164\n\t" \
  "v_perm_b32 v166, v138, v136, %[s73]\n\t" \
  "v_perm_b32 v167, v142, v140, %[s73]\n\t" \
  "v_bfi_b32 v164, %[k0f], v164, v165\n\t" \
  "v_perm_b32 v166, v167, v166, %[s10]\n\t" \
  "v_perm_b32 v167, v139, v137, %[s73]\n\t" \
  "v_perm_b32 v168, v143, v141, %[s73]\n\t" \
  "v_mbcnt_lo_u32_b32 v165, vcc_lo, 0\n\t" \
  "v_perm_b32 v167, v168, v167, %[s10]\n\t" \
  "v_mbcnt_hi_u32_b32 v165, vcc_hi, v165\n\t" \
  "v_lshl_or_b32 v166, v167, 2, v166\n\t" \
  "v_add_u32 v165, %[nexc], v165\n\t" \
  "v_add_u32 v168, 0x0, %[vx]\n\t" \
  "v_min_u32 v165, 63, v165\n\t" \
  "v_and_or_b32 v166, v166, %[k0f], v168\n\t" \
  "v_lshl_add_u32 v165, v165, 3, %[rb]\n\t" \
  "s_bcnt1_i32_b64 %[t], vcc\n\t" \
  "s_mov_b64 exec, vcc\n\t" \
  "ds_write2_b32 v165, v164, v166 offset1:1\n\t" \
  "s_add_u32 %[nexc], %[nexc], %[t]\n\t" \
  "1:\n" \
  "s_mov_b64 exec, -1\n\t" \
  "s_sub_u32 %[t2], %[t2], 64\n\t" \
  "v_cmpx_gt_i32 vcc, %[t2], %[lane]\n\t" \
  "s_cbranch_execz 4f\n\t" \
  "s_waitcnt lgkmcnt(3)\n\t" \
  "v_perm_b32 v160, v145, v144, %[s10]\n\t" \
  "v_perm_b32 v161, v147, v146, %[s10]\n\t" \
  "v_perm_b32 v162, v149, v148, %[s10]\n\t" \
  "v_perm_b32 v163, v151, v150, %[s10]\n\t" \
  "v_perm_b32 v164, v146, v144, %[s62]\n\t" \
  "v_perm_b32 v166, v150, v148, %[s62]\n\t" \
  "v_perm_b32 v165, v147, v145, %[s62]\n\t" \
  "v_perm_b32 v167, v151, v149, %[s62]\n\t" \
  "global_store_dwordx4 %[so], v[160:163], %[wp] offset:1024 sc0 sc1\n\t" \
  "v_perm_b32 v164, v166, v164, %[s10]\n\t" \
  "v_perm_b32 v165, v167, v165, %[s10]\n\t" \
  "v_or3_b32 v166, v144, v145, v146\n\t" \
  "v_or3_b32 v167, v147, v148, v149\n\t" \
  "v_lshlrev_b32 v168, 4, v165\n\t" \
  "v_or3_b32 v166, v150, v151, v166\n\t" \
  "v_bfi_b32 v168, %[k0f], v164, v168\n\t" \
  "v_or_b32 v166, v166, v167\n\t" \
  "ds_write_b32 %[c4], v168 offset:6912\n\t" \
  "v_cmp_lt_u32 vcc, %[lim], v166\n\t" \
  "s_cbranch_vccz 2f\n\t" \
  "v_lshrrev_b32 v164, 4, v164\n\t" \
  "v_perm_b32 v166, v146, v144, %[s73]\n\t" \
  "v_perm_b32 v167, v150, v148, %[s73]\n\t" \
  "v_bfi_b32 v164, %[k0f], v164, v165\n\t" \
  "v_perm_b32 v166, v167, v166, %[s10]\n\t" \
  "v_perm_b32 v167, v147, v145, %[s73]\n\t" \
  "v_perm_b32 v168, v151, v149, %[s73]\n\t" \
  "v_mbcnt_lo_u32_b32 v165, vcc_lo, 0\n\t" \
  "v_perm_b32 v167, v168, v167, %[s10]\n\t" \
  "v_mbcnt_hi_u32_b32 v165, vcc_hi, v165\n\t" \
  "v_lshl_or_b32 v166, v167, 2, v166\n\t" \
  "v_add_u32 v165, %[nexc], v165\n\t" \
  "v_add_u32 v168, 0x4000, %[vx]\n\t" \
  "v_min_u32 v165, 63, v165\n\t" \
  "v_and_or_b32 v166, v166, %[k0f], v168\n\t" \
  "v_lshl_add_u32 v165, v165, 3, %[rb]\n\t" \
  "s_bcnt1_i32_b64 %[t], vcc\n\t" \
  "s_mov_b64 exec, vcc\n\t" \
  "ds_write2_b32 v165, v164, v166 offset1:1\n\t" \
  "s_add_u32 %[nexc], %[nexc], %[t]\n\t" \
  "2:\n" \
  "s_mov_b64 exec, -1\n\t" \
  "s_sub_u32 %[t2], %[t2], 64\n\t" \
  "v_cmpx_gt_i32 vcc, %[t2], %[lane]\n\t" \
  "s_cbranch_execz 4f\n\t" \
  "s_waitcnt lgkmcnt(2)\n\t" \
  "v_perm_b32 v160, v153, v152, %[s10]\n\t" \
  "v_perm_b32 v161, v155, v154, %[s10]\n\t" \
  "v_perm_b32 v162, v157, v156, %[s10]\n\t" \
  "v_perm_b32 v163, v159, v158, %[s10]\n\t" \
  "v_perm_b32 v164, v154, v152, %[s62]\n\t" \
  "v_perm_b32 v166, v158, v156, %[s62]\n\t" \
  "v_perm_b32 v165, v155, v153, %[s62]\n\t" \
  "v_perm_b32 v167, v159, v157, %[s62]\n\t" \
  "global_store_dwordx4 %[so], v[160:163], %[wp] offset:2048 sc0 sc1\n\t" \
  "v_perm_b32 v164, v166, v164, %[s10]\n\t" \
  "v_perm_b32 v165, v167, v165, %[s10]\n\t" \
  "v_or3_b32 v166, v152, v153, v154\n\t" \
  "v_or3_b32 v167, v155, v156, v157\n\t" \
  "v_lshlrev_b32 v168, 4, v165\n\t" \
  "v_or3_b32 v166, v158, v159, v166\n\t" \
  "v_bfi_b32 v168, %[k0f], v164, v168\n\t" \
  "v_or_b32 v166, v166, v167\n\t" \
  "ds_write_b32 %[c4], v168 offset:7168\n\t" \
  "v_cmp_lt_u32 vcc, %[lim], v166\n\t" \
  "s_cbranch_vccz 3f\n\t" \
  "v_lshrrev_b32 v164, 4, v164\n\t" \
  "v_perm_b32 v166, v154, v152, %[s73]\n\t" \
  "v_perm_b32 v167, v158, v156, %[s73]\n\t" \
  "v_bfi_b32 v164, %[k0f], v164, v165\n\t" \
  "v_perm_b32 v166, v167, v166, %[s10]\n\t" \
  "v_perm_b32 v167, v155, v153, %[s73]\n\t" \
  "v_perm_b32 v168, v159, v157, %[s73]\n\t" \
  "v_mbcnt_lo_u32_b32 v165, vcc_lo, 0\n\t" \
  "v_perm_b32 v167, v168, v167, %[s10]\n\t" \
  "v_mbcnt_hi_u32_b32 v165, vcc_hi, v165\n\t" \
  "v_lshl_or_b32 v166, v167, 2, v166\n\t" \
  "v_add_u32 v165, %[nexc], v165\n\t" \
  "v_add_u32 v168, 0x8000, %[vx]\n\t" \
  "v_min_u32 v165, 63, v165\n\t" \
  "v_and_or_b32 v166, v166, %[k0f], v168\n\t" \
  "v_lshl_add_u32 v165, v165, 3, %[rb]\n\t" \
  "s_bcnt1_i32_b64 %[t], vcc\n\t" \
  "s_mov_b64 exec, vcc\n\t" \
  "ds_write2_b32 v165, v164, v166 offset1:1\n\t" \
  "s_add_u32 %[nexc], %[nexc], %[t]\n\t" \
  "3:\n" \
  "s_mov_b64 exec, -1\n\t" \
  "4:\n" \
  "s_mov_b64 exec, -1\n\t" \
  "ds_read_b128 v[136:139], %[t16]\n\t" \
  "ds_read_b128 v[140:143], %[t16] offset:1024\n\t" \
  "s_lshl_b32 %[t], %[nch], 4\n\t" \
  "s_min_u32 %[t2], %[nexc], 64\n\t" \
  "v_add_u32 v129, %[t], %[so]\n\t" \
  "s_sub_u32 %[t], 0x1000, %[t]\n\t" \
  "s_lshl_b32 %[t2], %[t2], 3\n\t" \
  "s_add_u32 %[t2], %[t2], %[rb]\n\t" \
  "s_sub_u32 %[t2], %[t2], %[rec0]\n\t" \
  "s_add_u32 %[t2], %[t2], 0x100\n\t" \
  "s_min_u32 %[t2], %[t2], %[t]\n\t" \
  "v_cmp_gt_i32 vcc, %[t2], %[l16]\n\t" \
  "s_sub_u32 %[t2], %[t2], 0x400\n\t" \
  "s_mov_b64 exec, vcc\n\t" \
  "v_cmp_gt_i32 vcc, %[t2], %[l16]\n\t" \
  "s_waitcnt lgkmcnt(0)\n\t" \
  "global_store_dwordx4 v129, v[136:139], %[wp] sc0 sc1\n\t" \
  "s_mov_b64 exec, vcc\n\t" \
  "global_store_dwordx4 v129, v[140:143], %[wp] offset:1024 sc0 sc1\n\t" \
  "s_mov_b64 exec, -1\n\t" \
  "s_nop 1\n\t" \
  ""
#define ASM_DEC \
  "s_lshl_b32 %[t], %[nch], 4\n\t" \
  "s_sub_u32 %[n3], %[nch], 0x80\n\t" \
  "s_add_u32 %[tb], %[bl], %[t]\n\t" \
  "s_max_i32 %[n3], %[n3], 0\n\t" \
  "s_lshl_b32 %[t], %[nch], 2\n\t" \
  "s_lshl_b32 %[t2], %[n3], 3\n\t" \
  "s_add_u32 %[t], %[t], %[tb]\n\t" \
  "s_sub_u32 %[t], %[t], %[t2]\n\t" \
  "v_lshl_add_u32 v198, %[l4], 1, %[t]\n\t" \
  "v_add_u32 v196, %[tb], %[l4]\n\t" \
  "v_lshl_add_u32 v197, %[l4], 2, %[bl]\n\t" \
  "ds_read2_b32 v[176:177], v198 offset0:64 offset1:65\n\t" \
  "ds_read_b32 v172, v196\n\t" \
  "ds_read_b32 v173, v196 offset:256\n\t" \
  "ds_read_b128 v[160:163], v197\n\t" \
  "ds_read_b32 v174, v196 offset:512\n\t" \
  "ds_read_b128 v[164:167], v197 offset:1024\n\t" \
  "s_add_u32 %[t], %[n3], 0x80\n\t" \
  "v_cmp_le_u32 vcc, %[t], %[l128]\n\t" \
  "s_waitcnt lgkmcnt(5)\n\t" \
  "v_bfe_u32 v206, v177, 4, 4\n\t" \
  "v_bfe_u32 v207, v177, 12, 4\n\t" \
  "v_cndmask_b32 v176, 0, v176, vcc\n\t" \
  "v_lshl_or_b32 v206, v207, 4, v206\n\t" \
  "v_cndmask_b32 v177, 0, v177, vcc\n\t" \
  "v_cndmask_b32 v199, %[l128], v206, vcc\n\t" \
  "v_min_u32 v199, 0xbf, v199\n\t" \
  "v_lshl_add_u32 v206, v199, 4, %[bl]\n\t" \
  "v_lshl_add_u32 v207, v199, 2, %[tb]\n\t" \
  "v_lshl_add_u32 v200, v199, 5, %[ib]\n\t" \
  "ds_read_b128 v[168:171], v206\n\t" \
  "ds_read_b32 v175, v207 offset:256\n\t" \
  "s_waitcnt lgkmcnt(4)\n\t" \
  "v_bcnt_u32_b32 v201, v172, 0\n\t" \
  "v_and_b32 v204, %[k0f], v173\n\t" \
  "v_lshrrev_b32 v205, 4, v173\n\t" \
  "v_add_u32_dpp v202, v201, v201 row_shr:1 row_mask:0xf bank_mask:0xf bound_ctrl:0\n\t" \
  "v_and_b32 v205, %[k0f], v205\n\t" \
  "v_perm_b32 v180, v204, v160, %[e0]\n\t" \
  "v_perm_b32 v181, v205, v160, %[o0]\n\t" \
  "v_add_u32_dpp v202, v202, v202 row_shr:2 row_mask:0xf bank_mask:0xf bound_ctrl:0\n\t" \
  "v_perm_b32 v182, v204, v161, %[e1]\n\t" \
  "v_perm_b32 v183, v205, v161, %[o1]\n\t" \
  "v_perm_b32 v184, v204, v162, %[e2]\n\t" \
  "v_add_u32_dpp v202, v202, v202 row_shr:4 row_mask:0xf bank_mask:0xf bound_ctrl:0\n\t" \
  "v_perm_b32 v185, v205, v162, %[o2]\n\t" \
  "v_perm_b32 v186, v204, v163, %[e3]\n\t" \
  "v_perm_b32 v187, v205, v163, %[o3]\n\t" \
  "v_add_u32_dpp v202, v202, v202 row_shr:8 row_mask:0xf bank_mask:0xf bound_ctrl:0\n\t" \
  "v_cvt_f32_u32 v180, v180\n\t" \
  "v_cvt_f32_u32 v181, v181\n\t" \
  "v_cvt_f32_u32 v182, v182\n\t" \
  "v_add_u32_dpp v202, v202, v202 row_bcast:15 row_mask:0xa bank_mask:0xf\n\t" \
  "v_cvt_f32_u32 v183, v183\n\t" \
  "v_cvt_f32_u32 v184, v184\n\t" \
  "v_cvt_f32_u32 v185, v185\n\t" \
  "v_add_u32_dpp v202, v202, v202 row_bcast:31 row_mask:0xc bank_mask:0xf\n\t" \
  "v_cvt_f32_u32 v186, v186\n\t" \
  "v_cvt_f32_u32 v187, v187\n\t" \
  "v_pk_fma_f32 v[180:181], v[180:181], %[sc], %[sc]\n\t" \
  "v_sub_u32 v201, v201, v202\n\t" \
  "v_pk_fma_f32 v[182:183], v[182:183], %[sc], %[sc]\n\t" \
  "v_pk_fma_f32 v[184:185], v[184:185], %[sc], %[sc]\n\t" \
  "v_pk_fma_f32 v[186:187], v[186:187], %[sc], %[sc]\n\t" \
  "v_lshl_add_u32 v203, v201, 2, %[p0]\n\t" \
  "ds_write_b128 %[e32], v[180:183] offset:0\n\t" \
  "ds_write_b128 %[e32], v[184:187] offset:16\n\t" \
  "s_waitcnt lgkmcnt(4)\n\t" \
  "v_and_b32 v204, %[k0f], v174\n\t" \
  "v_lshrrev_b32 v205, 4, v174\n\t" \
  "v_and_b32 v205, %[k0f], v205\n\t" \
  "v_perm_b32 v188, v204, v164, %[e0]\n\t" \
  "v_perm_b32 v189, v205, v164, %[o0]\n\t" \
  "v_perm_b32 v190, v204, v165, %[e1]\n\t" \
  "v_perm_b32 v191, v205, v165, %[o1]\n\t" \
  "v_perm_b32 v192, v204, v166, %[e2]\n\t" \
  "v_perm_b32 v193, v205, v166, %[o2]\n\t" \
  "v_perm_b32 v194, v204, v167, %[e3]\n\t" \
  "v_perm_b32 v195, v205, v167, %[o3]\n\t" \
  "v_cvt_f32_u32 v188, v188\n\t" \
  "v_cvt_f32_u32 v189, v189\n\t" \
  "v_cvt_f32_u32 v190, v190\n\t" \
  "v_cvt_f32_u32 v191, v191\n\t" \
  "v_cvt_f32_u32 v192, v192\n\t" \
  "v_cvt_f32_u32 v193, v193\n\t" \
  "v_cvt_f32_u32 v194, v194\n\t" \
  "v_cvt_f32_u32 v195, v195\n\t" \
  "v_pk_fma_f32 v[188:189], v[188:189], %[sc], %[sc]\n\t" \
  "v_pk_fma_f32 v[190:191], v[190:191], %[sc], %[sc]\n\t" \
  "v_pk_fma_f32 v[192:193], v[192:193], %[sc], %[sc]\n\t" \
  "v_pk_fma_f32 v[194:195], v[194:195], %[sc], %[sc]\n\t" \
  "ds_write_b128 %[e32], v[188:191] offset:2048\n\t" \
  "ds_write_b128 %[e32], v[192:195] offset:2064\n\t" \
  "s_waitcnt lgkmcnt(4)\n\t" \
  "v_lshlrev_b32 v206, 4, v176\n\t" \
  "v_lshrrev_b32 v207, 4, v175\n\t" \
  "v_bfi_b32 v204, %[k0f], v175, v206\n\t" \
  "v_bfi_b32 v205, %[k0f], v207, v176\n\t" \
  "v_lshrrev_b32 v207, 2, v177\n\t" \
  "v_and_b32 v206, 0x3030303, v177\n\t" \
  "v_and_b32 v207, 0x3030303, v207\n\t" \
  "v_perm_b32 v160, v206, v204, %[g0]\n\t" \
  "v_perm_b32 v161, v206, v204, %[g1]\n\t" \
  "v_perm_b32 v162, v207, v205, %[g0]\n\t" \
  "v_perm_b32 v163, v207, v205, %[g1]\n\t" \
  "v_perm_b32 v180, v160, v168, %[h0]\n\t" \
  "v_perm_b32 v181, v162, v168, %[h1]\n\t" \
  "v_perm_b32 v182, v160, v169, %[h2]\n\t" \
  "v_perm_b32 v183, v162, v169, %[h3]\n\t" \
  "v_perm_b32 v184, v161, v170, %[h0]\n\t" \
  "v_perm_b32 v185, v163, v170, %[h1]\n\t" \
  "v_perm_b32 v186, v161, v171, %[h2]\n\t" \
  "v_perm_b32 v187, v163, v171, %[h3]\n\t" \
  "v_add_u32 v180, 1, v180\n\t" \
  "v_add_u32 v181, 1, v181\n\t" \
  "v_add_u32 v182, 1, v182\n\t" \
  "v_add_u32 v183, 1, v183\n\t" \
  "v_add_u32 v184, 1, v184\n\t" \
  "v_add_u32 v185, 1, v185\n\t" \
  "v_add_u32 v186, 1, v186\n\t" \
  "v_add_u32 v187, 1, v187\n\t" \
  "v_cvt_f32_u32 v180, v180\n\t" \
  "v_cvt_f32_u32 v181, v181\n\t" \
  "v_cvt_f32_u32 v182, v182\n\t" \
  "v_cvt_f32_u32 v183, v183\n\t" \
  "v_cvt_f32_u32 v184, v184\n\t" \
  "v_cvt_f32_u32 v185, v185\n\t" \
  "v_cvt_f32_u32 v186, v186\n\t" \
  "v_cvt_f32_u32 v187, v187\n\t" \
  "v_pk_mul_f32 v[180:181], v[180:181], %[sc]\n\t" \
  "v_pk_mul_f32 v[182:183], v[182:183], %[sc]\n\t" \
  "v_pk_mul_f32 v[184:185], v[184:185], %[sc]\n\t" \
  "v_pk_mul_f32 v[186:187], v[186:187], %[sc]\n\t" \
  "s_bfm_b64 exec, %[n3], 0\n\t" \
  "ds_write_b128 v200, v[180:183]\n\t" \
  "ds_write_b128 v200, v[184:187] offset:16\n\t" \
  "s_bfm_b64 exec, %[nexc], %[n3]\n\t" \
  "ds_write_b128 v200, v[180:183]\n\t" \
  "ds_write_b128 v200, v[184:187] offset:16\n\t" \
  "s_mov_b64 exec, -1\n\t" \
  "v_bfe_i32 v160, v172, 31, 1\n\t" \
  "v_bfi_b32 v164, v160, %[z], v203\n\t" \
  "v_lshl_add_u32 v203, v160, 2, v203\n\t" \
  "ds_read_b32 v128, v164 offset:0\n\t" \
  "v_bfe_i32 v161, v172, 30, 1\n\t" \
  "v_bfi_b32 v165, v161, %[z], v203\n\t" \
  "v_lshl_add_u32 v203, v161, 2, v203\n\t" \
  "ds_read_b32 v129, v165 offset:4\n\t" \
  "v_bfe_i32 v162, v172, 29, 1\n\t" \
  "v_bfi_b32 v166, v162, %[z], v203\n\t" \
  "v_lshl_add_u32 v203, v162, 2, v203\n\t" \
  "ds_read_b32 v130, v166 offset:8\n\t" \
  "v_bfe_i32 v163, v172, 28, 1\n\t" \
  "v_bfi_b32 v167, v163, %[z], v203\n\t" \
  "v_lshl_add_u32 v203, v163, 2, v203\n\t" \
  "ds_read_b32 v131, v167 offset:12\n\t" \
  "v_bfe_i32 v160, v172, 27, 1\n\t" \
  "v_bfi_b32 v164, v160, %[z], v203\n\t" \
  "v_lshl_add_u32 v203, v160, 2, v203\n\t" \
  "ds_read_b32 v132, v164 offset:16\n\t" \
  "v_bfe_i32 v161, v172, 26, 1\n\t" \
  "v_bfi_b32 v165, v161, %[z], v203\n\t" \
  "v_lshl_add_u32 v203, v161, 2, v203\n\t" \
  "ds_read_b32 v133, v165 offset:20\n\t" \
  "v_bfe_i32 v162, v172, 25, 1\n\t" \
  "v_bfi_b32 v166, v162, %[z], v203\n\t" \
  "v_lshl_add_u32 v203, v162, 2, v203\n\t" \
  "ds_read_b32 v134, v166 offset:24\n\t" \
  "v_bfe_i32 v163, v172, 24, 1\n\t" \
  "v_bfi_b32 v167, v163, %[z], v203\n\t" \
  "v_lshl_add_u32 v203, v163, 2, v203\n\t" \
  "ds_read_b32 v135, v167 offset:28\n\t" \
  "v_bfe_i32 v160, v172, 23, 1\n\t" \
  "v_bfi_b32 v164, v160, %[z], v203\n\t" \
  "v_lshl_add_u32 v203, v160, 2, v203\n\t" \
  "ds_read_b32 v136, v164 offset:32\n\t" \
  "v_bfe_i32 v161, v172, 22, 1\n\t" \
  "v_bfi_b32 v165, v161, %[z], v203\n\t" \
  "v_lshl_add_u32 v203, v161, 2, v203\n\t" \
  "ds_read_b32 v137, v165 offset:36\n\t" \
  "v_bfe_i32 v162, v172, 21, 1\n\t" \
  "v_bfi_b32 v166, v162, %[z], v203\n\t" \
  "v_lshl_add_u32 v203, v162, 2, v203\n\t" \
  "ds_read_b32 v138, v166 offset:40\n\t" \
  "v_bfe_i32 v163, v172, 20, 1\n\t" \
  "v_bfi_b32 v167, v163, %[z], v203\n\t" \
  "v_lshl_add_u32 v203, v163, 2, v203\n\t" \
  "ds_read_b32 v139, v167 offset:44\n\t" \
  "s_waitcnt lgkmcnt(8)\n\t" \
  "global_store_dwordx4 %[w0], v[128:131], %[ob] sc0 sc1\n\t" \
  "v_bfe_i32 v160, v172, 19, 1\n\t" \
  "v_bfi_b32 v164, v160, %[z], v203\n\t" \
  "v_lshl_add_u32 v203, v160, 2, v203\n\t" \
  "ds_read_b32 v140, v164 offset:48\n\t" \
  "v_bfe_i32 v161, v172, 18, 1\n\t" \
  "v_bfi_b32 v165, v161, %[z], v203\n\t" \
  "v_lshl_add_u32 v203, v161, 2, v203\n\t" \
  "ds_read_b32 v141, v165 offset:52\n\t" \
  "v_bfe_i32 v162, v172, 17, 1\n\t" \
  "v_bfi_b32 v166, v162, %[z], v203\n\t" \
  "v_lshl_add_u32 v203, v162, 2, v203\n\t" \
  "ds_read_b32 v142, v166 offset:56\n\t" \
  "v_bfe_i32 v163, v172, 16, 1\n\t" \
  "v_bfi_b32 v167, v163, %[z], v203\n\t" \
  "v_lshl_add_u32 v203, v163, 2, v203\n\t" \
  "ds_read_b32 v143, v167 offset:60\n\t" \
  "s_waitcnt lgkmcnt(8)\n\t" \
  "global_store_dwordx4 %[w1], v[132:135], %[ob] sc0 sc1\n\t" \
  "v_bfe_i32 v160, v172, 15, 1\n\t" \
  "v_bfi_b32 v164, v160, %[z], v203\n\t" \
  "v_lshl_add_u32 v203, v160, 2, v203\n\t" \
  "ds_read_b32 v144, v164 offset:64\n\t" \
  "v_bfe_i32 v161, v172, 14, 1\n\t" \
  "v_bfi_b32 v165, v161, %[z], v203\n\t" \
  "v_lshl_add_u32 v203, v161, 2, v203\n\t" \
  "ds_read_b32 v145, v165 offset:68\n\t" \
  "v_bfe_i32 v162, v172, 13, 1\n\t" \
  "v_bfi_b32 v166, v162, %[z], v203\n\t" \
  "v_lshl_add_u32 v203, v162, 2, v203\n\t" \
  "ds_read_b32 v146, v166 offset:72\n\t" \
  "v_bfe_i32 v163, v172, 12, 1\n\t" \
  "v_bfi_b32 v167, v163, %[z], v203\n\t" \
  "v_lshl_add_u32 v203, v163, 2, v203\n\t" \
  "ds_read_b32 v147, v167 offset:76\n\t" \
  "s_waitcnt lgkmcnt(8)\n\t" \
  "global_store_dwordx4 %[w2], v[136:139], %[ob] sc0 sc1\n\t" \
  "v_bfe_i32 v160, v172, 11, 1\n\t" \
  "v_bfi_b32 v164, v160, %[z], v203\n\t" \
  "v_lshl_add_u32 v203, v160, 2, v203\n\t" \
  "ds_read_b32 v148, v164 offset:80\n\t" \
  "v_bfe_i32 v161, v172, 10, 1\n\t" \
  "v_bfi_b32 v165, v161, %[z], v203\n\t" \
  "v_lshl_add_u32 v203, v161, 2, v203\n\t" \
  "ds_read_b32 v149, v165 offset:84\n\t" \
  "v_bfe_i32 v162, v172, 9, 1\n\t" \
  "v_bfi_b32 v166, v162, %[z], v203\n\t" \
  "v_lshl_add_u32 v203, v162, 2, v203\n\t" \
  "ds_read_b32 v150, v166 offset:88\n\t" \
  "v_bfe_i32 v163, v172, 8, 1\n\t" \
  "v_bfi_b32 v167, v163, %[z], v203\n\t" \
  "v_lshl_add_u32 v203, v163, 2, v203\n\t" \
  "ds_read_b32 v151, v167 offset:92\n\t" \
  "s_waitcnt lgkmcnt(8)\n\t" \
  "global_store_dwordx4 %[w3], v[140:143], %[ob] sc0 sc1\n\t" \
  "v_bfe_i32 v160, v172, 7, 1\n\t" \
  "v_bfi_b32 v164, v160, %[z], v203\n\t" \
  "v_lshl_add_u32 v203, v160, 2, v203\n\t" \
  "ds_read_b32 v152, v164 offset:96\n\t" \
  "v_bfe_i32 v161, v172, 6, 1\n\t" \
  "v_bfi_b32 v165, v161, %[z], v203\n\t" \
  "v_lshl_add_u32 v203, v161, 2, v203\n\t" \
  "ds_read_b32 v153, v165 offset:100\n\t" \
  "v_bfe_i32 v162, v172, 5, 1\n\t" \
  "v_bfi_b32 v166, v162, %[z], v203\n\t" \
  "v_lshl_add_u32 v203, v162, 2, v203\n\t" \
  "ds_read_b32 v154, v166 offset:104\n\t" \
  "v_bfe_i32 v163, v172, 4, 1\n\t" \
  "v_bfi_b32 v167, v163, %[z], v203\n\t" \
  "v_lshl_add_u32 v203, v163, 2, v203\n\t" \
  "ds_read_b32 v155, v167 offset:108\n\t" \
  "s_waitcnt lgkmcnt(8)\n\t" \
  "global_store_dwordx4 %[w4], v[144:147], %[ob] sc0 sc1\n\t" \
  "v_bfe_i32 v160, v172, 3, 1\n\t" \
  "v_bfi_b32 v164, v160, %[z], v203\n\t" \
  "v_lshl_add_u32 v203, v160, 2, v203\n\t" \
  "ds_read_b32 v156, v164 offset:112\n\t" \
  "v_bfe_i32 v161, v172, 2, 1\n\t" \
  "v_bfi_b32 v165, v161, %[z], v203\n\t" \
  "v_lshl_add_u32 v203, v161, 2, v203\n\t" \
  "ds_read_b32 v157, v165 offset:116\n\t" \
  "v_bfe_i32 v162, v172, 1, 1\n\t" \
  "v_bfi_b32 v166, v162, %[z], v203\n\t" \
  "v_lshl_add_u32 v203, v162, 2, v203\n\t" \
  "ds_read_b32 v158, v166 offset:120\n\t" \
  "v_bfe_i32 v163, v172, 0, 1\n\t" \
  "v_bfi_b32 v167, v163, %[z], v203\n\t" \
  "v_lshl_add_u32 v203, v163, 2, v203\n\t" \
  "ds_read_b32 v159, v167 offset:124\n\t" \
  "s_waitcnt lgkmcnt(8)\n\t" \
  "global_store_dwordx4 %[w5], v[148:151], %[ob] sc0 sc1\n\t" \
  "s_waitcnt lgkmcnt(4)\n\t" \
  "global_store_dwordx4 %[w6], v[152:155], %[ob] sc0 sc1\n\t" \
  "s_waitcnt lgkmcnt(0)\n\t" \
  "global_store_dwordx4 %[w7], v[156:159], %[ob] sc0 sc1\n\t" \
  "s_nop 1\n\t" \
  ""
#define CLOB_GEMM "v128", "v129", "v130", "v131", "v132", "v133", "v134", "v135", "v136", "v137", "v138", "v139", "v140", "v141", "v142", "v143", "v144", "v145", "v146", "v147", "v148", "v149", "v150", "v151", "v152", "v153", "v154", "v155", "v156", "v157", "v158", "v159", "v160", "v161", "v162", "v163", "v164", "v165", "v166", "v167", "v168", "v169", "v170", "v171", "v172", "v173", "v174", "v175", "v176", "v177", "v178", "v179", "v180", "v181", "v182", "v183", "v184", "v185", "v186", "v187", "v188", "v189", "v190", "v191", "v192", "v193", "v194", "v195", "v196", "v197", "v198", "v199", "v200", "v201", "v202", "v203", "v204", "v205", "v206", "v207", "v208", "v209", "v210", "v211", "v212", "v213", "v214", "v215", "v216", "v217", "v218", "v219", "v220", "v221", "v222", "v223"
#define CLOB_ENC1 "v128", "v129", "v130", "v131", "v132", "v133", "v134", "v135"
#define CLOB_ENC2 "v128", "v129", "v130", "v131", "v132", "v133", "v134", "v135", "v136", "v137", "v138", "v139", "v140", "v141", "v142", "v143", "v144", "v145", "v146", "v147", "v148", "v149", "v150", "v151", "v152", "v153", "v154", "v155", "v156", "v157", "v158", "v159", "v160", "v161", "v162", "v163", "v164", "v165", "v166", "v167", "v168"
#define CLOB_DEC "v128", "v129", "v130", "v131", "v132", "v133", "v134", "v135", "v136", "v137", "v138", "v139", "v140", "v141", "v142", "v143", "v144", "v145", "v146", "v147", "v148", "v149", "v150", "v151", "v152", "v153", "v154", "v155", "v156", "v157", "v158", "v159", "v160", "v161", "v162", "v163", "v164", "v165", "v166", "v167", "v168", "v169", "v170", "v171", "v172", "v173", "v174", "v175", "v176", "v177", "v178", "v179", "v180", "v181", "v182", "v183", "v184", "v185", "v186", "v187", "v188", "v189", "v190", "v191", "v192", "v193", "v194", "v195", "v196", "v197", "v198", "v199", "v200", "v201", "v202", "v203", "v204", "v205", "v206", "v207"

__device__ __forceinline__ int rtmap(int rank, int i) {
  return i < 28 ? ((rank + 1 + i % 7) & 7) * 4 + i / 7 : rank * 4 + (i - 28);
}

struct Ctx {
  const char* wsme;
  u32 call;
  u64 t0;
  volatile LAS u32* abortw;
  __device__ __forceinline__ bool expired() const {
    const u64 dt = rtime() - t0;
    if (dt < 100000ull) return false;
    if (rfl(ld32w(wsme, TMO_OFF)) == call || dt > DEADLINE) {
      st32(wsme, TMO_OFF, call);
      *abortw = 1u;
      return true;
    }
    return false;
  }
  __device__ __forceinline__ bool expired_lds() const {
    const u64 dt = rtime() - t0;
    if (dt < 100000ull) return false;
    if (rfl(*abortw) != 0u) return true;
    if (dt > DEADLINE) { *abortw = 1u; return true; }
    return false;
  }
};

#define STAMPV(k, v) do { } while (0)
#define STAMP(k) do { } while (0)
#define ST2(k) do { } while (0)
#define PT(acc) do { } while (0)

__global__ void __launch_bounds__(512) gemm_rs_kernel(Args a) {
  extern __shared__ __attribute__((aligned(16))) char lds_raw[];
  LAS char* L8 = (LAS char*)lds_raw;
  const int lane = threadIdx.x & 63;
  const int wave = (int)rfl(threadIdx.x >> 6);
  const int b = blockIdx.x;
  const int rank = a.rank & 7;
  char* wsme = a.ws[rank];
  const u64 t0 = rtime();
  const int ti = b >> 3, ct = b & 7;
  const int rt = rtmap(rank, ti);

  u32x4 sv[16];
  const bool early_w = b < 64 && wave < 2, early_x = b >= 64 && b < 96 && wave == 0;
  const int e_w = (((b & 7) * 2 + wave) << 3) + (b >> 3), e_x = b - 64;
  const int s_n = b < 196 ? b * 2 + wave - 2 : 392 + (b - 224) * 3 + wave - 1, s_r = s_n % 160;
  const bool late = wave < 4 && (b < 196 ? wave >= 2 : b >= 224 && wave >= 1 && s_n < 480);
  const bool stage_w = late ? s_r < 128 : early_w, stage_x = late ? s_r >= 128 : early_x;
  const int s_id = late ? s_r : e_w;
  const int s_xp = late ? s_r - 128 : e_x;
  const int s_kp = late ? 1 + s_n / 160 : 0, s_dst = (rank + 1 + (s_id & 7)) & 7, s_a = stage_w ? s_id >> 3 : (s_xp & 7) * 4 + (s_xp >> 3);
  const u64 wsl = *(const __attribute__((address_space(4))) u64*)((const __attribute__((address_space(4))) char*)__builtin_amdgcn_kernarg_segment_ptr() + offsetof(Args, ws) + (lane & 7) * 8);
  auto stage_load = [&]() {
    if (stage_w) {
      const char* src = a.w + (size_t)(s_kp * 128 + (lane >> 3) * 16) * NN + s_dst * NSH + s_a * 128 + (lane & 7) * 16;
#pragma unroll
      for (int i = 0; i < 16; ++i) sv[i] = *(const GAS u32x4*)(src + (size_t)i * NN);
    } else {
      const char* src = a.x + (size_t)(s_a * 128 + (lane >> 3)) * KS + s_kp * 128;
#pragma unroll
      for (int i = 0; i < 16; ++i) {
        const int row = 8 * i + (lane >> 3);
        sv[i] = *(const GAS u32x4*)(src + i * 8 * KS + (((lane & 7) ^ ((row >> 1) & 7)) << 4));
      }
    }
  };
  if ((stage_w || stage_x) && !late) stage_load();
  if ((u32)(b - 96) < WARM && wave == 5) {
    const u32 pe = lane & 7, pa = ((lane >> 3) & 1) * PAR_BYTES, r = lane >> 4;
    const u64 a0 = wsl + pa + (r == 0 ? WI_OFF + rank * MiB : r == 1 ? XI_OFF + pe * 2 * MiB : r == 2 ? WSO_OFF + rank * MiB : FL_OFF + 65536u);
    const u64 a1 = wsl + pa + OSP_OFF + (r * 7 + ((rank - pe - 1) & 7)) * (MiB / 2);
    const u64 a2 = lane < 48 ? (u64)wsme + (lane >= 24 ? PAR_BYTES + (lane - 24) * 2 * MiB : lane * 2 * MiB) : (u64)a.out + (lane - 48) * 2 * MiB;
    u32 d0, d1, d2;
    asm volatile("global_load_dword %0, %3, off sc0 sc1\n\tglobal_load_dword %2, %5, off sc0 sc1\n\tglobal_load_dword %1, %4, off sc0 sc1\n\ts_waitcnt vmcnt(0)"
                 : "=&v"(d0), "=&v"(d1), "=&v"(d2) : "v"(a0), "v"(a1), "v"(a2) : "memory");
  }
  u32 call;
  {
    const char* cp = wsme + CNT_OFF + b * 4;
    asm volatile("s_load_dword %0, %1, 0x0\n\ts_waitcnt lgkmcnt(0)" : "=s"(call) : "s"(cp) : "memory");
    call += 1u;
  }
  if (wave == 7) { ((LAS u32*)(L8 + SYNC_L))[lane] = 0u; ((LAS u32*)(L8 + SYNC_L))[lane + 64] = 0u; }
  LGKM_WAIT0();
  __builtin_amdgcn_s_barrier();
  asm volatile("" ::: "memory");
  const u32 par = (call & 1u) * PAR_BYTES;
  const char* wp = wsme + par;
  volatile LAS u32* FULL = (volatile LAS u32*)(L8 + FULL_L);
  volatile LAS u32* FREE = (volatile LAS u32*)(L8 + FREE_L);
  Ctx cx{wsme, call, t0, (volatile LAS u32*)(L8 + ABORT_L)};
  STAMP(0); STAMPV(7, t0);
  if (wave >= 4) __builtin_amdgcn_s_waitcnt(0x0F70);
  if (wave == 6 && lane == 0) st32(wsme, CNT_OFF + b * 4, call);

  if (wave < 4) {
    if (late) {
      const u64 dl = STG_T0 + ((u32)s_n * STG_DT >> 4);
      while (rtime() - t0 < dl) __builtin_amdgcn_s_sleep(2);
      stage_load();
    }
    if (stage_w) {
      u32x4 U[16];
#pragma unroll
      for (int g = 0; g < 4; ++g)
#pragma unroll
        for (int q = 0; q < 4; ++q) {
          const u32 r0 = sv[4 * g][q], r1 = sv[4 * g + 1][q], r2 = sv[4 * g + 2][q], r3 = sv[4 * g + 3][q];
          const u32 p0 = __builtin_amdgcn_perm(r1, r0, 0x05010400u), p1 = __builtin_amdgcn_perm(r1, r0, 0x07030602u);
          const u32 p2 = __builtin_amdgcn_perm(r3, r2, 0x05010400u), p3 = __builtin_amdgcn_perm(r3, r2, 0x07030602u);
          U[4 * q + 0][g] = __builtin_amdgcn_perm(p2, p0, 0x05040100u);
          U[4 * q + 1][g] = __builtin_amdgcn_perm(p2, p0, 0x07060302u);
          U[4 * q + 2][g] = __builtin_amdgcn_perm(p3, p1, 0x05040100u);
          U[4 * q + 3][g] = __builtin_amdgcn_perm(p3, p1, 0x07060302u);
        }
      const bool direct = s_dst == rank || (s_kp == 0 && s_a < NPUSH);
      const char* dw = a.ws[s_dst & 7] + par;
      const char* tw = direct ? dw : wp;
      const u32 mo = direct ? WI_OFF + (u32)((rank * 16 + s_a) * 4 + s_kp) * MSG : WSO_OFF + (u32)((s_dst * 16 + s_a) * 4 + s_kp) * MSG;
      if (PACE < 16 && direct && s_dst != rank) {
#define PW() asm volatile("s_waitcnt vmcnt(%0)" :: "n"(PACE - 1) : "memory")
#pragma unroll
        for (int g = 0; g < 4; ++g) {
          const u32 off = mo + g * 4096 + ((lane ^ (g << 2)) << 4);
          st128<0>(tw, off, U[4 * g]); PW(); st128<1024>(tw, off, U[4 * g + 1]); PW();
          st128<2048>(tw, off, U[4 * g + 2]); PW(); st128<3072>(tw, off, U[4 * g + 3]); PW();
        }
      } else {
#pragma unroll
        for (int g = 0; g < 4; ++g) {
          const u32 off = mo + g * 4096 + ((lane ^ (g << 2)) << 4);
          st128<0>(tw, off, U[4 * g]); st128<1024>(tw, off, U[4 * g + 1]);
          st128<2048>(tw, off, U[4 * g + 2]); st128<3072>(tw, off, U[4 * g + 3]);
        }
      }
      VM_WAIT(0);
      if (lane == 0) st32(dw, direct ? FW_OFF + s_a * 128 + (rank * 4 + s_kp) * 4 : RDY_OFF + rank * 1024 + (128 + s_a * 4 + s_kp) * 4, call);
    } else if (stage_x) {
      const u32 mo = XI_OFF + (u32)((rank * 32 + s_a) * 4 + s_kp) * MSG;
#pragma unroll
      for (int g = 0; g < 4; ++g) {
        const u32 off = mo + g * 4096 + (lane << 4);
        st128<0>(wp, off, sv[4 * g]); st128<1024>(wp, off, sv[4 * g + 1]);
        st128<2048>(wp, off, sv[4 * g + 2]); st128<3072>(wp, off, sv[4 * g + 3]);
      }
      VM_WAIT(0);
      if (lane < 8) st32v(wsl + par + (lane == rank ? FX_OFF + s_a * 128 + (rank * 4 + s_kp) * 4 : RDY_OFF + rank * 1024 + (s_a * 4 + s_kp) * 4), call);
    }
    STAMP(1);
    const int wr = wave >> 1, wc = wave & 1;
    const int c = lane & 15, kq = lane >> 4;
    const u32 sbits = rfl(__builtin_bit_cast(u32, *(const GAS float*)a.sx * *(const GAS float*)a.sw));
    const float scale = __builtin_bit_cast(float, sbits & 0x7FFFFFFFu);
    i32x16 acc[8];
    u32 dead;
    {
      u32 a0 = wr * 8192 + c * 128 + ((kq ^ (c >> 1)) << 4);
      u32 b0 = 16384 + wc * 16384 + (c & 3) * 4096 + (((kq * 8 + (c >> 2)) ^ ((c & 3) << 2)) << 4);
      u32 a1 = a0 ^ 64u, b1 = b0 ^ 64u;
      u32 vt, vf, vi, st, need, sf, d, t, pc;
      asm volatile(ASM_GEMM
                   : "=&{v[0:15]}"(acc[0]), "=&{v[16:31]}"(acc[1]), "=&{v[32:47]}"(acc[2]), "=&{v[48:63]}"(acc[3]),
                     "=&{v[64:79]}"(acc[4]), "=&{v[80:95]}"(acc[5]), "=&{v[96:111]}"(acc[6]), "=&{v[112:127]}"(acc[7]),
                     [a0] "+v"(a0), [a1] "+v"(a1), [b0] "+v"(b0), [b1] "+v"(b1), [vt] "=&v"(vt), [vf] "=&v"(vf), [vi] "=&v"(vi),
                     [st] "=&s"(st), [need] "=&s"(need), [sf] "=&s"(sf), [d] "=&s"(d), [t] "=&s"(t), [dead] "=&s"(dead), [pc] "=&s"(pc)
                   : [vfree] "v"(FREE_L + wave * 4), [init] "s"((sbits >> 31) ? 0ull : ~0ull), [neg] "s"(sbits >> 31), [t0] "s"((u32)t0),
                     [wrap] "s"(0u - 2u * SLOT), [slot] "n"(SLOT), [ful] "n"(FULL_L), [abl] "n"(ABORT_L), [dl] "n"((u32)DEADLINE)
                   : CLOB_GEMM, "vcc", "scc", "memory");
    }
    STAMP(3);
    if (ti >= 28) {
      const u32 ob = (u32)(((ti - 28) * 128 + wr * 64 + kq * 4) * NN + rank * NSH + ct * 256 + wc * 128 + 4 * c) * 4u;
      if (dead) {
        const float nan = __builtin_nanf("");
        st_out(a.out, ob, f32x4{nan, nan, nan, nan});
      } else {
#pragma unroll
        for (int mt = 0; mt < 4; ++mt)
#pragma unroll
          for (int G = 0; G < 2; ++G)
#pragma unroll
            for (int r = 0; r < 4; ++r) {
              f32x4 y;
#pragma unroll
              for (int t = 0; t < 4; ++t) y[t] = (float)max(acc[mt * 2 + G][t * 4 + r] + 1, 0) * scale;
              st_out(a.out, ob + (u32)((mt * 16 + r) * NN + G * 64) * 4u, y);
            }
      }
    } else if (!dead) {
      const u32 C = 2 * SLOT + wave * PK_STRIDE;
      u32 m[4], p[4], nd[4];
      asm volatile(ASM_ENC1
                   : [m0] "=&v"(m[0]), [m1] "=&v"(m[1]), [m2] "=&v"(m[2]), [m3] "=&v"(m[3]),
                     [p0] "=&v"(p[0]), [p1] "=&v"(p[1]), [p2] "=&v"(p[2]), [p3] "=&v"(p[3]),
                     [n0] "=&s"(nd[0]), [n1] "=&s"(nd[1]), [n2] "=&s"(nd[2]), [n3] "=&s"(nd[3])
                   : "{v[0:15]}"(acc[0]), "{v[16:31]}"(acc[1]), "{v[32:47]}"(acc[2]), "{v[48:63]}"(acc[3]),
                     "{v[64:79]}"(acc[4]), "{v[80:95]}"(acc[5]), "{v[96:111]}"(acc[6]), "{v[112:127]}"(acc[7]),
                     [pb] "v"(C + lane * 128)
                   : CLOB_ENC1);
      for (;;) {
        const u32x4 f = *(volatile LAS u32x4*)FREE;
        const u32 mn = rfl(min(min(f[0], f[1]), min(f[2], f[3])));
        if (mn >= 30u) break;
        if (cx.expired_lds()) { dead = 1u; break; }
      }
      if (!dead) {
        const u32 slot0 = (u32)(b * 4 + wave) * 4u;
        u32 infos = 0;
#define ENC2(MT, R0, R1) { \
          const u32 cnt = 2048u - nd[MT], nch = (cnt + 7u) >> 3; \
          u32 nexc = 64u; \
          if (nch <= 191u) { \
            u32 t, t2, rb; \
            asm volatile(ASM_ENC2_##MT \
                         : [p] "+v"(p[MT]), [nexc] "=&s"(nexc), [t] "=&s"(t), [t2] "=&s"(t2), [rb] "=&s"(rb) \
                         : R0(acc[2 * MT]), R1(acc[2 * MT + 1]), [m] "v"(m[MT]), [cnt] "s"(cnt), [nch] "s"(nch), \
                           [c4] "v"(C + lane * 4), [c32] "v"(C + lane * 32), [t16] "v"(C + 6400u + lane * 16), [dum] ENC_DUMMY, \
                           [so] "v"(OSP_OFF + (slot0 + MT) * 4096u + lane * 16), [wp] "s"(wp), [lane] "v"(lane), [l16] "v"(lane * 16), \
                           [vx] "v"(((lane & 15) << 4) | ((lane >> 4) << 12)), [rec0] "s"(C + 6656u), \
                           [s10] "s"(0x05040100u), [s62] "s"(0x0c0c0602u), [s73] "s"(0x0c0c0703u), [k0f] "s"(0x0F0F0F0Fu), [lim] "s"(0xFFFFFu) \
                         : CLOB_ENC2, "vcc", "scc", "memory"); \
          } \
          u32 info = nch | (nexc << 8); \
          if ((nch > 128u ? nch - 128u : 0u) + nexc > 63u || nch * 20u + nexc * 8u > 3840u) { \
            const u32 so = OSR_OFF + (slot0 + MT) * 8192u + lane * 16; \
            _Pragma("unroll") for (int g = 0; g < 8; ++g) { \
              const i32x16 v = acc[2 * MT + (g >> 2)]; \
              st128<0>(wp, so + g * 1024, u32x4{(u32)v[g & 3], (u32)v[4 + (g & 3)], (u32)v[8 + (g & 3)], (u32)v[12 + (g & 3)]}); \
            } \
            info = RAW_MODE; \
          } \
          if (lane == MT) infos = info; }
        ENC2(0, "{v[0:15]}", "{v[16:31]}")
        ENC2(1, "{v[32:47]}", "{v[48:63]}")
        ENC2(2, "{v[64:79]}", "{v[80:95]}")
        ENC2(3, "{v[96:111]}", "{v[112:127]}")
        STAMP(5);
        VM_WAIT(0);
        const int owner = rt >> 2, j = rt & 3;
        if (lane < 4) st64(a.ws[owner & 7] + par, OF_OFF + (u32)((((rank * 4 + j) * 8 + ct) * 4 + wave) * 4 + lane) * 8u, u32x2{call, infos});
      }
    }
    STAMP(4);
  } else {
    const bool set1 = b >= 228, hosts = set1 || (b >= 196 && b < 224);
    if (wave >= 6 && hosts) {
      __builtin_amdgcn_s_setprio(3);
      const int hb = (b - 196) & 31, l = hb % 7, sub = hb / 7 * 2 + (wave - 6);
      const int p = (rank + 1 + l) & 7;
      const char* pp = a.ws[p] + par;
      const u32 tab = AUX_L + (wave - 6) * 4096;
      const u32 ctr = CTR_OFF + (set1 * 8 + l) * 128;
      const u32 m0 = set1 ? XCUT : NPUSH, nm = set1 ? 192 - XCUT : XCUT - NPUSH;
      if (!set1 && lane == 0) __hip_atomic_fetch_add((LAS u32*)(L8 + HOLD_L), 0x100u, __ATOMIC_RELAXED, __HIP_MEMORY_SCOPE_WORKGROUP);
      if (lane == 0) asm volatile("global_atomic_umax_x2 %0, %1, %2 sc1" :: "v"(ctr), "v"(((u64)call << 8) + 8u), "s"(wp) : "memory");
      u32 rf[3];
#pragma unroll
      for (int q = 0; q < 3; ++q) {
        const int m = min(q * 64 + lane + (int)m0, 191);
        constexpr int NSEG = 19;
        constexpr int SEG[NSEG][3] = {{16, 0, -1}, {4, 0, 28}, {28, 0, 0}, {16, 1, -1}, {14, 1, 0}, {16, 2, -1}, {7, 2, 0}, {7, 1, 14}, {16, 3, -1}, {7, 3, 0},
                                      {7, 2, 7}, {7, 3, 7}, {7, 1, 21}, {14, 2, 14}, {14, 3, 14}, {4, 1, 28}, {4, 2, 28}, {4, 3, 28}, {0, 0, 0}};
        int kp = 0, i = -1, ch = m, s0 = 0;
#pragma unroll
        for (int g = 1; g < NSEG; ++g) {
          s0 += SEG[g - 1][0];
          if (SEG[g][0] && m >= s0) { kp = SEG[g][1]; ch = m - s0; i = SEG[g][2] < 0 ? -1 : SEG[g][2] + m - s0; }
        }
        u32 so, dof, lf;
        if (i >= 0) {
          const int r2 = rtmap(rank, i);
          so = dof = XI_OFF + (u32)((p * 32 + r2) * 4 + kp) * MSG;
          lf = FX_OFF + r2 * 128 + (p * 4 + kp) * 4;
          rf[q] = RDY_OFF + p * 1024 + (r2 * 4 + kp) * 4;
        } else {
          so = WSO_OFF + (u32)((rank * 16 + ch) * 4 + kp) * MSG;
          dof = WI_OFF + (u32)((p * 16 + ch) * 4 + kp) * MSG;
          lf = FW_OFF + ch * 128 + (p * 4 + kp) * 4;
          rf[q] = RDY_OFF + p * 1024 + (128 + ch * 4 + kp) * 4;
        }
        *(LAS u32x4*)(L8 + tab + (q * 64 + lane) * 16) = u32x4{so, dof, lf, 0u};
      }
      LGKM_WAIT0();
      {
        u32 idx = sub;
        bool first = !set1;
        if (set1) {
          for (;;) {
            if (rfl(ld32w(wp, LNK_OFF + l * 128)) == call) break;
            __builtin_amdgcn_s_sleep(4);
            if (cx.expired()) { idx = nm; break; }
          }
          if (lane == 0) __hip_atomic_fetch_add((LAS u32*)(L8 + HOLD_L), 0x100u, __ATOMIC_RELAXED, __HIP_MEMORY_SCOPE_WORKGROUP);
          STAMP(15);
        }
        while (idx < nm) {
          const u32 v = ld32w(wp, idx < 64u ? rf[0] : idx < 128u ? rf[1] : rf[2]);
          if (rdl(v, idx & 63u) != call) { if (cx.expired()) break; continue; }
          if (first) {
            const u64 t1 = rtime();
            while (rtime() - t1 < (u32)(PULL_DLY + sub * PULL_STG)) __builtin_amdgcn_s_sleep(1);
            STAMP(15);
          }
          first = false;
          idx = pull_stream(idx, pp, wp, tab, ctr, LNK_OFF + (set1 * 8 + l) * 128, rf[0], rf[1], rf[2], lane * 16, call, nm, (set1 ? LNK_AT : GO_AT) - m0);
        }
        if (lane == 0) __hip_atomic_fetch_add((LAS u32*)(L8 + HOLD_L), 1u, __ATOMIC_RELAXED, __HIP_MEMORY_SCOPE_WORKGROUP);
        STAMP(14);
      }
    }
    if (wave < 7) {
    __builtin_amdgcn_s_setprio(LD_PRIO);
    const u32 fo = (lane < 16 ? FX_OFF + rt * 128 : FW_OFF + 2 * ct * 128 - 128) + min(lane, 47) * 8;
    u32 done = 0, ready = 0;
    const u32 vo = lane * 16, rot = (wave - 4) * 11;
    bool dead = false;
    while (!dead) {
      if (hosts) {
        const u32 h = rfl(*(volatile LAS u32*)(L8 + HOLD_L));
        if (h >= 0x100u && (h & 0xFFu) < HOLD) {
          __builtin_amdgcn_s_sleep(8);
          STAMP(13);
          if (cx.expired_lds()) break;
          continue;
        }
      }
      const u32 cand = ready & ~done;
      if (cand == 0u) {
        if (done == ~0u) break;
        const u32x2 v = ld64w(wp, fo);
        const u64 e0 = __builtin_amdgcn_ballot_w64(v[0] == call), e1 = __builtin_amdgcn_ballot_w64(v[1] == call);
        ready = ((u32)e0 & (u32)(e0 >> 16) & (u32)(e0 >> 32) & 0xFFFFu) | (((u32)e1 & (u32)(e1 >> 16) & (u32)(e1 >> 32)) << 16);
        done = rfl(*(volatile LAS u32*)(L8 + DONE_L));
        if ((ready & ~done) == 0u && done != ~0u) {
          __builtin_amdgcn_s_sleep(LD_SLEEP);
          if (cx.expired()) break;
        }
        continue;
      }
      const u32 pi = ((u32)__builtin_ctz(__builtin_amdgcn_alignbit(cand, cand, rot)) + rot) & 31u;
      u32 old, s;
      asm volatile("s_mov_b64 exec, 1\n\tds_or_rtn_b32 %0, %1, %2\n\ts_mov_b64 exec, -1\n\ts_waitcnt lgkmcnt(0)"
                   : "=&v"(old) : "v"(DONE_L), "v"(1u << pi) : "memory");
      old = rfl(old);
      done = old | (1u << pi);
      if (old == done) continue;
      asm volatile("s_mov_b64 exec, 1\n\tds_add_rtn_u32 %0, %1, %2\n\ts_mov_b64 exec, -1\n\ts_waitcnt lgkmcnt(0)"
                   : "=&v"(s) : "v"(STEP_L), "v"(1u) : "memory");
      s = rfl(s);
      PT(p_poll); ST2(n * 4);
      if (s >= 3u) {
        for (;;) {
          const u32x4 f = *(volatile LAS u32x4*)FREE;
          const u32 mn = rfl(min(min(f[0], f[1]), min(f[2], f[3])));
          if (mn + 2u >= s) break;
          if (FREE_SLEEP) __builtin_amdgcn_s_sleep(FREE_SLEEP);
          if (cx.expired_lds()) { dead = true; break; }
        }
        if (dead) break;
      }
      PT(p_free); ST2(n * 4 + 1);
      const u32 f = (pi & 15u) * 2u + (pi >> 4), src = f >> 2, kp = f & 3u;
      const u32 slot = s - (s * 43u >> 7) * 3u;
      const char* xa = wp + (XI_OFF + ((src * 32 + rt) * 4 + kp) * MSG);
      const char* wa = wp + (WI_OFF + ((src * 16 + 2 * ct) * 4 + kp) * MSG);
      const u32 lb = slot * SLOT;
      if (LD_CAP == 2 || (LD_CAP == 1 && ti >= 28)) {
#pragma unroll
        for (int g = 0; g < 4; ++g) { glds4k(lb + g * 4096, xa, vo + g * 4096); VM_WAIT(8); }
#pragma unroll
        for (int g = 0; g < 8; ++g) { glds4k(lb + 16384 + g * 4096, wa, vo + (g >> 2) * 4 * MSG + (g & 3) * 4096); VM_WAIT(8); }
      } else {
#pragma unroll
        for (int g = 0; g < 4; ++g) glds4k(lb + g * 4096, xa, vo + g * 4096);
#pragma unroll
        for (int g = 0; g < 8; ++g) glds4k(lb + 16384 + g * 4096, wa, vo + (g >> 2) * 4 * MSG + (g & 3) * 4096);
      }
      PT(p_issue); ST2(n * 4 + 2);
      u32x2 v;
      asm volatile("global_load_dwordx2 %0, %1, %2 sc0 sc1\n\ts_waitcnt vmcnt(1)\n\ts_mov_b64 exec, 1\n\tds_write_b32 %3, %4\n\t"
                   "s_mov_b64 exec, -1\n\ts_waitcnt vmcnt(0)"
                   : "=&v"(v) : "v"(fo), "s"(wp), "v"(FULL_L + slot * 4u), "v"(s + 1u) : "memory");
      PT(p_wait); ST2(n * 4 + 3);
      {
        const u64 e0 = __builtin_amdgcn_ballot_w64(v[0] == call), e1 = __builtin_amdgcn_ballot_w64(v[1] == call);
        ready = ((u32)e0 & (u32)(e0 >> 16) & (u32)(e0 >> 32) & 0xFFFFu) | (((u32)e1 & (u32)(e1 >> 16) & (u32)(e1 >> 32)) << 16);
        done = rfl(*(volatile LAS u32*)(L8 + DONE_L));
      }
    }
    STAMP(12);
    }
    if (b < 224 / DPC ? wave >= 8 - DPC : (b < JOIN_B && wave >= 8 - JOIN)) {
    {
      __builtin_amdgcn_s_setprio(3);
      const int l = b % 7;
      const int p = (rank + 1 + l) & 7;
      const char* pp = a.ws[p] + par;
      const u32 li = (rank - p - 1) & 7;
      const u32 octr = OCT_OFF + l * 128;
      if (lane == 0) asm volatile("global_atomic_umax_x2 %0, %1, %2 sc1" :: "v"(octr), "v"(((u64)call << 10) + 32u), "s"(wp) : "memory");
      const float scale = __builtin_bit_cast(float, rfl(__builtin_bit_cast(u32, *(const GAS float*)a.sx * *(const GAS float*)a.sw)) & 0x7FFFFFFFu);
      const u32 DB = wave == 7 ? AUX_L : (u32)(wave - 4) * 16384u;
      const u32 IB = DB + 8192, ZB = IB + 6144;
      const int c = lane & 15, kq = lane >> 4;
      const u32 ob = (u32)(kq * 4 * NN + p * NSH + 4 * c) * 4u;
      const u32 fb = OF_OFF + p * 4096;
      bool dead = false;
      while (GATE7 || wave != 7) {
        const u32x4 f = *(volatile LAS u32x4*)FREE;
        if (rfl(min(min(f[0], f[1]), min(f[2], f[3]))) >= 31u) break;
        __builtin_amdgcn_s_sleep(8);
        if (cx.expired_lds()) { dead = true; break; }
      }
      if (lane < 32) *(LAS u32*)(L8 + ZB + lane * 4) = 0u;
      if (wave == 7 || !LNK_RELAY) {
        while (!dead) {
          if (rfl(ld32w(wp, LNK_OFF + (8 + l) * 128)) == call && (!hosts || (rfl(*(volatile LAS u32*)(L8 + HOLD_L)) & 0xFFu) == 2u)) break;
          __builtin_amdgcn_s_sleep(LNK_SLEEP);
          if (cx.expired()) dead = true;
        }
        if (lane == 0) *(volatile LAS u32*)(L8 + OPEN_L) = 1u;
      } else {
        while (!dead) {
          if (rfl(*(volatile LAS u32*)(L8 + OPEN_L))) break;
          __builtin_amdgcn_s_sleep(1);
          if (cx.expired_lds()) dead = true;
        }
      }
      STAMP(1);
      u32 info_next = 0;
      auto look = [&](u32 n) -> bool { const u32x2 f = ld64w(wp, fb + n * 8); info_next = rfl(f[1]); return rfl(f[0]) == call; };
      auto await = [&](u32 n) -> bool {
        for (;;) {
          if (look(n)) return true;
          __builtin_amdgcn_s_sleep(4);
          if (cx.expired()) return false;
        }
      };
      auto slot_of = [&](u32 n) -> u32 { return ((n >> 7) * 7 + li) * 128 + (n & 127u); };
      auto issue = [&](u32 n, u32 info, u32 lb) {
        if (info & RAW_MODE) return;
        const u32 len = 256u + (info & 0xFFu) * 20u + ((info >> 8) & 0x3Fu) * 8u;
        const u32 vo = lane * 16;
        const u32 so = OSP_OFF + slot_of(n) * 4096u + vo;
        if (vo < len) glds1k<0>(lb, pp, so);
        if (vo + 1024u < len) glds1k<1024>(lb, pp, so);
        if (vo + 2048u < len) glds1k<2048>(lb, pp, so);
        if (vo + 3072u < len) glds1k<3072>(lb, pp, so);
      };
      u64 dr;
      u32x2 F;
#define DRAW(g) asm volatile("s_mov_b64 exec, 1\n\tglobal_atomic_add_x2 v[208:209], %2, %3, %5 sc0 sc1\n\ts_mov_b64 exec, %6\n\t" \
                             "global_load_dwordx2 v[212:213], %4, %5 sc0 sc1\n\ts_mov_b64 exec, -1" \
                             : "=&{v[208:209]}"(dr), "=&{v[212:213]}"(F) \
                             : "v"(octr), "v"(1ull), "v"(fb + min((g) + lane, 511u) * 8u), "s"(wp), "s"(WIN_N == 64 ? ~0ull : (1ull << (WIN_N & 63)) - 1ull) : "memory")
#define DRAWN(cnt) asm volatile("s_waitcnt vmcnt(" #cnt ")" : "+{v[208:209]}"(dr), "+{v[212:213]}"(F) :: "memory")
      u32 n = b / 7 * DPC + wave - (8 - DPC), lb = DB;
      if (b >= 224 / DPC && !dead) {
        while (JOIN_N) {
          if (rfl(ld32w(wp, octr)) - (call << 10) >= JOIN_N) break;
          __builtin_amdgcn_s_sleep(16);
          if (cx.expired()) break;
        }
        DRAW(0u);
        DRAWN(0);
        n = rfl((u32)dr) - (call << 10);
      }
      if (dead) n &= 31u;
      if (!dead && n < 512u) dead = !await(n);
      if (!dead && n < 512u) {
        issue(n, info_next, lb);
        u32 wb = (n + WIN_0) & ~15u;
        DRAW(wb);
        VM_WAIT(0);
        for (;;) {
          const u32 info = info_next;
          PT(p_dec); ST2(it * 4);
          DRAWN(8);
          PT(p_wait); ST2(it * 4 + 1);
          const u32 nn = rfl((u32)dr) - (call << 10);
          bool have = false;
          if (nn < 512u) {
            const u32 ln = nn - wb;
            const u32 tag = rdl(F[0], ln & 63u);
            info_next = rdl(F[1], ln & 63u);
            have = (ln < WIN_N && tag == call) || look(nn);
          }
          wb = (nn + WIN_0) & ~15u;
          DRAW(wb);
          if (have) issue(nn, info_next, lb ^ 4096u);
          PT(p_ens); ST2(it * 4 + 2);
          const char* on = a.out + (((n >> 7) * 128 + ((n >> 3) & 1u) * 64 + (n & 3u) * 16) * NN + ((n >> 4) & 7u) * 256 + ((n >> 2) & 1u) * 128) * 4u;
          if (info & RAW_MODE) {
            const u32 so = OSR_OFF + slot_of(n) * 8192u + lane * 16;
            u32x4 R[8];
            ld8kw(R, pp, so);
#pragma unroll
            for (int g = 0; g < 8; ++g) {
              const u32x4 q = R[g];
              f32x4 y;
#pragma unroll
              for (int t = 0; t < 4; ++t) y[t] = (float)max((int)q[t] + 1, 0) * scale;
              st_out(on, ob + (u32)((g & 3) * NN + (g >> 2) * 64) * 4u, y);
            }
          } else {
            u32 t, t2, n3, tb;
            const u32 sb = __builtin_bit_cast(u32, scale);
            asm volatile(ASM_DEC
                         : [t] "=&s"(t), [t2] "=&s"(t2), [n3] "=&s"(n3), [tb] "=&s"(tb)
                         : [nch] "s"(min(info & 0xFFu, 191u)), [nexc] "s"((info >> 8) & 0x3Fu), [bl] "s"(lb), [ob] "s"(on),
                           [sc] "s"(((u64)sb << 32) | sb), [ib] "s"(IB), [z] "s"(ZB), [l4] "v"(lane * 4), [l128] "v"(lane + 128),
                           [e32] "v"(IB + lane * 32), [p0] "v"(IB + lane * 128),
                           [w0] "v"(ob), [w1] "v"(ob + NN * 4u), [w2] "v"(ob + 2u * NN * 4u), [w3] "v"(ob + 3u * NN * 4u),
                           [w4] "v"(ob + 256u), [w5] "v"(ob + NN * 4u + 256u), [w6] "v"(ob + 2u * NN * 4u + 256u), [w7] "v"(ob + 3u * NN * 4u + 256u),
                           [k0f] "s"(0x0F0F0F0Fu), [e0] "s"(0x0c040100u), [e1] "s"(0x0c050100u), [e2] "s"(0x0c060100u), [e3] "s"(0x0c070100u),
                           [o0] "s"(0x0c040302u), [o1] "s"(0x0c050302u), [o2] "s"(0x0c060302u), [o3] "s"(0x0c070302u),
                           [g0] "s"(0x05010400u), [g1] "s"(0x07030602u),
                           [h0] "s"(0x05040100u), [h1] "s"(0x05040302u), [h2] "s"(0x07060100u), [h3] "s"(0x07060302u)
                         : CLOB_DEC, "vcc", "scc", "memory");
          }
          if (nn >= 512u) break;
          lb ^= 4096u;
          n = nn;
          if (!have) {
            if (!await(n)) { dead = true; break; }
            issue(n, info_next, lb);
            VM_WAIT(0);
          }
        }
      }
      if (dead) {
        const float nan = __builtin_nanf("");
        st_out(a.out, ob + (((n >> 7) * 128 + ((n >> 3) & 1u) * 64 + (n & 3u) * 16) * NN + ((n >> 4) & 7u) * 256 + ((n >> 2) & 1u) * 128) * 4u, f32x4{nan, nan, nan, nan});
      }
      STAMP(2);
    }
    }
  }
  VM_WAIT(0);
}

extern "C" void kernel_launch(int rank, int nranks, ncclComm_t comm,
                              hipStream_t stream,
                              void* const* d_in, const int* in_sizes, int n_in,
                              void* d_out, int out_size,
                              const int* in_dtypes, int out_dtype,
                              void* const* d_out_all,
                              void* const* d_ws_all, size_t ws_bytes) {
  (void)comm; (void)in_dtypes; (void)out_dtype; (void)d_out_all;
  if (nranks != NR || (unsigned)rank >= (unsigned)NR || n_in < 4 || ws_bytes < (size_t)WS_NEED ||
      in_sizes[0] != MM * KS || in_sizes[1] != KS * NN || out_size != (MM / NR) * NN) {
    fprintf(stderr, "kernel_launch: unexpected problem shape\n");
    return;
  }
  static bool attr_set[NR];
  if (!attr_set[rank]) {
    HIP_CHECK(hipFuncSetAttribute((const void*)gemm_rs_kernel, hipFuncAttributeMaxDynamicSharedMemorySize, (int)LDS_BYTES));
    attr_set[rank] = true;
  }
  Args a;
  a.x = (const char*)d_in[0]; a.w = (const char*)d_in[1];
  a.sx = (const float*)d_in[2]; a.sw = (const float*)d_in[3];
  a.out = (char*)d_out;
  for (int p = 0; p < NR; ++p) a.ws[p] = (char*)d_ws_all[p];
  a.rank = rank;
  hipLaunchKernelGGL(gemm_rs_kernel, dim3(256), dim3(512), LDS_BYTES, stream, a);
  HIP_CHECK(hipGetLastError());
}
```
